```python
import numpy as np

kernel_name = "gpu_gemm_rs_m4096_k4096_n16384_w8a8_dqsilu"


ABSMAX_RTOL = 0.001

COMPUTE_DTYPE = "int8"
EPILOGUE = "dequant_silu"
IN_QUANT = "int8"
OUT_QUANT = None
IN_DTYPES = {"x": "int8", "w_mat": "int8",
             "scale_x": "float32", "scale_w": "float32"}
WORLD_SIZE = 8

def setup_inputs():

    x = np.random.randint(-127, 128, size=(4096, 4096)).astype(np.int8)
    w_mat = np.random.randint(-127, 128, size=(4096, 16384)).astype(np.int8)
    scale_x = (0.02 * (1.0 + np.random.rand(1))).astype(np.float32)
    scale_w = (0.02 * (1.0 + np.random.rand(1))).astype(np.float32)
    return {"x": x, "w_mat": w_mat, "scale_x": scale_x, "scale_w": scale_w}

def reference(x, w_mat, scale_x, scale_w):
    acc = x.astype(np.float64) @ w_mat.astype(np.float64)
    y = (acc.astype(np.float32) * (scale_x * scale_w)).astype(np.float32)
    return (y / (1.0 + np.exp(-np.clip(y, -60.0, 60.0)))).astype(np.float32)
```

```cpp
#include <hip/hip_runtime.h>
#include <rccl/rccl.h>
#include <cstdio>

#define HIP_CHECK(expr) do { hipError_t _e = (expr); if (_e != hipSuccess) { \
    fprintf(stderr, "HIP error %s at %s:%d\n", hipGetErrorString(_e), __FILE__, __LINE__); \
    return; } } while (0)

#define OUT_PULL 1
#define PULL_U 2
#define PULL_NW 4

namespace {

typedef int v2i __attribute__((ext_vector_type(2)));
typedef int v4i __attribute__((ext_vector_type(4)));
typedef unsigned v4u __attribute__((ext_vector_type(4)));
typedef float v4f __attribute__((ext_vector_type(4)));
#define LDS_AS __attribute__((address_space(3)))
#define GLB_AS __attribute__((address_space(1)))

constexpr int NR = 8;
constexpr int NTOT = 16384;
constexpr int SAME_BLK = 7, CROSS_BLK = 9;
constexpr int SLOT_X0 = 3 * SAME_BLK;
constexpr int SLOT_OWN = SLOT_X0 + 4 * CROSS_BLK;
constexpr int NSLOT = SLOT_OWN + 16;
constexpr int NCC = 32;
constexpr int NDMA = 64;
constexpr int NWG = 256;
constexpr int PUSH_NW = 4;
constexpr int NV = 8 * PUSH_NW;
constexpr int IT_W0 = 2 * SLOT_OWN;
constexpr int IT_X1 = IT_W0 + 8 * NCC;
constexpr int IT_END = IT_X1 + 32;
constexpr unsigned SEQ_MUL = 1024;
constexpr unsigned SEQ_FINAL = 1023;

constexpr unsigned OFF_CNT = 0;
constexpr unsigned OFF_TMO = 8192;
constexpr unsigned OFF_PROG = 16384;
constexpr unsigned OFF_RDY = 32768;
constexpr unsigned OFF_DBG = 65536;
constexpr unsigned XS_SRC = NSLOT * 16384;
constexpr unsigned XS_PAR = NR * XS_SRC;
constexpr unsigned WS_SRC = NCC * 65536;
constexpr unsigned WS_PAR = NR * WS_SRC;
constexpr unsigned OFF_XS = 1u << 20;
constexpr unsigned OFF_WS = OFF_XS + 2 * XS_PAR;
constexpr unsigned OFF_OS = OFF_WS + 2 * WS_PAR;
constexpr unsigned WS_NEED = OFF_OS + NCC * 6 * 131072;
constexpr unsigned OUT_BYTES = 512u * NTOT * 4u;
constexpr unsigned X_BYTES = 4096u * 512u, W_BYTES = 512u * NTOT;

constexpr unsigned long long DEADLINE = 50000000ull;
constexpr int STAGE = 49152;
constexpr int AUX_SYS = 17;

struct Params {
    const char* x; const char* w; const float* sx; const float* sw;
    char* out[NR]; char* ws[NR];
    int rank;
};

__device__ __forceinline__ int rfl(int v) { return __builtin_amdgcn_readfirstlane(v); }
__device__ __forceinline__ auto rsrc(const void* p, unsigned bytes) {
    return __builtin_amdgcn_make_buffer_rsrc((void*)p, 0, bytes, 0x00020000);
}
template <int AUX, class R> __device__ __forceinline__ void st128(v4u v, R rs, unsigned off) {
    __builtin_amdgcn_raw_buffer_store_b128(v, rs, off, 0, AUX);
}
__device__ __forceinline__ void slot_owner(int c, int s, int& r, int& row) {
    if (s < SLOT_X0) { int j = s / SAME_BLK; r = (c & 4) | ((c + 1 + j) & 3); row = (s - j * SAME_BLK) * 32; }
    else if (s < SLOT_OWN) { int t = s - SLOT_X0, g = t / CROSS_BLK; r = ((c & 4) ^ 4) | g; row = SAME_BLK * 32 + (t - g * CROSS_BLK) * 32; }
    else { r = c; row = (s - SLOT_OWN) * 32; }
}
__device__ __forceinline__ int chunk_col(int c, int cc) {
    int g = cc & 3, gid = (c & 4) ? g * 4 + (c & 3) : (c & 3) * 4 + g;
    return gid * 1024 + (cc >> 2) * 128;
}
__device__ __forceinline__ void st_sys(char* p, unsigned v) {
    __hip_atomic_store((GLB_AS unsigned*)p, v, __ATOMIC_RELAXED, __HIP_MEMORY_SCOPE_SYSTEM);
}
__device__ __forceinline__ unsigned ld_tmo(char* ws) {
    return __hip_atomic_load((GLB_AS unsigned*)(ws + OFF_TMO), __ATOMIC_RELAXED, __HIP_MEMORY_SCOPE_AGENT);
}
__device__ __forceinline__ bool expired(char* ws, unsigned call, unsigned long long t0) {
    if (ld_tmo(ws) == call || __builtin_amdgcn_s_memrealtime() - t0 > DEADLINE) {
        __hip_atomic_store((GLB_AS unsigned*)(ws + OFF_TMO), call, __ATOMIC_RELAXED, __HIP_MEMORY_SCOPE_AGENT);
        return true;
    }
    return false;
}
#define WAIT_VM0() asm volatile("s_waitcnt vmcnt(0)" ::: "memory")

#define STAMP(ws, idx) do {} while (0)

__device__ __forceinline__ void wait_items(char* ws, unsigned call, int T, int rank, int lane) {
    auto rs = rsrc(ws + OFF_PROG, 1024);
    unsigned need[4];
    for (int j = 0; j < 4; ++j) {
        int v = (lane & 7) * 4 + j;
        need[j] = call * SEQ_MUL + (T < 0 ? SEQ_FINAL : (unsigned)((T - v + NV - 1) / NV));
        if (T < 0 && (lane >> 3) == rank) need[j] = 0;
    }
    unsigned long long t0 = __builtin_amdgcn_s_memrealtime();
    for (;;) {
        asm volatile("" ::: "memory");
        v4u f = __builtin_amdgcn_raw_buffer_load_b128(rs, lane * 16, 0, AUX_SYS);
        bool ok = f[0] >= need[0] && f[1] >= need[1] && f[2] >= need[2] && f[3] >= need[3];
        if (__all(ok)) break;
        if (__any(expired(ws, call, t0))) break;
        __builtin_amdgcn_s_sleep(8);
    }
}

__device__ __forceinline__ float silu(int acc, float s) {
    float y = (float)acc * s;
    float c = fminf(fmaxf(y, -60.0f), 60.0f);
    float e = __builtin_amdgcn_exp2f(c * -1.4426950408889634f);
    return y * __builtin_amdgcn_rcpf(1.0f + e);
}

__global__ void __launch_bounds__(512, 2) fused_gemm_scatter(Params P) {
    __shared__ __attribute__((aligned(16))) char smem[3 * STAGE];
    const int rank = P.rank;
    const int lane = threadIdx.x & 63, wid = rfl(threadIdx.x >> 6);
    char* const ws = P.ws[rank];
    const unsigned call = (unsigned)rfl((int)(lane == 0 ? __hip_atomic_fetch_add((GLB_AS unsigned*)(ws + OFF_CNT) + blockIdx.x * 8 + wid,
                                           1u, __ATOMIC_RELAXED, __HIP_MEMORY_SCOPE_AGENT) : 0u)) + 1u;
    const unsigned par = call & 1u;
    const int xcd = blockIdx.x & 7;
    STAMP(ws, 0);

    if (blockIdx.x < NDMA) {
        const int peer = (rank + (blockIdx.x >> 3)) & 7;
        char* const pws = P.ws[peer];
        const auto prs = rsrc(pws, WS_NEED);
        char* const prog = pws + OFF_PROG + (rank * NV + xcd * PUSH_NW + wid) * 4;
        if (wid < PUSH_NW) {
            const auto xrs = rsrc(P.x, X_BYTES);
            const auto wrs = rsrc(P.w, W_BYTES);
            const unsigned lin = lane * 16;
            const unsigned wl = (lane >> 3) * NTOT + (lane & 7) * 16;
            unsigned cnt = 0;
            for (int it = xcd * PUSH_NW + wid; it < IT_END; it += NV) {
                const bool isw = it >= IT_W0 && it < IT_X1;
                unsigned so, dof;
                if (isw) {
                    int cc = (it - IT_W0) >> 3, kb = (it - IT_W0) & 7;
                    so = kb * 64 * NTOT + chunk_col(peer, cc);
                    dof = OFF_WS + par * WS_PAR + rank * WS_SRC + cc * 65536 + kb * 8192;
                } else {
                    int h = it < IT_W0 ? it : it - IT_X1 + IT_W0, r, row;
                    slot_owner(peer, h >> 1, r, row);
                    so = (r * 512 + row + (h & 1) * 16) * 512;
                    dof = OFF_XS + par * XS_PAR + rank * XS_SRC + h * 8192;
                }
#pragma unroll
                for (int h = 0; h < 2; ++h) {
                    v4u v[4];
                    if (isw) {
#pragma unroll
                        for (int u = 0; u < 4; ++u) v[u] = __builtin_amdgcn_raw_buffer_load_b128(wrs, wl, so + (h * 4 + u) * 8 * NTOT, 0);
                    } else {
#pragma unroll
                        for (int u = 0; u < 4; ++u) v[u] = __builtin_amdgcn_raw_buffer_load_b128(xrs, lin, so + (h * 4 + u) * 1024, 0);
                    }
                    WAIT_VM0();
#pragma unroll
                    for (int u = 0; u < 4; ++u) st128<AUX_SYS>(v[u], prs, lin + dof + (h * 4 + u) * 1024);
                    if (h == 0 && cnt && lane == 0) st_sys(prog, call * SEQ_MUL + cnt);
                }
                ++cnt;
            }
            WAIT_VM0();
            if (lane == 0) st_sys(prog, call * SEQ_MUL + cnt);
        }
        STAMP(ws, 1);
        if (peer != rank && wid < PULL_NW) {
            const int c = peer, r = rank;
            const auto srs = prs;
            const auto frs = rsrc(ws + OFF_RDY + c * 1024, 1024);
            constexpr int LAUX = AUX_SYS, SAUX = 0;
            const auto ors = rsrc(P.out[r], OUT_BYTES);
            const bool same = ((c ^ r) & 4) == 0;
            const int nblk = same ? SAME_BLK : CROSS_BLK;
            const int slot0 = same ? SAME_BLK * (((r - c) & 3) - 1) : SLOT_X0 + CROSS_BLK * (r & 3);
            const int row0 = same ? 0 : SAME_BLK * 32;
            constexpr int PARTS = 16 / PULL_U;
            const int nunits = (same ? NCC : 24) * nblk * PARTS;
            bool chk[4];
            {
                int g = (lane >> 1) & 3;
                int e0 = (same || slot0 < SLOT_X0 + CROSS_BLK * g) ? slot0 : slot0 - CROSS_BLK;
                bool valid = same || g != (r & 3);
                for (int j = 0; j < 4; ++j) { int rt = (lane & 1) * 4 + j; chk[j] = valid && rt >= (e0 >> 3) && rt <= ((e0 + nblk - 1) >> 3); }
            }
            int ready = 0;
            bool dead = false;
            const unsigned ol = (lane >> 5) * (NTOT * 4) + (lane & 31) * 16;
            for (int un = xcd * PULL_NW + wid; un < nunits; un += 8 * PULL_NW) {
                int part = un % PARTS, t = un / PARTS;
                int ci = t / nblk, blk = t - ci * nblk, cc = ci;
                if (!same) { int sub = ci / 3, gi = ci - sub * 3; cc = sub * 4 + (gi < (r & 3) ? gi : gi + 1); }
                if (cc >= ready && !dead) {
                    unsigned long long t0 = __builtin_amdgcn_s_memrealtime();
                    for (;;) {
                        asm volatile("" ::: "memory");
                        v4u f = __builtin_amdgcn_raw_buffer_load_b128(frs, lane * 16, 0, AUX_SYS);
                        bool miss = (chk[0] && f[0] < call) || (chk[1] && f[1] < call) || (chk[2] && f[2] < call) || (chk[3] && f[3] < call);
                        unsigned long long m = __ballot(miss);
                        ready = m ? (__builtin_ctzll(m) >> 1) : NCC;
                        if (cc < ready) break;
                        if (__any(expired(ws, call, t0))) { dead = true; break; }
                        __builtin_amdgcn_s_sleep(4);
                    }
                    asm volatile("" ::: "memory");
                }
                int s = slot0 + blk, g = cc & 3;
                int e = (same || s < SLOT_X0 + CROSS_BLK * g) ? s : s - CROSS_BLK;
                unsigned so = OFF_OS + (cc * 48 + e) * 16384 + part * (PULL_U * 1024);
                unsigned dof = (unsigned)(row0 + blk * 32 + part * (2 * PULL_U)) * (NTOT * 4) + chunk_col(c, cc) * 4;
                v4u v[PULL_U];
#pragma unroll
                for (int u = 0; u < PULL_U; ++u) v[u] = __builtin_amdgcn_raw_buffer_load_b128(srs, lane * 16, so + u * 1024, LAUX);
                if (dead) {
#pragma unroll
                    for (int u = 0; u < PULL_U; ++u) v[u] = v4u{0x7fc00000u, 0x7fc00000u, 0x7fc00000u, 0x7fc00000u};
                }
#pragma unroll
                for (int u = 0; u < PULL_U; ++u) st128<SAUX>(v[u], ors, ol + dof + u * 2 * (NTOT * 4));
            }
        }
        STAMP(ws, 2);
        return;
    }

    const int li = (blockIdx.x - NDMA) >> 3;
    const int ntiles = (xcd < 4 && li < 16) ? 2 : 1;
    const float scale = P.sx[0] * P.sw[0];
    const int wm = wid >> 1, wn = wid & 1, fr = lane & 15, fq = lane >> 4;
    const unsigned lds0 = (unsigned)(size_t)(LDS_AS char*)smem;
    unsigned xa[2], wa[4];
    for (int kh = 0; kh < 2; ++kh) xa[kh] = (wm * 64 + fr) * 128 + (((kh * 4 + fq) ^ (fr >> 1)) << 4);
    for (int ni = 0; ni < 4; ++ni)
        wa[ni] = 32768 + (fq * 16 + (fr >> 1)) * 128 + (((wn * 4 + ni) ^ (((fr >> 2) & 3) | ((fq & 1) << 2))) << 4) + 8 * (fr & 1);

    for (int ti = 0; ti < ntiles; ++ti) {
        int cc, rt;
        if (ti == 0) { cc = xcd * 4 + li / 6; rt = li % 6; }
        else { int idx = xcd * 16 + li; cc = idx >> 1; rt = 6 + (idx & 1); }
        const int g = cc & 3;
        auto slot_of = [&](int b) {
            int e = rt * 8 + b;
            if (rt >= 6) return SLOT_OWN + e - 48;
            return (e < SLOT_X0 + CROSS_BLK * g) ? e : e + CROSS_BLK;
        };
        if (wid == 0) {
            wait_items(ws, call, rt < 6 ? IT_W0 + 8 * (cc + 1) : IT_END, rank, lane);
            __builtin_amdgcn_fence(__ATOMIC_ACQUIRE, "");
            WAIT_VM0();
        }
        __syncthreads();
        const bool dead = rfl(ld_tmo(ws)) == (int)call;
        STAMP(ws, 1 + ti * 3);

        v4i acc[4][4];
#pragma unroll
        for (int i = 0; i < 4; ++i)
#pragma unroll
            for (int j = 0; j < 4; ++j) acc[i][j] = v4i{0, 0, 0, 0};

        if (!dead) {
            unsigned xo[4], wo[2];
#pragma unroll
            for (int i = 0; i < 4; ++i) {
                int row = i * 64 + wid * 8 + (lane >> 3);
                xo[i] = slot_of(i * 2 + (wid >> 2)) * 16384 + (row & 31) * 512 + (((lane & 7) ^ ((row >> 1) & 7)) << 4);
            }
#pragma unroll
            for (int i = 0; i < 2; ++i) {
                int k = i * 64 + wid * 8 + (lane >> 3);
                wo[i] = k * 128 + (((lane & 7) ^ (((k >> 1) & 3) | (((k >> 4) & 1) << 2))) << 4);
            }
            const char* xs = ws + OFF_XS + par * XS_PAR;
            const char* wsb = ws + OFF_WS + par * WS_PAR + cc * 65536;
            auto stage = [&](int t, int buf) {
                const char* xb = xs + (t >> 2) * XS_SRC + (t & 3) * 128;
                const char* wb = wsb + (t >> 2) * WS_SRC + (t & 3) * 16384;
                const unsigned l = lds0 + buf * STAGE + wid * 1024;
                unsigned keep;
                asm volatile(
                    "s_mov_b32 %0, m0\n\t"
                    "s_mov_b32 m0, %1\n\ts_nop 0\n\tglobal_load_lds_dwordx4 %4, %2\n\t"
                    "s_add_u32 m0, m0, 0x2000\n\ts_nop 0\n\tglobal_load_lds_dwordx4 %5, %2\n\t"
                    "s_add_u32 m0, m0, 0x2000\n\ts_nop 0\n\tglobal_load_lds_dwordx4 %6, %2\n\t"
                    "s_add_u32 m0, m0, 0x2000\n\ts_nop 0\n\tglobal_load_lds_dwordx4 %7, %2\n\t"
                    "s_add_u32 m0, m0, 0x2000\n\ts_nop 0\n\tglobal_load_lds_dwordx4 %8, %3\n\t"
                    "s_add_u32 m0, m0, 0x2000\n\ts_nop 0\n\tglobal_load_lds_dwordx4 %9, %3\n\t"
                    "s_mov_b32 m0, %0"
                    : "=&s"(keep)
                    : "s"(l), "s"(xb), "s"(wb), "v"(xo[0]), "v"(xo[1]), "v"(xo[2]), "v"(xo[3]), "v"(wo[0]), "v"(wo[1])
                    : "memory", "scc");
            };
            stage(0, 0);
            stage(1, 1);
            int buf = 0;
            for (int t = 0; t < 32; ++t) {
                if (t < 31) asm volatile("s_waitcnt vmcnt(6) lgkmcnt(0)" ::: "memory");
                else asm volatile("s_waitcnt vmcnt(0) lgkmcnt(0)" ::: "memory");
                __builtin_amdgcn_s_barrier();
                asm volatile("" ::: "memory");
                if (t < 30) stage(t + 2, buf == 0 ? 2 : buf - 1);
                LDS_AS char* sb = (LDS_AS char*)smem + buf * STAGE;
#pragma unroll
                for (int kh = 0; kh < 2; ++kh) {
                    v4i xf[4], wf[4];
#pragma unroll
                    for (int ni = 0; ni < 4; ++ni) {
                        v2i lo = __builtin_amdgcn_ds_read_tr8_b64_v2i32((LDS_AS v2i*)(sb + wa[ni] + kh * 8192));
                        v2i hi = __builtin_amdgcn_ds_read_tr8_b64_v2i32((LDS_AS v2i*)(sb + wa[ni] + kh * 8192 + 1024));
                        wf[ni] = v4i{lo[0], lo[1], hi[0], hi[1]};
                    }
#pragma unroll
                    for (int mi = 0; mi < 4; ++mi) xf[mi] = *(LDS_AS v4i*)(sb + xa[kh] + mi * 2048);
#pragma unroll
                    for (int mi = 0; mi < 4; ++mi)
#pragma unroll
                        for (int ni = 0; ni < 4; ++ni)
                            acc[mi][ni] = __builtin_amdgcn_mfma_i32_16x16x64_i8(wf[ni], xf[mi], acc[mi][ni], 0, 0, 0);
                }
                buf = buf == 2 ? 0 : buf + 1;
            }
        }
        STAMP(ws, 2 + ti * 3);
        {
            const bool loc = rt >= 6;
            const float sc = dead ? __builtin_nanf("") : scale;
            const auto rs = loc ? rsrc(P.out[rank], OUT_BYTES) : rsrc(ws, WS_NEED);
            const unsigned pitch = loc ? NTOT * 4 : 512;
            const unsigned base = loc ? (unsigned)(rt - 6) * 256 * (NTOT * 4) + chunk_col(rank, cc) * 4 : OFF_OS + (cc * 6 + rt) * 131072;
            const unsigned lo = (wm * 64 + fr) * pitch + wn * 256 + fq * 16;
#pragma unroll
            for (int mi = 0; mi < 4; ++mi)
#pragma unroll
                for (int ni = 0; ni < 4; ++ni) {
                    v4f o;
#pragma unroll
                    for (int j = 0; j < 4; ++j) o[j] = silu(acc[mi][ni][j], sc);
                    st128<AUX_SYS>(__builtin_bit_cast(v4u, o), rs, lo + base + mi * 16 * pitch + ni * 64);
                }
            WAIT_VM0();
            __syncthreads();
            if (!loc && threadIdx.x < 8) {
                int r, row;
                slot_owner(rank, slot_of(threadIdx.x), r, row);
                st_sys(P.ws[r & 7] + OFF_RDY + ((rank * NCC + cc) * 8 + rt) * 4, call);
            }
        }
        STAMP(ws, 3 + ti * 3);
    }
}

}

extern "C" void kernel_launch(int rank, int nranks, ncclComm_t comm,
                              hipStream_t stream,
                              void* const* d_in, const int* in_sizes, int n_in,
                              void* d_out, int out_size,
                              const int* in_dtypes, int out_dtype,
                              void* const* d_out_all,
                              void* const* d_ws_all, size_t ws_bytes) {
    if (nranks != NR || (unsigned)rank >= (unsigned)NR || ws_bytes < WS_NEED || n_in != 4 ||
        in_sizes[0] != (int)X_BYTES || in_sizes[1] != (int)W_BYTES || out_size != 512 * NTOT) {
        fprintf(stderr, "unexpected problem shape\n");
        return;
    }
    Params p;
    p.x = (const char*)d_in[0]; p.w = (const char*)d_in[1];
    p.sx = (const float*)d_in[2]; p.sw = (const float*)d_in[3];
    for (int i = 0; i < NR; ++i) { p.out[i] = (char*)d_out_all[i]; p.ws[i] = (char*)d_ws_all[i]; }
    p.out[rank] = (char*)d_out;
    p.rank = rank;
    hipLaunchKernelGGL(fused_gemm_scatter, dim3(NWG), dim3(512), 0, stream, p);
    HIP_CHECK(hipGetLastError());
}
```

```cpp
#include <hip/hip_runtime.h>
#include <rccl/rccl.h>
#include <cstdio>

#define HIP_CHECK(expr) do { hipError_t _e = (expr); if (_e != hipSuccess) { \
    fprintf(stderr, "HIP error %s at %s:%d\n", hipGetErrorString(_e), __FILE__, __LINE__); \
    return; } } while (0)

#define PULL_U 4
#define PULL_NW 2
#define PUSH_NW 5
#define GATE_ITEMS 1
#define GATE_MS MS_XO
#define OUT_AUX 16

namespace {

typedef int v2i __attribute__((ext_vector_type(2)));
typedef int v4i __attribute__((ext_vector_type(4)));
typedef unsigned v4u __attribute__((ext_vector_type(4)));
typedef float v4f __attribute__((ext_vector_type(4)));
#define LDS_AS __attribute__((address_space(3)))
#define GLB_AS __attribute__((address_space(1)))
typedef GLB_AS unsigned gu32;

constexpr int NR = 8;
constexpr int NTOT = 16384;
constexpr int SAME_BLK = 7, CROSS_BLK = 9;
constexpr int SLOT_A = 28;
constexpr int SLOT_OWN = 57;
constexpr int NSLOT = SLOT_OWN + 16;
constexpr int NCC = 32;
constexpr int CC_A = 8;
constexpr int NDMA = 64;
constexpr int NWG = 256;
constexpr int NV = 8 * PUSH_NW;
constexpr int NL = 8 * PULL_NW;
constexpr int IT_WA = 2 * SLOT_A;
constexpr int IT_XB = IT_WA + 8 * CC_A;
constexpr int IT_WB = IT_XB + 2 * (SLOT_OWN - SLOT_A);
constexpr int IT_XO = IT_WB + 8 * (NCC - CC_A);
constexpr int IT_END = IT_XO + 32;
constexpr int MS_XA = 32, MS_XB = 33, MS_XO = 34;

constexpr unsigned OFF_CNT = 0;
constexpr unsigned OFF_TMO = 8192;
constexpr unsigned OFF_Q = 12288;
constexpr unsigned OFF_RDY = 16384;
constexpr unsigned OFF_DBG = 65536;
constexpr unsigned OFF_MS = 262144;
constexpr unsigned XS_SRC = NSLOT * 16384;
constexpr unsigned XS_PAR = NR * XS_SRC;
constexpr unsigned WS_SRC = NCC * 65536;
constexpr unsigned WS_PAR = NR * WS_SRC;
constexpr unsigned OFF_XS = 1u << 20;
constexpr unsigned OFF_WS = OFF_XS + 2 * XS_PAR;
constexpr unsigned OFF_OS = OFF_WS + 2 * WS_PAR;
constexpr unsigned WS_NEED = OFF_OS + NCC * 6 * 131072;
constexpr unsigned OUT_BYTES = 512u * NTOT * 4u;
constexpr unsigned X_BYTES = 4096u * 512u, W_BYTES = 512u * NTOT;

constexpr unsigned long long DEADLINE = 50000000ull;
constexpr int STAGE = 49152;
constexpr int AUX_SYS = 17;

struct Params {
    const char* x; const char* w; const float* sx; const float* sw;
    char* out; char* ws[NR];
    int rank;
};

__device__ __forceinline__ int rfl(int v) { return __builtin_amdgcn_readfirstlane(v); }
__device__ __forceinline__ auto rsrc(const void* p, unsigned bytes) {
    return __builtin_amdgcn_make_buffer_rsrc((void*)p, 0, bytes, 0x00020000);
}
template <int AUX, class R> __device__ __forceinline__ void st128(v4u v, R rs, unsigned off) {
    __builtin_amdgcn_raw_buffer_store_b128(v, rs, off, 0, AUX);
}
__device__ __forceinline__ int dest_rank(int c, int d) { return d < 3 ? (c & 4) | ((c + 1 + d) & 3) : ((c & 4) ^ 4) | (d - 3); }
__device__ __forceinline__ int slot_row(int c, int s) {
    if (s >= SLOT_OWN) return c * 512 + (s - SLOT_OWN) * 32;
    int j, d;
    if (s < 49) { j = s / 7; d = s - j * 7; } else { j = 7 + ((s - 49) >> 2); d = 3 + ((s - 49) & 3); }
    return dest_rank(c, d) * 512 + (d < 3 ? 0 : SAME_BLK * 32) + j * 32;
}
__device__ __forceinline__ int entry_of(int g, int d, int j) {
    int i = d < 3 + g ? d : d - 1;
    return j < 7 ? j * 6 + i : 42 + (j - 7) * 3 + (i - 3);
}
__device__ __forceinline__ void entry_inv(int g, int e, int& d, int& slot) {
    int j, i;
    if (e < 42) { j = e / 6; i = e - j * 6; } else { j = 7 + (e - 42) / 3; i = 3 + (e - 42) % 3; }
    d = i < 3 + g ? i : i + 1;
    slot = j < 7 ? j * 7 + d : 49 + (j - 7) * 4 + (d - 3);
}
__device__ __forceinline__ int chunk_col(int c, int cc) {
    int g = cc & 3, gid = (c & 4) ? g * 4 + (c & 3) : (c & 3) * 4 + g;
    return gid * 1024 + (cc >> 2) * 128;
}
__device__ __forceinline__ unsigned ld_sys(const char* p) {
    return __hip_atomic_load((gu32*)p, __ATOMIC_RELAXED, __HIP_MEMORY_SCOPE_SYSTEM);
}
__device__ __forceinline__ unsigned ld_tmo(char* ws) {
    return __hip_atomic_load((gu32*)(ws + OFF_TMO), __ATOMIC_RELAXED, __HIP_MEMORY_SCOPE_AGENT);
}
__device__ __forceinline__ bool expired(char* ws, unsigned call, unsigned long long t0) {
    if (ld_tmo(ws) == call || __builtin_amdgcn_s_memrealtime() - t0 > DEADLINE) {
        __hip_atomic_store((gu32*)(ws + OFF_TMO), call, __ATOMIC_RELAXED, __HIP_MEMORY_SCOPE_AGENT);
        return true;
    }
    return false;
}
template <int SLEEP> __device__ __forceinline__ bool wait_words(char* ws, unsigned call, const char* p, unsigned target) {
    unsigned long long t0 = __builtin_amdgcn_s_memrealtime();
    while (!__all((int)(ld_sys(p) - target) >= 0)) {
        if (__any(expired(ws, call, t0))) return false;
        __builtin_amdgcn_s_sleep(SLEEP);
    }
    return true;
}
#define WAIT_VM0() asm volatile("s_waitcnt vmcnt(0)" ::: "memory")

#define STAMP(ws, idx) do {} while (0)

__device__ __forceinline__ float silu(int acc, float s) {
    float y = (float)acc * s;
    float c = fminf(fmaxf(y, -60.0f), 60.0f);
    float e = __builtin_amdgcn_exp2f(c * -1.4426950408889634f);
    return y * __builtin_amdgcn_rcpf(1.0f + e);
}
__device__ __forceinline__ unsigned ticket(char* p, int lane) {
    unsigned v = 0, z = 0;
    asm volatile("" : "+v"(z));
    if (lane == 0) v = __hip_atomic_fetch_add((gu32*)(p + z), 1u, __ATOMIC_RELAXED, __HIP_MEMORY_SCOPE_AGENT);
    return v;
}

__global__ void __launch_bounds__(512, 2) fused_gemm_scatter(Params P) {
    __shared__ __attribute__((aligned(16))) char smem[3 * STAGE];
    const int rank = P.rank;
    const int lane = threadIdx.x & 63, wid = rfl(threadIdx.x >> 6);
    char* const ws = P.ws[rank];
    const int xcd = blockIdx.x & 7;
    STAMP(ws, 0);
    const unsigned callv = ticket(ws + OFF_CNT + (blockIdx.x * 8 + wid) * 4, lane);

    if (blockIdx.x < NDMA) {
        const int ps = blockIdx.x >> 3, peer = (rank + ps) & 7;
        char* const pws = P.ws[peer];
        const auto prs = rsrc(pws, WS_NEED);
        unsigned call = 0;
        if (wid < PUSH_NW) {
            const auto xrs = rsrc(P.x, X_BYTES);
            const auto wrs = rsrc(P.w, W_BYTES);
            const unsigned lin = lane * 16;
            const unsigned wl = (lane >> 3) * NTOT + (lane & 7) * 16;
            char* const q = ws + OFF_Q + ps * 128;
            char* const msb = pws + OFF_MS + rank * 128;
            int prev = -1;
            unsigned it = xcd * PUSH_NW + wid;
            while (it < IT_END) {
                const unsigned nxt = ticket(q, lane);
                const bool isw = (it >= IT_WA && it < IT_XB) || (it >= IT_WB && it < IT_XO);
                unsigned so, dof;
                int ms;
                if (isw) {
                    int i = it < IT_XB ? it - IT_WA : it - IT_WB + 8 * CC_A;
                    ms = i >> 3;
                    so = (i & 7) * 64 * NTOT + chunk_col(peer, ms);
                    dof = OFF_WS + rank * WS_SRC + i * 8192;
                } else {
                    int h = it < IT_WA ? it : it < IT_WB ? it - 8 * CC_A : it - 8 * NCC;
                    ms = it < IT_WA ? MS_XA : it < IT_WB ? MS_XB : MS_XO;
                    so = (slot_row(peer, h >> 1) + (h & 1) * 16) * 512;
                    dof = OFF_XS + rank * XS_SRC + h * 8192;
                }
#pragma unroll
                for (int h = 0; h < 2; ++h) {
                    v4u v[4];
                    if (isw) {
#pragma unroll
                        for (int u = 0; u < 4; ++u) v[u] = __builtin_amdgcn_raw_buffer_load_b128(wrs, wl, so + (h * 4 + u) * 8 * NTOT, 0);
                    } else {
#pragma unroll
                        for (int u = 0; u < 4; ++u) v[u] = __builtin_amdgcn_raw_buffer_load_b128(xrs, lin, so + (h * 4 + u) * 1024, 0);
                    }
                    WAIT_VM0();
                    if (h == 0) {
                        call = (unsigned)rfl((int)callv) + 1u;
                        dof += (call & 1u) * (isw ? WS_PAR : XS_PAR);
                    }
#pragma unroll
                    for (int u = 0; u < 4; ++u) st128<AUX_SYS>(v[u], prs, lin + dof + (h * 4 + u) * 1024);
                    if (h == 0 && prev >= 0 && lane == 0)
                        __hip_atomic_fetch_add((gu32*)(msb + prev * 1024), 1u, __ATOMIC_RELAXED, __HIP_MEMORY_SCOPE_SYSTEM);
                }
                prev = ms;
                it = (unsigned)rfl((int)nxt) - (call - 1u) * IT_END + NV;
            }
            WAIT_VM0();
            if (lane == 0) __hip_atomic_fetch_add((gu32*)(msb + prev * 1024), 1u, __ATOMIC_RELAXED, __HIP_MEMORY_SCOPE_SYSTEM);
        }
        STAMP(ws, 1);
        if (peer != rank && wid >= 8 - PULL_NW) {
            call = (unsigned)rfl((int)callv) + 1u;
            const int c = peer;
            const auto frs = rsrc(ws + OFF_RDY + c * 1024, 1024);
            const auto ors = rsrc(P.out, OUT_BYTES);
            const bool same = ((c ^ rank) & 4) == 0;
            const int nblk = same ? SAME_BLK : CROSS_BLK;
            const int d = same ? ((rank - c) & 3) - 1 : 3 + (rank & 3);
            const int row0 = same ? 0 : SAME_BLK * 32;
            constexpr int PARTS = 16 / PULL_U;
            const int nca = same ? CC_A : CC_A / 4 * 3, ncb = same ? NCC - CC_A : (NCC - CC_A) / 4 * 3;
            const int nb1 = nca * 4, nb2 = nca * nblk;
            const unsigned nunits = (nca + ncb) * nblk * PARTS;
            char* const q = ws + OFF_Q + 1024 + ps * 128;
            bool dead = !wait_words<4>(ws, call, ws + OFF_MS + GATE_MS * 1024 + c * 128, (call - 1u) * (GATE_MS == MS_XO ? 32u : 8u) + GATE_ITEMS);
            STAMP(ws, 3);
            unsigned long long rdy[4] = {0, 0, 0, 0};
            const unsigned ol = (lane >> 5) * (NTOT * 4) + (lane & 31) * 16;
            unsigned un = xcd * PULL_NW + (wid - (8 - PULL_NW));
            while (un < nunits) {
                const unsigned nxt = ticket(q, lane);
                int part = un % PARTS, t = un / PARTS, ci, blk;
                if (t < nb1) { ci = t >> 2; blk = t & 3; }
                else if (t < nb2) { t -= nb1; ci = t / (nblk - 4); blk = 4 + t - ci * (nblk - 4); }
                else { ci = t / nblk; blk = t - ci * nblk; }
                int cc = ci;
                if (!same) { int sub = ci / 3, gi = ci - sub * 3; cc = sub * 4 + (gi < (rank & 3) ? gi : gi + 1); }
                const int e = entry_of(cc & 3, d, blk), rt = e >> 3;
                const int bit = cc * 2 + (rt >> 2);
                auto have = [&]() {
                    unsigned long long m = (rt & 2) ? ((rt & 1) ? rdy[3] : rdy[2]) : ((rt & 1) ? rdy[1] : rdy[0]);
                    return (m >> bit) & 1;
                };
                if (!dead && !have()) {
                    unsigned long long t0 = __builtin_amdgcn_s_memrealtime();
                    for (;;) {
                        asm volatile("" ::: "memory");
                        v4u f = __builtin_amdgcn_raw_buffer_load_b128(frs, lane * 16, 0, AUX_SYS);
#pragma unroll
                        for (int j = 0; j < 4; ++j) rdy[j] = __ballot((int)(f[j] - call) >= 0);
                        if (have()) break;
                        if (__any(expired(ws, call, t0))) { dead = true; break; }
                        __builtin_amdgcn_s_sleep(4);
                    }
                    asm volatile("" ::: "memory");
                }
                unsigned so = OFF_OS + (cc * 48 + e) * 16384 + part * (PULL_U * 1024);
                unsigned dof = (unsigned)(row0 + blk * 32 + part * (2 * PULL_U)) * (NTOT * 4) + chunk_col(c, cc) * 4;
                v4u v[PULL_U];
#pragma unroll
                for (int u = 0; u < PULL_U; ++u) v[u] = __builtin_amdgcn_raw_buffer_load_b128(prs, lane * 16, so + u * 1024, AUX_SYS);
                if (dead) {
#pragma unroll
                    for (int u = 0; u < PULL_U; ++u) v[u] = v4u{0x7fc00000u, 0x7fc00000u, 0x7fc00000u, 0x7fc00000u};
                }
#pragma unroll
                for (int u = 0; u < PULL_U; ++u) st128<OUT_AUX>(v[u], ors, ol + dof + u * 2 * (NTOT * 4));
                un = (unsigned)rfl((int)nxt) - (call - 1u) * nunits + NL;
            }
        }
        STAMP(ws, 2);
        return;
    }

    const unsigned call = (unsigned)rfl((int)callv) + 1u;
    const unsigned par = call & 1u;
    const int li = (blockIdx.x - NDMA) >> 3;
    const int ntiles = (xcd < 4 && li < 16) ? 2 : 1;
    const float scale = P.sx[0] * P.sw[0];
    const int wm = wid >> 1, wn = wid & 1, fr = lane & 15, fq = lane >> 4;
    const unsigned lds0 = (unsigned)(size_t)(LDS_AS char*)smem;
    unsigned xa[2], wa[4];
    for (int kh = 0; kh < 2; ++kh) xa[kh] = (wm * 64 + fr) * 128 + (((kh * 4 + fq) ^ (fr >> 1)) << 4);
    for (int ni = 0; ni < 4; ++ni)
        wa[ni] = 32768 + (fq * 16 + (fr >> 1)) * 128 + (((wn * 4 + ni) ^ (((fr >> 2) & 3) | ((fq & 1) << 2))) << 4) + 8 * (fr & 1);

    for (int ti = 0; ti < ntiles; ++ti) {
        int cc, rt;
        if (ti != 0) { int idx = xcd * 16 + li; cc = idx >> 1; rt = 6 + (idx & 1); }
        else if (xcd < 2) { cc = li / 3; rt = xcd * 3 + li % 3; }
        else { cc = xcd * 4 + li / 6; rt = li % 6; }
        cc = rfl(cc); rt = rfl(rt);
        const int g = cc & 3;
        if (wid == 0) {
            const int msx = rt < 3 ? MS_XA : rt < 6 ? MS_XB : MS_XO;
            const unsigned nx = rt < 3 ? IT_WA : rt < 6 ? IT_WB - IT_XB : IT_END - IT_XO;
            const bool lx = (lane & 8) != 0;
            wait_words<16>(ws, call, ws + OFF_MS + (lx ? msx : cc) * 1024 + (lane & 7) * 128, call * (lx ? nx : 8u));
            __builtin_amdgcn_fence(__ATOMIC_ACQUIRE, "");
            WAIT_VM0();
        }
        __syncthreads();
        const bool dead = rfl(ld_tmo(ws)) == (int)call;
        STAMP(ws, 1 + ti * 3);

        v4i acc[4][4];
#pragma unroll
        for (int i = 0; i < 4; ++i)
#pragma unroll
            for (int j = 0; j < 4; ++j) acc[i][j] = v4i{0, 0, 0, 0};

        if (!dead) {
            unsigned xo[4], wo[2];
#pragma unroll
            for (int i = 0; i < 4; ++i) {
                int row = i * 64 + wid * 8 + (lane >> 3), e = rt * 8 + i * 2 + (wid >> 2), d, slot;
                if (rt >= 6) slot = SLOT_OWN + e - 48; else entry_inv(g, e, d, slot);
                xo[i] = slot * 16384 + (row & 31) * 512 + (((lane & 7) ^ ((row >> 1) & 7)) << 4);
            }
#pragma unroll
            for (int i = 0; i < 2; ++i) {
                int k = i * 64 + wid * 8 + (lane >> 3);
                wo[i] = k * 128 + (((lane & 7) ^ (((k >> 1) & 3) | (((k >> 4) & 1) << 2))) << 4);
            }
            const char* xs = ws + OFF_XS + par * XS_PAR;
            const char* wsb = ws + OFF_WS + par * WS_PAR + cc * 65536;
            auto stage = [&](int t, int buf) {
                const char* xb = xs + (t >> 2) * XS_SRC + (t & 3) * 128;
                const char* wb = wsb + (t >> 2) * WS_SRC + (t & 3) * 16384;
                const unsigned l = lds0 + buf * STAGE + wid * 1024;
                unsigned keep;
                asm volatile(
                    "s_mov_b32 %0, m0\n\t"
                    "s_mov_b32 m0, %1\n\ts_nop 0\n\tglobal_load_lds_dwordx4 %4, %2\n\t"
                    "s_add_u32 m0, m0, 0x2000\n\ts_nop 0\n\tglobal_load_lds_dwordx4 %5, %2\n\t"
                    "s_add_u32 m0, m0, 0x2000\n\ts_nop 0\n\tglobal_load_lds_dwordx4 %6, %2\n\t"
                    "s_add_u32 m0, m0, 0x2000\n\ts_nop 0\n\tglobal_load_lds_dwordx4 %7, %2\n\t"
                    "s_add_u32 m0, m0, 0x2000\n\ts_nop 0\n\tglobal_load_lds_dwordx4 %8, %3\n\t"
                    "s_add_u32 m0, m0, 0x2000\n\ts_nop 0\n\tglobal_load_lds_dwordx4 %9, %3\n\t"
                    "s_mov_b32 m0, %0"
                    : "=&s"(keep)
                    : "s"(l), "s"(xb), "s"(wb), "v"(xo[0]), "v"(xo[1]), "v"(xo[2]), "v"(xo[3]), "v"(wo[0]), "v"(wo[1])
                    : "memory", "scc");
            };
            stage(0, 0);
            stage(1, 1);
            int buf = 0;
            for (int t = 0; t < 32; ++t) {
                if (t < 31) asm volatile("s_waitcnt vmcnt(6) lgkmcnt(0)" ::: "memory");
                else asm volatile("s_waitcnt vmcnt(0) lgkmcnt(0)" ::: "memory");
                __builtin_amdgcn_s_barrier();
                asm volatile("" ::: "memory");
                if (t < 30) stage(t + 2, buf == 0 ? 2 : buf - 1);
                LDS_AS char* sb = (LDS_AS char*)smem + buf * STAGE;
#pragma unroll
                for (int kh = 0; kh < 2; ++kh) {
                    v4i xf[4], wf[4];
#pragma unroll
                    for (int ni = 0; ni < 4; ++ni) {
                        v2i lo = __builtin_amdgcn_ds_read_tr8_b64_v2i32((LDS_AS v2i*)(sb + wa[ni] + kh * 8192));
                        v2i hi = __builtin_amdgcn_ds_read_tr8_b64_v2i32((LDS_AS v2i*)(sb + wa[ni] + kh * 8192 + 1024));
                        wf[ni] = v4i{lo[0], lo[1], hi[0], hi[1]};
                    }
#pragma unroll
                    for (int mi = 0; mi < 4; ++mi) xf[mi] = *(LDS_AS v4i*)(sb + xa[kh] + mi * 2048);
#pragma unroll
                    for (int mi = 0; mi < 4; ++mi)
#pragma unroll
                        for (int ni = 0; ni < 4; ++ni)
                            acc[mi][ni] = __builtin_amdgcn_mfma_i32_16x16x64_i8(wf[ni], xf[mi], acc[mi][ni], 0, 0, 0);
                }
                buf = buf == 2 ? 0 : buf + 1;
            }
        }
        STAMP(ws, 2 + ti * 3);
        {
            const bool loc = rt >= 6;
            const float sc = dead ? __builtin_nanf("") : scale;
            const auto rs = loc ? rsrc(P.out, OUT_BYTES) : rsrc(ws, WS_NEED);
            const unsigned pitch = loc ? NTOT * 4 : 512;
            const unsigned base = loc ? (unsigned)(rt - 6) * 256 * (NTOT * 4) + chunk_col(rank, cc) * 4 : OFF_OS + (cc * 6 + rt) * 131072;
            const unsigned lo = (wm * 64 + fr) * pitch + wn * 256 + fq * 16;
#pragma unroll
            for (int mi = 0; mi < 4; ++mi)
#pragma unroll
                for (int ni = 0; ni < 4; ++ni) {
                    v4f o;
#pragma unroll
                    for (int j = 0; j < 4; ++j) o[j] = silu(acc[mi][ni][j], sc);
                    st128<AUX_SYS>(__builtin_bit_cast(v4u, o), rs, lo + base + mi * 16 * pitch + ni * 64);
                }
            WAIT_VM0();
            __syncthreads();
            if (!loc && threadIdx.x < 8) {
                int d, slot;
                entry_inv(g, rt * 8 + threadIdx.x, d, slot);
                __hip_atomic_store((gu32*)(P.ws[dest_rank(rank, d) & 7] + OFF_RDY + ((rank * NCC + cc) * 8 + rt) * 4), call,
                                   __ATOMIC_RELAXED, __HIP_MEMORY_SCOPE_SYSTEM);
            }
        }
        STAMP(ws, 3 + ti * 3);
    }
}

}

extern "C" void kernel_launch(int rank, int nranks, ncclComm_t comm,
                              hipStream_t stream,
                              void* const* d_in, const int* in_sizes, int n_in,
                              void* d_out, int out_size,
                              const int* in_dtypes, int out_dtype,
                              void* const* d_out_all,
                              void* const* d_ws_all, size_t ws_bytes) {
    if (nranks != NR || (unsigned)rank >= (unsigned)NR || ws_bytes < WS_NEED || n_in != 4 ||
        in_sizes[0] != (int)X_BYTES || in_sizes[1] != (int)W_BYTES || out_size != 512 * NTOT) {
        fprintf(stderr, "unexpected problem shape\n");
        return;
    }
    Params p;
    p.x = (const char*)d_in[0]; p.w = (const char*)d_in[1];
    p.sx = (const float*)d_in[2]; p.sw = (const float*)d_in[3];
    p.out = (char*)d_out;
    for (int i = 0; i < NR; ++i) p.ws[i] = (char*)d_ws_all[i];
    p.rank = rank;
    hipLaunchKernelGGL(fused_gemm_scatter, dim3(NWG), dim3(512), 0, stream, p);
    HIP_CHECK(hipGetLastError());
}
```

```cpp
#include <hip/hip_runtime.h>
#include <rccl/rccl.h>
#include <cstdio>

#define HIP_CHECK(expr) do { hipError_t _e = (expr); if (_e != hipSuccess) { \
    fprintf(stderr, "HIP error %s at %s:%d\n", hipGetErrorString(_e), __FILE__, __LINE__); \
    return; } } while (0)

#define PULL_U 2
#define PULL_NL 36
#define PUSH_NW 2
#define PUSH_NP 2
#define PUSH_WIN 6
#define ACQ_SCOPE ""

namespace {

typedef int v2i __attribute__((ext_vector_type(2)));
typedef int v4i __attribute__((ext_vector_type(4)));
typedef unsigned v4u __attribute__((ext_vector_type(4)));
typedef float v4f __attribute__((ext_vector_type(4)));
typedef unsigned long long u64;
#define LDS_AS __attribute__((address_space(3)))
#define GLB_AS __attribute__((address_space(1)))
typedef GLB_AS unsigned gu32;

constexpr int NR = 8;
constexpr int NTOT = 16384;
constexpr int SAME_BLK = 7, CROSS_BLK = 9;
constexpr int SLOT_A = 28;
constexpr int SLOT_B2 = 42, SLOT_B3 = 49, SLOT_OWN = 57;
constexpr int NSLOT = SLOT_OWN + 16;
constexpr int NCC = 32;
constexpr int CC_A = 8;
constexpr int NDMA = 64;
constexpr int NWG = 256;
constexpr int NL = PULL_NL;
constexpr int MS_XA = 32, MS_XB1 = 33, MS_XB2 = 34, MS_XB3 = 35, MS_XO = 36;
constexpr int NPS = 8 * PUSH_NW * PUSH_NP / 2;
constexpr int B_WA = SLOT_A - NPS, B_XB = B_WA + 4 * CC_A, B_WB = B_XB + (SLOT_OWN - SLOT_A);
constexpr int B_XO = B_WB + 4 * (NCC - CC_A), B_OUT = B_XO + 16;
static_assert(NPS <= SLOT_A, "pushed slots");

constexpr unsigned OFF_CNT = 0;
constexpr unsigned OFF_TMO = 8192;
constexpr unsigned OFF_Q = 12288;
constexpr unsigned OFF_RDY = 16384;
constexpr unsigned OFF_EXF = 32768;
constexpr unsigned OFF_DBG = 65536;
constexpr unsigned OFF_MS = 262144;
constexpr unsigned XS_SRC = NSLOT * 16384;
constexpr unsigned XS_PAR = NR * XS_SRC;
constexpr unsigned WS_SRC = NCC * 65536;
constexpr unsigned EX_PAR = 160 * 65536;
constexpr unsigned OFF_XS = 1u << 20;
constexpr unsigned OFF_WS = OFF_XS + 2 * XS_PAR;
constexpr unsigned OFF_EX = OFF_WS + NR * WS_SRC;
constexpr unsigned OFF_OS = OFF_EX + 2 * EX_PAR;
constexpr unsigned WS_NEED = OFF_OS + NCC * 6 * 131072;
constexpr unsigned OUT_BYTES = 512u * NTOT * 4u;
constexpr unsigned X_BYTES = 4096u * 512u, W_BYTES = 512u * NTOT;

constexpr u64 DEADLINE = 50000000ull;
constexpr int STAGE = 49152;
constexpr int AUX_SYS = 17;
constexpr int AUX_DEV = 16;

struct Params {
    const char* x; const char* w; const float* sx; const float* sw;
    char* out; char* ws[NR];
    int rank;
};

__device__ __forceinline__ int rfl(int v) { return __builtin_amdgcn_readfirstlane(v); }
__device__ __forceinline__ u64 ballot(bool b) { return __builtin_amdgcn_ballot_w64(b); }
__device__ __forceinline__ auto rsrc(const void* p, unsigned bytes) {
    return __builtin_amdgcn_make_buffer_rsrc((void*)p, 0, bytes, 0x00020000);
}
template <int AUX, class R> __device__ __forceinline__ void st128(v4u v, R rs, unsigned off) {
    __builtin_amdgcn_raw_buffer_store_b128(v, rs, off, 0, AUX);
}
__device__ __forceinline__ int dest_rank(int c, int d) { return d < 3 ? (c & 4) | ((c + 1 + d) & 3) : ((c & 4) ^ 4) | (d - 3); }
__device__ __forceinline__ int slot_row(int c, int s) {
    if (s >= SLOT_OWN) return c * 512 + (s - SLOT_OWN) * 32;
    int j, d;
    if (s < SLOT_B3) { j = s / 7; d = s - j * 7; } else { j = 7 + ((s - SLOT_B3) >> 2); d = 3 + ((s - SLOT_B3) & 3); }
    return dest_rank(c, d) * 512 + (d < 3 ? 0 : SAME_BLK * 32) + j * 32;
}
__device__ __forceinline__ int entry_of(int g, int d, int j) {
    int i = d < 3 + g ? d : d - 1;
    return j < 7 ? j * 6 + i : 42 + (j - 7) * 3 + (i - 3);
}
__device__ __forceinline__ void entry_inv(int g, int e, int& d, int& slot) {
    int j, i;
    if (e < 42) { j = e / 6; i = e - j * 6; } else { j = 7 + (e - 42) / 3; i = 3 + (e - 42) % 3; }
    d = i < 3 + g ? i : i + 1;
    slot = j < 7 ? j * 7 + d : SLOT_B3 + (j - 7) * 4 + (d - 3);
}
__device__ __forceinline__ int chunk_col(int c, int cc) {
    int g = cc & 3, gid = (c & 4) ? g * 4 + (c & 3) : (c & 3) * 4 + g;
    return gid * 1024 + (cc >> 2) * 128;
}
__device__ __forceinline__ unsigned ld_sys(const char* p) {
    return __hip_atomic_load((gu32*)p, __ATOMIC_RELAXED, __HIP_MEMORY_SCOPE_SYSTEM);
}
__device__ __forceinline__ void st_sys(char* p, unsigned v) {
    __hip_atomic_store((gu32*)p, v, __ATOMIC_RELAXED, __HIP_MEMORY_SCOPE_SYSTEM);
}
__device__ __forceinline__ void set_tmo(char* ws, unsigned call) {
    __hip_atomic_store((gu32*)(ws + OFF_TMO), call, __ATOMIC_RELAXED, __HIP_MEMORY_SCOPE_AGENT);
}
__device__ __forceinline__ bool expired(char* ws, unsigned call, u64 t0, unsigned& round) {
    bool out = __builtin_amdgcn_s_memrealtime() - t0 > DEADLINE;
    if ((round++ & 7u) == 0u) out = out || rfl((int)ld_sys(ws + OFF_TMO)) == (int)call;
    if (out) set_tmo(ws, call);
    return out;
}
template <int SLEEP> __device__ __forceinline__ bool wait_words(char* ws, unsigned call, int lane, u64 mask, const char* p, unsigned target) {
    const u64 t0 = __builtin_amdgcn_s_memrealtime();
    if (!((mask >> lane) & 1)) p = ws + OFF_TMO;
    for (;;) {
        const unsigned v = ld_sys(p);
        if (ballot(v == call) >> 63) return false;
        if ((ballot((int)(v - target) < 0) & mask) == 0) return true;
        if (__builtin_amdgcn_s_memrealtime() - t0 > DEADLINE) { set_tmo(ws, call); return false; }
        __builtin_amdgcn_s_sleep(SLEEP);
    }
}
#define WAIT_VM0() asm volatile("s_waitcnt vmcnt(0)" ::: "memory")
#define STR2(x) #x
#define STR(x) STR2(x)

#define STAMPV(ws, idx, v) do {} while (0)
#define STAMP(ws, idx) STAMPV(ws, idx, __builtin_amdgcn_s_memrealtime())

__device__ __forceinline__ float silu(int acc, float s) {
    float y = (float)acc * s;
    float c = fminf(fmaxf(y, -60.0f), 60.0f);
    float e = __builtin_amdgcn_exp2f(c * -1.4426950408889634f);
    return y * __builtin_amdgcn_rcpf(1.0f + e);
}
__device__ __forceinline__ unsigned ticket(char* p, int lane) {
    unsigned v = 0, z = 0;
    asm volatile("" : "+v"(z));
    if (lane == 0) v = __hip_atomic_fetch_add((gu32*)(p + z), 1u, __ATOMIC_RELAXED, __HIP_MEMORY_SCOPE_AGENT);
    return v;
}

__global__ void __launch_bounds__(512, 2) fused_gemm_scatter(Params P) {
    __shared__ __attribute__((aligned(16))) char smem[3 * STAGE + 16];
    const int rank = P.rank;
    const int lane = threadIdx.x & 63, wid = rfl(threadIdx.x >> 6);
    char* const ws = P.ws[rank];
    const int xcd = blockIdx.x & 7;
    const unsigned lin = lane * 16;
    STAMP(ws, 0);
    if (blockIdx.x < 8) return;
    const unsigned callv = ticket(ws + OFF_CNT + (blockIdx.x * 8 + wid) * 4, lane);

    if (blockIdx.x < NDMA) {
        const int ps = blockIdx.x >> 3, c = (rank + ps) & 7;
        char* const pws = P.ws[c];
        const auto prs = rsrc(pws, WS_NEED);
        if (wid < PUSH_NW) {
            const auto xrs = rsrc(P.x, X_BYTES);
            const int h0 = (xcd * PUSH_NW + wid) * PUSH_NP;
            v4u v[PUSH_NP * 8];
#pragma unroll
            for (int i = 0; i < PUSH_NP; ++i) {
                const unsigned so = (slot_row(c, (h0 + i) >> 1) + ((h0 + i) & 1) * 16) * 512;
#pragma unroll
                for (int u = 0; u < 8; ++u) v[i * 8 + u] = __builtin_amdgcn_raw_buffer_load_b128(xrs, lin, so + u * 1024, 0);
            }
            const unsigned call = (unsigned)rfl((int)callv) + 1u;
            const unsigned dof = OFF_XS + (call & 1u) * XS_PAR + rank * XS_SRC + h0 * 8192 + lin;
            WAIT_VM0();
#pragma unroll
            for (int u = 0; u < PUSH_NP * 8; ++u) {
                if (u >= PUSH_WIN) asm volatile("s_waitcnt vmcnt(%0)" :: "n"(PUSH_WIN - 1) : "memory");
                st128<AUX_SYS>(v[u], prs, dof + u * 1024);
            }
            WAIT_VM0();
            if (lane == 0) __hip_atomic_fetch_add((gu32*)(pws + OFF_MS + rank * 128 + MS_XA * 1024), 8u * PUSH_NP, __ATOMIC_RELAXED, __HIP_MEMORY_SCOPE_SYSTEM);
        }
        STAMP(ws, 1);
        if ((7 - wid) * 8 + xcd < NL) {
            const unsigned call = (unsigned)rfl((int)callv) + 1u;
            const unsigned par = call & 1u;
            const auto frs = rsrc(ws + OFF_RDY + c * 1024, 1024);
            const char* const exf = ws + OFF_EXF + c * 256 + lane * 4;
            const bool same = ((c ^ rank) & 4) == 0;
            const int nblk = same ? SAME_BLK : CROSS_BLK;
            const int d = same ? ((rank - c) & 3) - 1 : 3 + (rank & 3);
            const int row0 = same ? 0 : SAME_BLK * 32;
            constexpr int PARTS = 16 / PULL_U;
            const int nca = same ? CC_A : CC_A / 4 * 3, ncb = same ? NCC - CC_A : (NCC - CC_A) / 4 * 3;
            const int nb1 = nca * 4, nb2 = nca * nblk;
            const unsigned nunits = (B_OUT + (nca + ncb) * nblk) * PARTS;
            char* const q = ws + OFF_Q + ps * 128;
            char* const msb = ws + OFF_MS + c * 128;
            u64 exm = 0;
            u64 r0 = 0, r1 = 0, r2 = 0, r3 = 0;
            const int last_bit = (!same && (rank & 3) == 3 ? NCC - 2 : NCC - 1) * 2;
            const unsigned ol = (lane >> 5) * (NTOT * 4) + (lane & 31) * 16;
            u64 stall = 0, npoll = 0;
            bool dead = false;
            auto decode = [&](unsigned un, unsigned& so, unsigned& dof, int& ms, int& need) {
                const int part = un % PARTS;
                int t = un / PARTS;
                if (t < B_OUT) {
                    if ((t >= B_WA && t < B_XB) || (t >= B_WB && t < B_XO)) {
                        int i = t < B_XB ? t - B_WA : t - B_WB + 4 * CC_A;
                        need = ms = i >> 2;
                        so = OFF_EX + par * EX_PAR + chunk_col(rank, ms) * 512 + (i & 3) * 16384;
                        dof = OFF_WS + c * WS_SRC + i * 16384;
                    } else {
                        int sl = t < B_WA ? t + NPS : t < B_WB ? t - B_XB + SLOT_A : t - B_XO + SLOT_OWN;
                        ms = sl < SLOT_A ? MS_XA : sl < SLOT_B2 ? MS_XB1 : sl < SLOT_B3 ? MS_XB2 : sl < SLOT_OWN ? MS_XB3 : MS_XO;
                        const int row = slot_row(rank, sl);
                        need = 32 + (row >> 7);
                        so = OFF_EX + par * EX_PAR + W_BYTES + row * 512;
                        dof = OFF_XS + par * XS_PAR + c * XS_SRC + sl * 16384;
                    }
                    so += part * (PULL_U * 1024);
                    dof += part * (PULL_U * 1024);
                    return;
                }
                t -= B_OUT;
                int ci, blk;
                if (t < nb1) { ci = t >> 2; blk = t & 3; }
                else if (t < nb2) { blk = t / nca; ci = t - blk * nca; }
                else { ci = t / nblk; blk = t - ci * nblk; }
                int cc = ci;
                if (!same) { int sub = ci / 3, gi = ci - sub * 3; cc = sub * 4 + (gi < (rank & 3) ? gi : gi + 1); }
                const int e = entry_of(cc & 3, d, blk);
                ms = -1;
                need = cc * 8 + (e >> 3);
                so = OFF_OS + (cc * 48 + e) * 16384 + part * (PULL_U * 1024);
                dof = (unsigned)(row0 + blk * 32 + part * (2 * PULL_U)) * (NTOT * 4) + chunk_col(c, cc) * 4;
            };
            auto put = [&](v4u (&v)[PULL_U], unsigned dof, int ms) {
                const bool out = ms < 0;
                const auto rs = rsrc(out ? P.out : ws, out ? OUT_BYTES : WS_NEED);
                const unsigned b = dof + (out ? ol : lin), st = out ? 2 * (NTOT * 4) : 1024;
#pragma unroll
                for (int u = 0; u < PULL_U; ++u) st128<AUX_DEV>(v[u], rs, b + u * st);
            };
            auto signal = [&](int ms) {
                if (ms >= 0 && lane == 0)
                    __hip_atomic_fetch_add((gu32*)(msb + ms * 1024), (unsigned)PULL_U, __ATOMIC_RELAXED, __HIP_MEMORY_SCOPE_AGENT);
            };
            auto masks = [&](v4u f) {
                r0 = ballot((int)(f[0] - call) >= 0); r1 = ballot((int)(f[1] - call) >= 0);
                r2 = ballot((int)(f[2] - call) >= 0); r3 = ballot((int)(f[3] - call) >= 0);
            };
            auto have = [&](int ms, int k) {
                if (ms >= 0) return rfl((int)((exm >> k) & 1)) != 0;
                const int sh = k >> 2;
                const unsigned m = (unsigned)((r0 >> sh) & 1) | (unsigned)((r1 >> sh) & 1) << 1 | (unsigned)((r2 >> sh) & 1) << 2 | (unsigned)((r3 >> sh) & 1) << 3;
                return rfl((int)((m >> (k & 3)) & 1)) != 0;
            };
            auto await = [&](int ms, int k) {
                const u64 t0 = __builtin_amdgcn_s_memrealtime();
                unsigned round = 1;
                for (;;) {
                    asm volatile("" ::: "memory");
                    if (ms >= 0) {
                        exm = ballot((int)(ld_sys(exf) - call) >= 0);
                    } else {
                        masks(__builtin_amdgcn_raw_buffer_load_b128(frs, lin, 0, AUX_SYS));
                    }
                    if (have(ms, k)) break;
                    if (expired(ws, call, t0, round)) { dead = true; break; }
                    __builtin_amdgcn_s_sleep(1);
                }
                if (ms < 0) { stall += __builtin_amdgcn_s_memrealtime() - t0; ++npoll; }
            };
            constexpr int NONE = -1000;
            unsigned un1 = (7 - wid) * 8 + xcd;
            unsigned so, dof, pdof = 0;
            int ms, need, pms = NONE, ppms = NONE;
            v4u pv[PULL_U];
            decode(un1, so, dof, ms, need);
            un1 += NL;
            auto flush = [&]() {
                if (pms != NONE) put(pv, pdof, pms);
                signal(ppms);
                if (pms >= 0) { WAIT_VM0(); signal(pms); }
                pms = ppms = NONE;
            };
            for (;;) {
                if (!dead && !have(ms, need)) {
                    flush();
                    await(ms, need);
                }
                asm volatile("" ::: "memory");
                v4u v[PULL_U], f;
                unsigned fe;
#pragma unroll
                for (int u = 0; u < PULL_U; ++u) v[u] = __builtin_amdgcn_raw_buffer_load_b128(prs, lin, so + u * 1024, AUX_SYS);
                if (pms != NONE) put(pv, pdof, pms);
                signal(ppms);
                const unsigned nxt = ticket(q, lane);
                ppms = pms; pms = ms; pdof = dof;
                const bool ref_e = exm != ~0ull;
                const bool ref_r = un1 + 2 * NL >= B_OUT * PARTS && !rfl((int)(r1 >> last_bit & 1));
                if (ref_e) fe = ld_sys(exf);
                if (ref_r) f = __builtin_amdgcn_raw_buffer_load_b128(frs, lin, 0, AUX_SYS);
                const bool more = un1 < nunits;
                if (more) decode(un1, so, dof, ms, need);
                WAIT_VM0();
#pragma unroll
                for (int u = 0; u < PULL_U; ++u) pv[u] = dead ? v4u{0x7fc00000u, 0x7fc00000u, 0x7fc00000u, 0x7fc00000u} : v[u];
                if (!more) break;
                if (ref_e) exm = ballot((int)(fe - call) >= 0);
                if (ref_r) masks(f);
                un1 = (unsigned)rfl((int)nxt) - (call - 1u) * nunits + 2 * NL;
            }
            flush();
            STAMPV(ws, 4, stall);
            STAMPV(ws, 5, npoll);
        }
        STAMP(ws, 2);
        return;
    }

    const int job = blockIdx.x - NDMA;
    unsigned call;
    if (job < 160) {
        const bool isw = job < 128;
        const auto rs = rsrc(isw ? P.w : P.x, isw ? W_BYTES : X_BYTES);
        const unsigned so = isw ? (wid * 64 + (lane >> 3)) * NTOT + job * 128 + (lane & 7) * 16 : (job - 128) * 65536 + wid * 8192 + lin;
        const unsigned st = isw ? 8 * NTOT : 1024;
        v4u v[8];
#pragma unroll
        for (int u = 0; u < 8; ++u) v[u] = __builtin_amdgcn_raw_buffer_load_b128(rs, so + u * st, 0, 0);
        call = (unsigned)rfl((int)callv) + 1u;
        const auto wrs = rsrc(ws, WS_NEED);
        const unsigned dof = OFF_EX + (call & 1u) * EX_PAR + job * 65536 + wid * 8192 + lin;
#pragma unroll
        for (int u = 0; u < 8; ++u) st128<AUX_SYS>(v[u], wrs, dof + u * 1024);
        WAIT_VM0();
        __syncthreads();
        if (threadIdx.x < (isw ? 2 : 8)) {
            const int gid = job >> 3, t = threadIdx.x;
            const int to = isw ? (t ? 4 + (gid & 3) : gid >> 2) : t;
            const int bit = isw ? (job & 7) * 4 + (t ? gid >> 2 : gid & 3) : job - 96;
            st_sys(P.ws[to & 7] + OFF_EXF + (rank * 64 + bit) * 4, call);
        }
    } else {
        call = (unsigned)rfl((int)callv) + 1u;
    }
    STAMP(ws, 7);
    const unsigned par = call & 1u;
    const int li = job >> 3;
    const int ntiles = (xcd < 4 && li < 16) ? 2 : 1;
    const float scale = P.sx[0] * P.sw[0];
    const int wm = wid >> 1, wn = wid & 1, fr = lane & 15, fq = lane >> 4;
    const unsigned lds0 = (unsigned)(size_t)(LDS_AS char*)smem;
    volatile LDS_AS int* const dead_w = (volatile LDS_AS int*)((LDS_AS char*)smem + 3 * STAGE);
    unsigned xa[2], wa[4];
    for (int kh = 0; kh < 2; ++kh) xa[kh] = (wm * 64 + fr) * 128 + (((kh * 4 + fq) ^ (fr >> 1)) << 4);
    for (int ni = 0; ni < 4; ++ni)
        wa[ni] = 32768 + (fq * 16 + (fr >> 1)) * 128 + (((wn * 4 + ni) ^ (((fr >> 2) & 3) | ((fq & 1) << 2))) << 4) + 8 * (fr & 1);

    for (int ti = 0; ti < ntiles; ++ti) {
        int cc, rt;
        if (ti != 0) { int idx = xcd * 16 + li; cc = idx >> 1; rt = 6 + (idx & 1); }
        else if (xcd < 2) { cc = li / 3; rt = xcd * 3 + li % 3; }
        else { cc = xcd * 4 + li / 6; rt = li % 6; }
        cc = rfl(cc); rt = rfl(rt);
        const int g = cc & 3;
        bool dead = false;

        v4i acc[4][4];
#pragma unroll
        for (int i = 0; i < 4; ++i)
#pragma unroll
            for (int j = 0; j < 4; ++j) acc[i][j] = v4i{0, 0, 0, 0};

#pragma unroll 1
        for (int seg = 0; seg < 2; ++seg) {
            if (seg == 1) {
                if (wid == 0) {
                    const int m1 = rt < 3 ? MS_XA : rt < 5 ? MS_XB1 : rt < 6 ? MS_XB3 : MS_XO;
                    const unsigned n1 = rt < 3 ? SLOT_A : rt < 5 ? SLOT_B2 - SLOT_A : rt < 6 ? SLOT_OWN - SLOT_B3 : 16;
                    const u64 peers = 0xffull & ~(1ull << rank);
                    const char* mb = ws + OFF_MS + (lane & 7) * 128;
                    bool ok = wait_words<8>(ws, call, lane, peers, mb + cc * 1024, call * 64u);
                    if (ok) ok = wait_words<8>(ws, call, lane, (rt == 4 || rt == 5) ? peers | peers << 8 : peers,
                                               mb + (lane & 8 ? MS_XB2 : m1) * 1024, call * 16u * (lane & 8 ? SLOT_B3 - SLOT_B2 : n1));
                    __builtin_amdgcn_fence(__ATOMIC_ACQUIRE, ACQ_SCOPE);
                    *dead_w = ok ? 0 : 1;
                    WAIT_VM0();
                }
                __syncthreads();
                dead = rfl(*dead_w) != 0;
                STAMP(ws, 1 + ti * 3);
                if (dead) break;
            }
            unsigned xo[4], wo[2];
#pragma unroll
            for (int i = 0; i < 4; ++i) {
                int row = i * 64 + wid * 8 + (lane >> 3), e = rt * 8 + i * 2 + (wid >> 2), d, slot;
                if (rt >= 6) slot = SLOT_OWN + e - 48; else entry_inv(g, e, d, slot);
                xo[i] = (seg ? slot * 32 : slot_row(rank, slot)) * 512 + (row & 31) * 512 + (((lane & 7) ^ ((row >> 1) & 7)) << 4);
            }
#pragma unroll
            for (int i = 0; i < 2; ++i) {
                int k = i * 64 + wid * 8 + (lane >> 3);
                wo[i] = k * (seg ? 128 : NTOT) + (((lane & 7) ^ (((k >> 1) & 3) | (((k >> 4) & 1) << 2))) << 4);
            }
            const char* xs = seg ? ws + OFF_XS + par * XS_PAR : P.x;
            const char* wsb = seg ? ws + OFF_WS + cc * 65536 : P.w + chunk_col(rank, cc);
            const unsigned wstep = seg ? 16384 : 128 * NTOT;
            const int nt = seg ? 28 : 4;
            auto stage = [&](int t, int buf) {
                int p = t >> 2;
                p = seg ? p + (p >= rank) : 0;
                const char* xb = xs + p * XS_SRC + (t & 3) * 128;
                const char* wb = wsb + p * WS_SRC + (t & 3) * wstep;
                const unsigned l = lds0 + buf * STAGE + wid * 1024;
                unsigned keep;
                asm volatile(
                    "s_mov_b32 %0, m0\n\t"
                    "s_mov_b32 m0, %1\n\ts_nop 0\n\tglobal_load_lds_dwordx4 %4, %2\n\t"
                    "s_add_u32 m0, m0, 0x2000\n\ts_nop 0\n\tglobal_load_lds_dwordx4 %5, %2\n\t"
                    "s_add_u32 m0, m0, 0x2000\n\ts_nop 0\n\tglobal_load_lds_dwordx4 %6, %2\n\t"
                    "s_add_u32 m0, m0, 0x2000\n\ts_nop 0\n\tglobal_load_lds_dwordx4 %7, %2\n\t"
                    "s_add_u32 m0, m0, 0x2000\n\ts_nop 0\n\tglobal_load_lds_dwordx4 %8, %3\n\t"
                    "s_add_u32 m0, m0, 0x2000\n\ts_nop 0\n\tglobal_load_lds_dwordx4 %9, %3\n\t"
                    "s_mov_b32 m0, %0"
                    : "=&s"(keep)
                    : "s"(l), "s"(xb), "s"(wb), "v"(xo[0]), "v"(xo[1]), "v"(xo[2]), "v"(xo[3]), "v"(wo[0]), "v"(wo[1])
                    : "memory", "scc");
            };
            v4i xf[2][4], wf[2][4];
            auto frags = [&](int buf, int kh) {
                LDS_AS char* sb = (LDS_AS char*)smem + buf * STAGE;
#pragma unroll
                for (int ni = 0; ni < 4; ++ni) {
                    v2i lo = __builtin_amdgcn_ds_read_tr8_b64_v2i32((LDS_AS v2i*)(sb + wa[ni] + kh * 8192));
                    v2i hi = __builtin_amdgcn_ds_read_tr8_b64_v2i32((LDS_AS v2i*)(sb + wa[ni] + kh * 8192 + 1024));
                    wf[kh][ni] = v4i{lo[0], lo[1], hi[0], hi[1]};
                }
#pragma unroll
                for (int mi = 0; mi < 4; ++mi) xf[kh][mi] = *(LDS_AS v4i*)(sb + xa[kh] + mi * 2048);
            };
            auto mma = [&](int kh) {
#pragma unroll
                for (int mi = 0; mi < 4; ++mi)
#pragma unroll
                    for (int ni = 0; ni < 4; ++ni)
                        acc[mi][ni] = __builtin_amdgcn_mfma_i32_16x16x64_i8(wf[kh][ni], xf[kh][mi], acc[mi][ni], 0, 0, 0);
            };
            WAIT_VM0();
            stage(0, 0);
            stage(1, 1);
            stage(2, 2);
            asm volatile("s_waitcnt vmcnt(12)" ::: "memory");
            __builtin_amdgcn_s_barrier();
            asm volatile("" ::: "memory");
            frags(0, 0);
            int buf = 0;
            for (int t = 0; t < nt; ++t) {
                frags(buf, 1);
                __builtin_amdgcn_sched_barrier(0);
                mma(0);
                __builtin_amdgcn_sched_barrier(0);
                if (t < nt - 2) asm volatile("s_waitcnt vmcnt(6) lgkmcnt(0)" ::: "memory");
                else asm volatile("s_waitcnt vmcnt(0) lgkmcnt(0)" ::: "memory");
                __builtin_amdgcn_s_barrier();
                asm volatile("" ::: "memory");
                if (t < nt - 3) stage(t + 3, buf);
                buf = buf == 2 ? 0 : buf + 1;
                if (t < nt - 1) frags(buf, 0);
                __builtin_amdgcn_sched_barrier(0);
                mma(1);
                __builtin_amdgcn_sched_barrier(0);
            }
        }
        STAMP(ws, 2 + ti * 3);
        {
            const bool loc = rt >= 6;
            const float sc = dead ? __builtin_nanf("") : scale;
            const auto rs = loc ? rsrc(P.out, OUT_BYTES) : rsrc(ws, WS_NEED);
            const unsigned pitch = loc ? NTOT * 4 : 512;
            const unsigned base = loc ? (unsigned)(rt - 6) * 256 * (NTOT * 4) + chunk_col(rank, cc) * 4 : OFF_OS + (cc * 6 + rt) * 131072;
            const unsigned lo = (wm * 64 + fr) * pitch + wn * 256 + fq * 16;
#pragma unroll
            for (int mi = 0; mi < 4; ++mi)
#pragma unroll
                for (int ni = 0; ni < 4; ++ni) {
                    v4f o;
#pragma unroll
                    for (int j = 0; j < 4; ++j) o[j] = silu(acc[mi][ni][j], sc);
                    st128<AUX_SYS>(__builtin_bit_cast(v4u, o), rs, lo + base + mi * 16 * pitch + ni * 64);
                }
            WAIT_VM0();
            __syncthreads();
            if (!loc && threadIdx.x < 8) {
                int d, slot;
                entry_inv(g, rt * 8 + threadIdx.x, d, slot);
                st_sys(P.ws[dest_rank(rank, d) & 7] + OFF_RDY + ((rank * NCC + cc) * 8 + rt) * 4, call);
            }
        }
        STAMP(ws, 3 + ti * 3);
    }
}

}

extern "C" void kernel_launch(int rank, int nranks, ncclComm_t comm,
                              hipStream_t stream,
                              void* const* d_in, const int* in_sizes, int n_in,
                              void* d_out, int out_size,
                              const int* in_dtypes, int out_dtype,
                              void* const* d_out_all,
                              void* const* d_ws_all, size_t ws_bytes) {
    if (nranks != NR || (unsigned)rank >= (unsigned)NR || ws_bytes < WS_NEED || n_in != 4 ||
        in_sizes[0] != (int)X_BYTES || in_sizes[1] != (int)W_BYTES || out_size != 512 * NTOT) {
        fprintf(stderr, "unexpected problem shape\n");
        return;
    }
    Params p;
    p.x = (const char*)d_in[0]; p.w = (const char*)d_in[1];
    p.sx = (const float*)d_in[2]; p.sw = (const float*)d_in[3];
    p.out = (char*)d_out;
    for (int i = 0; i < NR; ++i) p.ws[i] = (char*)d_ws_all[i];
    p.rank = rank;
    hipLaunchKernelGGL(fused_gemm_scatter, dim3(NWG), dim3(512), 0, stream, p);
    HIP_CHECK(hipGetLastError());
}
```

```cpp
#include <hip/hip_runtime.h>
#include <rccl/rccl.h>
#include <cstdio>

#define HIP_CHECK(expr) do { hipError_t _e = (expr); if (_e != hipSuccess) { \
    fprintf(stderr, "HIP error %s at %s:%d\n", hipGetErrorString(_e), __FILE__, __LINE__); \
    return; } } while (0)

#define PULL_U 2
#define PULL_NL 36
#define PUSH_NW 1
#define PUSH_NP 4
#define PUSH_WIN 8
#define GEMM_SLEEP 32
#define TOUCH 0
#define PERM 0x75416320u
#define EXP_FIRST 0
#define EXP_DELAY 100
#define EPI_AUX 17
#define ACQ_SCOPE ""

namespace {

typedef int v2i __attribute__((ext_vector_type(2)));
typedef int v4i __attribute__((ext_vector_type(4)));
typedef unsigned v4u __attribute__((ext_vector_type(4)));
typedef float v4f __attribute__((ext_vector_type(4)));
typedef unsigned long long u64;
#define LDS_AS __attribute__((address_space(3)))
#define GLB_AS __attribute__((address_space(1)))
typedef GLB_AS unsigned gu32;

constexpr int NR = 8;
constexpr int NTOT = 16384;
constexpr int SAME_BLK = 7, CROSS_BLK = 9;
constexpr int SLOT_A = 28;
constexpr int SLOT_B2 = 42, SLOT_B3 = 49, SLOT_OWN = 57;
constexpr int NSLOT = SLOT_OWN + 16;
constexpr int NCC = 32;
constexpr int CC_A = 8;
constexpr int NDMA = 64;
constexpr int NWG = 256;
constexpr int NL = PULL_NL;
constexpr int MS_XA = 32, MS_XB1 = 33, MS_XB2 = 34, MS_XB3 = 35, MS_XO = 36;
constexpr int NPS = 8 * PUSH_NW * PUSH_NP / 2;
constexpr int B_WA = SLOT_A - NPS, B_XB = B_WA + 4 * CC_A, B_WB = B_XB + (SLOT_OWN - SLOT_A);
constexpr int B_XO = B_WB + 4 * (NCC - CC_A), B_OUT = B_XO + 16;
static_assert(NPS <= SLOT_A, "pushed slots");

constexpr unsigned OFF_CNT = 0;
constexpr unsigned OFF_TMO = 8192;
constexpr unsigned OFF_Q = 12288;
constexpr unsigned OFF_RDY = 16384;
constexpr unsigned OFF_EXF = 32768;
constexpr unsigned OFF_DBG = 65536;
constexpr unsigned OFF_MS = 262144;
constexpr unsigned XS_SRC = NSLOT * 16384;
constexpr unsigned XS_PAR = NR * XS_SRC;
constexpr unsigned WS_SRC = NCC * 65536;
constexpr unsigned EX_PAR = 160 * 65536;
constexpr unsigned OFF_XS = 1u << 20;
constexpr unsigned OFF_WS = OFF_XS + 2 * XS_PAR;
constexpr unsigned OFF_EX = OFF_WS + NR * WS_SRC;
constexpr unsigned OFF_OS = OFF_EX + 2 * EX_PAR;
constexpr unsigned WS_NEED = OFF_OS + NCC * 6 * 131072;
constexpr unsigned OUT_BYTES = 512u * NTOT * 4u;
constexpr unsigned X_BYTES = 4096u * 512u, W_BYTES = 512u * NTOT;

constexpr u64 DEADLINE = 50000000ull;
constexpr int STAGE = 49152;
constexpr int AUX_SYS = 17;
constexpr int AUX_DEV = 16;

struct Params {
    const char* x; const char* w; const float* sx; const float* sw;
    char* out; char* ws[NR];
    int rank;
};

__device__ __forceinline__ int rfl(int v) { return __builtin_amdgcn_readfirstlane(v); }
__device__ __forceinline__ u64 ballot(bool b) { return __builtin_amdgcn_ballot_w64(b); }
__device__ __forceinline__ auto rsrc(const void* p, unsigned bytes) {
    return __builtin_amdgcn_make_buffer_rsrc((void*)p, 0, bytes, 0x00020000);
}
template <int AUX, class R> __device__ __forceinline__ void st128(v4u v, R rs, unsigned off) {
    __builtin_amdgcn_raw_buffer_store_b128(v, rs, off, 0, AUX);
}
__host__ __device__ __forceinline__ int phys(int l) { return (PERM >> (4 * l)) & 7; }
__device__ __forceinline__ int dest_rank(int c, int d) { return d < 3 ? (c & 4) | ((c + 1 + d) & 3) : ((c & 4) ^ 4) | (d - 3); }
__device__ __forceinline__ int slot_row(int c, int s) {
    if (s >= SLOT_OWN) return phys(c) * 512 + (s - SLOT_OWN) * 32;
    int j, d;
    if (s < SLOT_B3) { j = s / 7; d = s - j * 7; } else { j = 7 + ((s - SLOT_B3) >> 2); d = 3 + ((s - SLOT_B3) & 3); }
    return phys(dest_rank(c, d)) * 512 + (d < 3 ? 0 : SAME_BLK * 32) + j * 32;
}
__device__ __forceinline__ int entry_of(int g, int d, int j) {
    int i = d < 3 + g ? d : d - 1;
    return j < 7 ? j * 6 + i : 42 + (j - 7) * 3 + (i - 3);
}
__device__ __forceinline__ void entry_inv(int g, int e, int& d, int& slot) {
    int j, i;
    if (e < 42) { j = e / 6; i = e - j * 6; } else { j = 7 + (e - 42) / 3; i = 3 + (e - 42) % 3; }
    d = i < 3 + g ? i : i + 1;
    slot = j < 7 ? j * 7 + d : SLOT_B3 + (j - 7) * 4 + (d - 3);
}
__device__ __forceinline__ int chunk_col(int c, int cc) {
    int g = cc & 3, gid = (c & 4) ? g * 4 + (c & 3) : (c & 3) * 4 + g;
    return gid * 1024 + (cc >> 2) * 128;
}
__device__ __forceinline__ unsigned ld_sys(const char* p) {
    return __hip_atomic_load((gu32*)p, __ATOMIC_RELAXED, __HIP_MEMORY_SCOPE_SYSTEM);
}
__device__ __forceinline__ void st_sys(char* p, unsigned v) {
    __hip_atomic_store((gu32*)p, v, __ATOMIC_RELAXED, __HIP_MEMORY_SCOPE_SYSTEM);
}
__device__ __forceinline__ void set_tmo(char* ws, unsigned call) {
    __hip_atomic_store((gu32*)(ws + OFF_TMO), call, __ATOMIC_RELAXED, __HIP_MEMORY_SCOPE_AGENT);
}
__device__ __forceinline__ bool expired(char* ws, unsigned call, u64 t0, unsigned& round) {
    bool out = __builtin_amdgcn_s_memrealtime() - t0 > DEADLINE;
    if ((round++ & 7u) == 0u) out = out || rfl((int)ld_sys(ws + OFF_TMO)) == (int)call;
    if (out) set_tmo(ws, call);
    return out;
}
template <int SLEEP> __device__ __forceinline__ bool wait_words(char* ws, unsigned call, int lane, u64 mask, const char* p, unsigned target) {
    const u64 t0 = __builtin_amdgcn_s_memrealtime();
    if (!((mask >> lane) & 1)) p = ws + OFF_TMO;
    for (;;) {
        const unsigned v = ld_sys(p);
        if (ballot(v == call) >> 63) return false;
        if ((ballot((int)(v - target) < 0) & mask) == 0) return true;
        if (__builtin_amdgcn_s_memrealtime() - t0 > DEADLINE) { set_tmo(ws, call); return false; }
        __builtin_amdgcn_s_sleep(SLEEP);
    }
}
#define WAIT_VM0() asm volatile("s_waitcnt vmcnt(0)" ::: "memory")
#define STR2(x) #x
#define STR(x) STR2(x)

#define STAMPV(ws, idx, v) do {} while (0)
#define STAMP(ws, idx) STAMPV(ws, idx, __builtin_amdgcn_s_memrealtime())

__device__ __forceinline__ float silu(int acc, float s) {
    float y = (float)acc * s;
    float c = fminf(fmaxf(y, -60.0f), 60.0f);
    float e = __builtin_amdgcn_exp2f(c * -1.4426950408889634f);
    return y * __builtin_amdgcn_rcpf(1.0f + e);
}
__device__ __forceinline__ unsigned ticket(char* p, int lane) {
    unsigned v = 0, z = 0;
    asm volatile("" : "+v"(z));
    if (lane == 0) v = __hip_atomic_fetch_add((gu32*)(p + z), 1u, __ATOMIC_RELAXED, __HIP_MEMORY_SCOPE_AGENT);
    return v;
}

__global__ void __launch_bounds__(512, 2) fused_gemm_scatter(Params P) {
    __shared__ __attribute__((aligned(16))) char smem[3 * STAGE + 16];
    const int rank = P.rank;
    const int lane = threadIdx.x & 63, wid = rfl(threadIdx.x >> 6);
    char* const ws = P.ws[rank];
    const int xcd = blockIdx.x & 7;
    const unsigned lin = lane * 16;
    STAMP(ws, 0);
    if (blockIdx.x < 8) {
        if (TOUCH && wid < 3) {
            for (int i = 1; i < TOUCH; ++i) __builtin_amdgcn_s_sleep(127);
            const char* p = wid == 2 ? P.out + (lane & 31) * (1u << 20) : ws + min((wid * 64 + lane) * (1u << 20), WS_NEED - 4);
            unsigned v = ld_sys(p);
            asm volatile("" :: "v"(v));
        }
        return;
    }
    const unsigned callv = ticket(ws + OFF_CNT + (blockIdx.x * 8 + wid) * 4, lane);

    if (blockIdx.x < NDMA) {
        const int ps = blockIdx.x >> 3, c = (rank + ps) & 7;
        char* const pws = P.ws[c];
        const auto prs = rsrc(pws, WS_NEED);
        if (wid < PUSH_NW) {
            const auto xrs = rsrc(P.x, X_BYTES);
            const int h0 = (xcd * PUSH_NW + wid) * PUSH_NP;
            v4u v[PUSH_NP * 8];
#pragma unroll
            for (int i = 0; i < PUSH_NP; ++i) {
                const unsigned so = (slot_row(c, (h0 + i) >> 1) + ((h0 + i) & 1) * 16) * 512;
#pragma unroll
                for (int u = 0; u < 8; ++u) v[i * 8 + u] = __builtin_amdgcn_raw_buffer_load_b128(xrs, lin, so + u * 1024, 0);
            }
            const unsigned call = (unsigned)rfl((int)callv) + 1u;
            const unsigned dof = OFF_XS + (call & 1u) * XS_PAR + rank * XS_SRC + h0 * 8192 + lin;
            WAIT_VM0();
#pragma unroll
            for (int u = 0; u < PUSH_NP * 8; ++u) {
                if (u >= PUSH_WIN) asm volatile("s_waitcnt vmcnt(%0)" :: "n"(PUSH_WIN - 1) : "memory");
                st128<AUX_SYS>(v[u], prs, dof + u * 1024);
            }
            WAIT_VM0();
            STAMPV(ws, 3, ta); STAMPV(ws, 4, tb); STAMPV(ws, 5, tc); STAMPV(ws, 6, td);
            if (lane == 0) __hip_atomic_fetch_add((gu32*)(pws + OFF_MS + rank * 128 + MS_XA * 1024), 8u * PUSH_NP, __ATOMIC_RELAXED, __HIP_MEMORY_SCOPE_SYSTEM);
        }
        if (TOUCH && wid == PUSH_NW) {
            for (int i = 1; i < TOUCH; ++i) __builtin_amdgcn_s_sleep(127);
            unsigned v = ld_sys(pws + (lane == 63 ? 0u : min(OFF_EX + lane * (1u << 20), WS_NEED - 4)));
            asm volatile("" :: "v"(v));
        }
        STAMP(ws, 1);
        if ((7 - wid) * 8 + xcd < NL) {
            const unsigned call = (unsigned)rfl((int)callv) + 1u;
            const unsigned par = call & 1u;
            const auto frs = rsrc(ws + OFF_RDY + c * 1024, 1024);
            const char* const exf = ws + OFF_EXF + c * 256 + lane * 4;
            const bool same = ((c ^ rank) & 4) == 0;
            const int nblk = same ? SAME_BLK : CROSS_BLK;
            const int d = same ? ((rank - c) & 3) - 1 : 3 + (rank & 3);
            const int row0 = same ? 0 : SAME_BLK * 32;
            constexpr int PARTS = 16 / PULL_U;
            const int nca = same ? CC_A : CC_A / 4 * 3, ncb = same ? NCC - CC_A : (NCC - CC_A) / 4 * 3;
            const int nb1 = nca * 4, nb2 = nca * nblk;
            const unsigned nunits = (B_OUT + (nca + ncb) * nblk) * PARTS;
            char* const q = ws + OFF_Q + ps * 128;
            char* const msb = ws + OFF_MS + c * 128;
            u64 exm = 0;
            u64 r0 = 0, r1 = 0, r2 = 0, r3 = 0;
            const unsigned ol = (lane >> 5) * (NTOT * 4) + (lane & 31) * 16;
            u64 stall = 0, npoll = 0, niter = 0;
            bool dead = false;
            auto decode = [&](unsigned un, unsigned& so, unsigned& dof, int& ms, int& need) {
                const int part = un % PARTS;
                int t = un / PARTS;
                if (t < B_OUT) {
                    if ((t >= B_WA && t < B_XB) || (t >= B_WB && t < B_XO)) {
                        int i = t < B_XB ? t - B_WA : t - B_WB + 4 * CC_A;
                        need = ms = i >> 2;
                        so = OFF_EX + par * EX_PAR + chunk_col(rank, ms) * 512 + (i & 3) * 16384;
                        dof = OFF_WS + c * WS_SRC + i * 16384;
                    } else {
                        int sl = t < B_WA ? t + NPS : t < B_WB ? t - B_XB + SLOT_A : t - B_XO + SLOT_OWN;
                        ms = sl < SLOT_A ? MS_XA : sl < SLOT_B2 ? MS_XB1 : sl < SLOT_B3 ? MS_XB2 : sl < SLOT_OWN ? MS_XB3 : MS_XO;
                        const int row = slot_row(rank, sl);
                        need = 32 + (row >> 7);
                        so = OFF_EX + par * EX_PAR + W_BYTES + row * 512;
                        dof = OFF_XS + par * XS_PAR + c * XS_SRC + sl * 16384;
                    }
                    so += part * (PULL_U * 1024);
                    dof += part * (PULL_U * 1024);
                    return;
                }
                t -= B_OUT;
                int ci, blk;
                if (t < nb1) { ci = t >> 2; blk = t & 3; }
                else if (t < nb2) { blk = t / nca; ci = t - blk * nca; }
                else { ci = t / nblk; blk = t - ci * nblk; }
                int cc = ci;
                if (!same) { int sub = ci / 3, gi = ci - sub * 3; cc = sub * 4 + (gi < (rank & 3) ? gi : gi + 1); }
                const int e = entry_of(cc & 3, d, blk);
                ms = -1;
                need = cc * 8 + (e >> 3);
                so = OFF_OS + (cc * 48 + e) * 16384 + part * (PULL_U * 1024);
                dof = (unsigned)(row0 + blk * 32 + part * (2 * PULL_U)) * (NTOT * 4) + chunk_col(c, cc) * 4;
            };
            auto put = [&](v4u (&v)[PULL_U], unsigned dof, int ms) {
                const bool out = ms < 0;
                const auto rs = rsrc(out ? P.out : ws, out ? OUT_BYTES : WS_NEED);
                const unsigned b = dof + (out ? ol : lin), st = out ? 2 * (NTOT * 4) : 1024;
#pragma unroll
                for (int u = 0; u < PULL_U; ++u) st128<AUX_DEV>(v[u], rs, b + u * st);
            };
            auto signal = [&](int ms) {
                if (ms >= 0 && lane == 0)
                    __hip_atomic_fetch_add((gu32*)(msb + ms * 1024), (unsigned)PULL_U, __ATOMIC_RELAXED, __HIP_MEMORY_SCOPE_AGENT);
            };
            auto masks = [&](v4u f) {
                r0 = ballot((int)(f[0] - call) >= 0); r1 = ballot((int)(f[1] - call) >= 0);
                r2 = ballot((int)(f[2] - call) >= 0); r3 = ballot((int)(f[3] - call) >= 0);
            };
            auto have = [&](int ms, int k) {
                if (ms >= 0) return rfl((int)((exm >> k) & 1)) != 0;
                const int sh = k >> 2;
                const unsigned m = (unsigned)((r0 >> sh) & 1) | (unsigned)((r1 >> sh) & 1) << 1 | (unsigned)((r2 >> sh) & 1) << 2 | (unsigned)((r3 >> sh) & 1) << 3;
                return rfl((int)((m >> (k & 3)) & 1)) != 0;
            };
            auto await = [&](int ms, int k) {
                const u64 t0 = __builtin_amdgcn_s_memrealtime();
                unsigned round = 1;
                for (;;) {
                    asm volatile("" ::: "memory");
                    if (ms >= 0) {
                        exm = ballot((int)(ld_sys(exf) - call) >= 0);
                    } else {
                        masks(__builtin_amdgcn_raw_buffer_load_b128(frs, lin, 0, AUX_SYS));
                    }
                    if (have(ms, k)) break;
                    if (expired(ws, call, t0, round)) { dead = true; break; }
                    __builtin_amdgcn_s_sleep(1);
                }
                if (ms < 0) { stall += __builtin_amdgcn_s_memrealtime() - t0; ++npoll; }
            };
            constexpr int NONE = -1000;
            unsigned un1 = (7 - wid) * 8 + xcd;
            unsigned so, dof, pdof = 0;
            int ms, need, pms = NONE, ppms = NONE;
            v4u pv[PULL_U];
            decode(un1, so, dof, ms, need);
            un1 += NL;
            auto flush = [&]() {
                if (pms != NONE) put(pv, pdof, pms);
                signal(ppms);
                if (pms >= 0) { WAIT_VM0(); signal(pms); }
                pms = ppms = NONE;
            };
            for (;;) {
                if (!dead && !have(ms, need)) {
                    flush();
                    await(ms, need);
                }
                asm volatile("" ::: "memory");
                v4u v[PULL_U], f;
                unsigned fe;
#pragma unroll
                for (int u = 0; u < PULL_U; ++u) v[u] = __builtin_amdgcn_raw_buffer_load_b128(prs, lin, so + u * 1024, AUX_SYS);
                if (pms != NONE) put(pv, pdof, pms);
                signal(ppms);
                const unsigned nxt = ticket(q, lane);
                ppms = pms; pms = ms; pdof = dof;
                const bool more = un1 < nunits;
                if (more) decode(un1, so, dof, ms, need);
                const bool miss = more && !have(ms, need);
                const bool ref_e = miss && ms >= 0, ref_r = miss && ms < 0;
                if (ref_e) fe = ld_sys(exf);
                if (ref_r) f = __builtin_amdgcn_raw_buffer_load_b128(frs, lin, 0, AUX_SYS);
                WAIT_VM0();
#pragma unroll
                for (int u = 0; u < PULL_U; ++u) pv[u] = dead ? v4u{0x7fc00000u, 0x7fc00000u, 0x7fc00000u, 0x7fc00000u} : v[u];
                if (!more) break;
                if (ref_e) exm = ballot((int)(fe - call) >= 0);
                if (ref_r) masks(f);
                un1 = (unsigned)rfl((int)nxt) - (call - 1u) * nunits + 2 * NL;
            }
            flush();
            STAMPV(ws, 4, stall);
            STAMPV(ws, 5, npoll);
            STAMPV(ws, 7, niter | (u64)(__builtin_amdgcn_s_getreg(20 | (0 << 6) | (3 << 11)) & 15) << 32);
        }
        STAMP(ws, 2);
        return;
    }

    const int job = blockIdx.x - NDMA;
    unsigned call;
    if (job < 160) {
        const bool isw = job < 128;
        const int gid = job & 15, sub = job >> 4, piece = isw ? gid * 8 + sub : job;
        const auto rs = rsrc(isw ? P.w : P.x, isw ? W_BYTES : X_BYTES);
        const unsigned so = isw ? (wid * 64 + (lane >> 3)) * NTOT + piece * 128 + (lane & 7) * 16 : (job - 128) * 65536 + wid * 8192 + lin;
        const unsigned st = isw ? 8 * NTOT : 1024;
        v4u v[8];
        const int to = isw ? (lane & 1 ? 4 + (gid & 3) : gid >> 2) : lane & 7;
        char* const flag = P.ws[to] + OFF_EXF + (rank * 64 + (isw ? sub * 4 + (lane & 1 ? gid >> 2 : gid & 3) : job - 96)) * 4;
        if (EXP_DELAY && isw && sub >= EXP_FIRST) __builtin_amdgcn_s_sleep(EXP_DELAY);
#pragma unroll
        for (int u = 0; u < 8; ++u) v[u] = __builtin_amdgcn_raw_buffer_load_b128(rs, so + u * st, 0, 0);
        call = (unsigned)rfl((int)callv) + 1u;
        const auto wrs = rsrc(ws, WS_NEED);
        const unsigned dof = OFF_EX + (call & 1u) * EX_PAR + piece * 65536 + wid * 8192 + lin;
#pragma unroll
        for (int u = 0; u < 8; ++u) st128<AUX_SYS>(v[u], wrs, dof + u * 1024);
        WAIT_VM0();
        __syncthreads();
        if (threadIdx.x < (isw ? 2 : 8)) st_sys(flag, call);
    } else {
        call = (unsigned)rfl((int)callv) + 1u;
    }
    STAMP(ws, 7);
    const unsigned par = call & 1u;
    const int li = job >> 3;
    const int ntiles = (xcd < 4 && li < 16) ? 2 : 1;
    const float scale = P.sx[0] * P.sw[0];
    const int wm = wid >> 1, wn = wid & 1, fr = lane & 15, fq = lane >> 4;
    const unsigned lds0 = (unsigned)(size_t)(LDS_AS char*)smem;
    volatile LDS_AS int* const dead_w = (volatile LDS_AS int*)((LDS_AS char*)smem + 3 * STAGE);
    unsigned xa[2], wa[4];
    for (int kh = 0; kh < 2; ++kh) xa[kh] = (wm * 64 + fr) * 128 + (((kh * 4 + fq) ^ (fr >> 1)) << 4);
    for (int ni = 0; ni < 4; ++ni)
        wa[ni] = 32768 + (fq * 16 + (fr >> 1)) * 128 + (((wn * 4 + ni) ^ (((fr >> 2) & 3) | ((fq & 1) << 2))) << 4) + 8 * (fr & 1);

    for (int ti = 0; ti < ntiles; ++ti) {
        int cc, rt;
        if (ti != 0) { int idx = xcd * 16 + li; cc = idx >> 1; rt = 6 + (idx & 1); }
        else if (xcd < 2) { cc = li / 3; rt = xcd * 3 + li % 3; }
        else { cc = xcd * 4 + li / 6; rt = li % 6; }
        cc = rfl(cc); rt = rfl(rt);
        const int g = cc & 3;
        bool dead = false;

        v4i acc[4][4];
#pragma unroll
        for (int i = 0; i < 4; ++i)
#pragma unroll
            for (int j = 0; j < 4; ++j) acc[i][j] = v4i{0, 0, 0, 0};

#pragma unroll 1
        for (int seg = 0; seg < 2; ++seg) {
            if (seg == 1) {
                if (wid == 0) {
                    const int m1 = rt < 3 ? MS_XA : rt < 5 ? MS_XB1 : rt < 6 ? MS_XB3 : MS_XO;
                    const unsigned n1 = rt < 3 ? SLOT_A : rt < 5 ? SLOT_B2 - SLOT_A : rt < 6 ? SLOT_OWN - SLOT_B3 : 16;
                    const u64 peers = 0xffull & ~(1ull << rank);
                    const char* mb = ws + OFF_MS + (lane & 7) * 128;
                    bool ok = wait_words<GEMM_SLEEP>(ws, call, lane, peers, mb + cc * 1024, call * 64u);
                    if (ok) ok = wait_words<GEMM_SLEEP>(ws, call, lane, (rt == 4 || rt == 5) ? peers | peers << 8 : peers,
                                               mb + (lane & 8 ? MS_XB2 : m1) * 1024, call * 16u * (lane & 8 ? SLOT_B3 - SLOT_B2 : n1));
                    __builtin_amdgcn_fence(__ATOMIC_ACQUIRE, ACQ_SCOPE);
                    *dead_w = ok ? 0 : 1;
                    WAIT_VM0();
                }
                __syncthreads();
                dead = rfl(*dead_w) != 0;
                if (wid == 0) STAMP(ws, 1 + ti * 3);
                if (dead) break;
            }
            unsigned xo[4], wo[2];
#pragma unroll
            for (int i = 0; i < 4; ++i) {
                int row = i * 64 + wid * 8 + (lane >> 3), e = rt * 8 + i * 2 + (wid >> 2), d, slot;
                if (rt >= 6) slot = SLOT_OWN + e - 48; else entry_inv(g, e, d, slot);
                xo[i] = (seg ? slot * 32 : slot_row(rank, slot)) * 512 + (row & 31) * 512 + (((lane & 7) ^ ((row >> 1) & 7)) << 4);
            }
#pragma unroll
            for (int i = 0; i < 2; ++i) {
                int k = i * 64 + wid * 8 + (lane >> 3);
                wo[i] = k * (seg ? 128 : NTOT) + (((lane & 7) ^ (((k >> 1) & 3) | (((k >> 4) & 1) << 2))) << 4);
            }
            const char* xs = seg ? ws + OFF_XS + par * XS_PAR : P.x;
            const char* wsb = seg ? ws + OFF_WS + cc * 65536 : P.w + chunk_col(rank, cc);
            const unsigned wstep = seg ? 16384 : 128 * NTOT;
            const int nt = seg ? 28 : 4;
            auto stage = [&](int t, int buf) {
                int p = t >> 2;
                p = seg ? p + (p >= rank) : 0;
                const char* xb = xs + p * XS_SRC + (t & 3) * 128;
                const char* wb = wsb + p * WS_SRC + (t & 3) * wstep;
                const unsigned l = lds0 + buf * STAGE + wid * 1024;
                unsigned keep;
                asm volatile(
                    "s_mov_b32 %0, m0\n\t"
                    "s_mov_b32 m0, %1\n\ts_nop 0\n\tglobal_load_lds_dwordx4 %4, %2\n\t"
                    "s_add_u32 m0, m0, 0x2000\n\ts_nop 0\n\tglobal_load_lds_dwordx4 %5, %2\n\t"
                    "s_add_u32 m0, m0, 0x2000\n\ts_nop 0\n\tglobal_load_lds_dwordx4 %6, %2\n\t"
                    "s_add_u32 m0, m0, 0x2000\n\ts_nop 0\n\tglobal_load_lds_dwordx4 %7, %2\n\t"
                    "s_add_u32 m0, m0, 0x2000\n\ts_nop 0\n\tglobal_load_lds_dwordx4 %8, %3\n\t"
                    "s_add_u32 m0, m0, 0x2000\n\ts_nop 0\n\tglobal_load_lds_dwordx4 %9, %3\n\t"
                    "s_mov_b32 m0, %0"
                    : "=&s"(keep)
                    : "s"(l), "s"(xb), "s"(wb), "v"(xo[0]), "v"(xo[1]), "v"(xo[2]), "v"(xo[3]), "v"(wo[0]), "v"(wo[1])
                    : "memory", "scc");
            };
            v4i xf[2][4], wf[2][4];
            auto frags = [&](int buf, int kh) {
                LDS_AS char* sb = (LDS_AS char*)smem + buf * STAGE;
#pragma unroll
                for (int ni = 0; ni < 4; ++ni) {
                    v2i lo = __builtin_amdgcn_ds_read_tr8_b64_v2i32((LDS_AS v2i*)(sb + wa[ni] + kh * 8192));
                    v2i hi = __builtin_amdgcn_ds_read_tr8_b64_v2i32((LDS_AS v2i*)(sb + wa[ni] + kh * 8192 + 1024));
                    wf[kh][ni] = v4i{lo[0], lo[1], hi[0], hi[1]};
                }
#pragma unroll
                for (int mi = 0; mi < 4; ++mi) xf[kh][mi] = *(LDS_AS v4i*)(sb + xa[kh] + mi * 2048);
            };
            auto mma = [&](int kh) {
#pragma unroll
                for (int mi = 0; mi < 4; ++mi)
#pragma unroll
                    for (int ni = 0; ni < 4; ++ni)
                        acc[mi][ni] = __builtin_amdgcn_mfma_i32_16x16x64_i8(wf[kh][ni], xf[kh][mi], acc[mi][ni], 0, 0, 0);
            };
            WAIT_VM0();
            stage(0, 0);
            stage(1, 1);
            stage(2, 2);
            asm volatile("s_waitcnt vmcnt(12)" ::: "memory");
            __builtin_amdgcn_s_barrier();
            asm volatile("" ::: "memory");
            frags(0, 0);
            int buf = 0;
            for (int t = 0; t < nt; ++t) {
                frags(buf, 1);
                __builtin_amdgcn_sched_barrier(0);
                mma(0);
                __builtin_amdgcn_sched_barrier(0);
                if (t < nt - 2) asm volatile("s_waitcnt vmcnt(6) lgkmcnt(0)" ::: "memory");
                else asm volatile("s_waitcnt vmcnt(0) lgkmcnt(0)" ::: "memory");
                __builtin_amdgcn_s_barrier();
                asm volatile("" ::: "memory");
                if (t < nt - 3) stage(t + 3, buf);
                buf = buf == 2 ? 0 : buf + 1;
                if (t < nt - 1) frags(buf, 0);
                __builtin_amdgcn_sched_barrier(0);
                mma(1);
                __builtin_amdgcn_sched_barrier(0);
            }
        }
        if (wid == 0) STAMP(ws, 2 + ti * 3);
        {
            const bool loc = rt >= 6;
            const float sc = dead ? __builtin_nanf("") : scale;
            const auto rs = loc ? rsrc(P.out, OUT_BYTES) : rsrc(ws, WS_NEED);
            const unsigned pitch = loc ? NTOT * 4 : 512;
            const unsigned base = loc ? (unsigned)(rt - 6) * 256 * (NTOT * 4) + chunk_col(rank, cc) * 4 : OFF_OS + (cc * 6 + rt) * 131072;
            const unsigned lo = (wm * 64 + fr) * pitch + wn * 256 + fq * 16;
#pragma unroll
            for (int mi = 0; mi < 4; ++mi)
#pragma unroll
                for (int ni = 0; ni < 4; ++ni) {
                    v4f o;
#pragma unroll
                    for (int j = 0; j < 4; ++j) o[j] = silu(acc[mi][ni][j], sc);
                    st128<EPI_AUX>(__builtin_bit_cast(v4u, o), rs, lo + base + mi * 16 * pitch + ni * 64);
                }
            WAIT_VM0();
            __syncthreads();
            if (!loc && threadIdx.x < 8) {
                int d, slot;
                entry_inv(g, rt * 8 + threadIdx.x, d, slot);
                st_sys(P.ws[dest_rank(rank, d) & 7] + OFF_RDY + ((rank * NCC + cc) * 8 + rt) * 4, call);
            }
        }
        if (wid == 0) STAMP(ws, 3 + ti * 3);
    }
}

}

extern "C" void kernel_launch(int rank, int nranks, ncclComm_t comm,
                              hipStream_t stream,
                              void* const* d_in, const int* in_sizes, int n_in,
                              void* d_out, int out_size,
                              const int* in_dtypes, int out_dtype,
                              void* const* d_out_all,
                              void* const* d_ws_all, size_t ws_bytes) {
    if (nranks != NR || (unsigned)rank >= (unsigned)NR || ws_bytes < WS_NEED || n_in != 4 ||
        in_sizes[0] != (int)X_BYTES || in_sizes[1] != (int)W_BYTES || out_size != 512 * NTOT) {
        fprintf(stderr, "unexpected problem shape\n");
        return;
    }
    Params p;
    p.x = (const char*)d_in[0]; p.w = (const char*)d_in[1];
    p.sx = (const float*)d_in[2]; p.sw = (const float*)d_in[3];
    p.out = (char*)d_out;
    for (int i = 0; i < NR; ++i) { p.ws[i] = (char*)d_ws_all[phys(i)]; if (phys(i) == rank) p.rank = i; }
    hipLaunchKernelGGL(fused_gemm_scatter, dim3(NWG), dim3(512), 0, stream, p);
    HIP_CHECK(hipGetLastError());
}
```

```cpp
#include <hip/hip_runtime.h>
#include <rccl/rccl.h>
#include <cstdio>

#define HIP_CHECK(expr) do { hipError_t _e = (expr); if (_e != hipSuccess) { \
    fprintf(stderr, "HIP error %s at %s:%d\n", hipGetErrorString(_e), __FILE__, __LINE__); \
    return; } } while (0)

#define PULL_U 2
#define PULL_NL 40
#define PUSH_NW 1
#define PUSH_NP 4
#define PUSH_WIN 6
#define GEMM_SLEEP 64
#define RELAY 1
#define CREDIT 0
#define PACE 2100
#define PERM 0x75416320u
#define EXP_FIRST 1
#define EXP_DELAY 100
#define PART_T0 600u
#define PART_DT 300u
#define ACQ_SCOPE ""

namespace {

typedef int v2i __attribute__((ext_vector_type(2)));
typedef int v4i __attribute__((ext_vector_type(4)));
typedef unsigned v4u __attribute__((ext_vector_type(4)));
typedef float v4f __attribute__((ext_vector_type(4)));
typedef unsigned long long u64;
#define LDS_AS __attribute__((address_space(3)))
#define GLB_AS __attribute__((address_space(1)))
typedef GLB_AS unsigned gu32;

constexpr int NR = 8;
constexpr int NTOT = 16384;
constexpr int SAME_BLK = 7, CROSS_BLK = 9;
constexpr int SLOT_A = 28;
constexpr int SLOT_B2 = 42, SLOT_B3 = 49, SLOT_OWN = 57;
constexpr int NSLOT = SLOT_OWN + 16;
constexpr int NCC = 32;
constexpr int CC_A = 8;
constexpr int NDMA = 64;
constexpr int NWG = 256;
constexpr int NL = PULL_NL;
constexpr int MS_XA = 32, MS_XB1 = 33, MS_XB2 = 34, MS_XB3 = 35, MS_XO = 36, MS_PART = 37;
constexpr int NPS = 8 * PUSH_NW * PUSH_NP / 2;
constexpr int NXA = 24;
constexpr int B_WA = NXA - NPS, B_XB = B_WA + 4 * CC_A;
constexpr int CC_R = 28, NEED_RELAY = 7, NEED_EX = 256;
static_assert(NPS <= NXA, "pushed slots");

constexpr unsigned OFF_CNT = 0;
constexpr unsigned OFF_TMO = 8192;
constexpr unsigned OFF_Q = 12288;
constexpr unsigned OFF_RDY = 16384;
constexpr unsigned OFF_EXF = 32768;
constexpr unsigned OFF_DBG = 65536;
constexpr unsigned OFF_MS = 262144;
constexpr unsigned XS_SRC = NSLOT * 16384;
constexpr unsigned XS_PAR = NR * XS_SRC;
constexpr unsigned WS_SRC = NCC * 65536;
constexpr unsigned WS_PAR = NR * WS_SRC;
constexpr unsigned EX_PAR = 160 * 65536;
constexpr unsigned OFF_XS = 1u << 20;
constexpr unsigned OFF_WS = OFF_XS + 2 * XS_PAR;
constexpr unsigned OFF_EX = OFF_WS + 2 * WS_PAR;
constexpr unsigned OFF_OS = OFF_EX + 2 * EX_PAR;
constexpr unsigned OFF_ZERO = OFF_OS + NCC * 6 * 131072;
constexpr unsigned WS_NEED = OFF_ZERO + 4096;
constexpr unsigned OUT_BYTES = 512u * NTOT * 4u;
constexpr unsigned X_BYTES = 4096u * 512u, W_BYTES = 512u * NTOT;

constexpr u64 DEADLINE = 50000000ull;
constexpr int STAGE = 49152;
constexpr int AUX_SYS = 17;
constexpr int AUX_DEV = 16;

struct Params {
    const char* x; const char* w; const float* sx; const float* sw;
    char* out; char* ws[NR];
    int rank;
};

__device__ __forceinline__ int rfl(int v) { return __builtin_amdgcn_readfirstlane(v); }
__device__ __forceinline__ u64 ballot(bool b) { return __builtin_amdgcn_ballot_w64(b); }
__device__ __forceinline__ auto rsrc(const void* p, unsigned bytes) {
    return __builtin_amdgcn_make_buffer_rsrc((void*)p, 0, bytes, 0x00020000);
}
template <int AUX, class R> __device__ __forceinline__ void st128(v4u v, R rs, unsigned off) {
    __builtin_amdgcn_raw_buffer_store_b128(v, rs, off, 0, AUX);
}
__host__ __device__ __forceinline__ int phys(int l) { return (PERM >> (4 * l)) & 7; }
__device__ __forceinline__ int dest_rank(int c, int d) { return d < 3 ? (c & 4) | ((c + 1 + d) & 3) : ((c & 4) ^ 4) | (d - 3); }
__device__ __forceinline__ int slot_row(int c, int s) {
    if (s >= SLOT_OWN) return phys(c) * 512 + (s - SLOT_OWN) * 32;
    int j, d;
    if (s < SLOT_B3) { j = s / 7; d = s - j * 7; } else { j = 7 + ((s - SLOT_B3) >> 2); d = 3 + ((s - SLOT_B3) & 3); }
    return phys(dest_rank(c, d)) * 512 + (d < 3 ? 0 : SAME_BLK * 32) + j * 32;
}
__device__ __forceinline__ int xslot(int n, int skip) {
    if (n < 42) { int j = n / 6, i = n - j * 6; return j * 7 + (i < skip ? i : i + 1); }
    n -= 42;
    if (skip < 3) return SLOT_B3 + n;
    int j = n / 3, i = n - j * 3;
    return SLOT_B3 + j * 4 + (i < skip - 3 ? i : i + 1);
}
__device__ __forceinline__ int entry_of(int g, int d, int j) {
    int i = d < 3 + g ? d : d - 1;
    return j < 7 ? j * 6 + i : 42 + (j - 7) * 3 + (i - 3);
}
__device__ __forceinline__ void entry_inv(int g, int e, int& d, int& slot) {
    int j, i;
    if (e < 42) { j = e / 6; i = e - j * 6; } else { j = 7 + (e - 42) / 3; i = 3 + (e - 42) % 3; }
    d = i < 3 + g ? i : i + 1;
    slot = j < 7 ? j * 7 + d : SLOT_B3 + (j - 7) * 4 + (d - 3);
}
__device__ __forceinline__ int chunk_col(int c, int cc) {
    int g = cc & 3, gid = (c & 4) ? g * 4 + (c & 3) : (c & 3) * 4 + g;
    return gid * 1024 + (cc >> 2) * 128;
}
__device__ __forceinline__ unsigned ld_sys(const char* p) {
    return __hip_atomic_load((gu32*)p, __ATOMIC_RELAXED, __HIP_MEMORY_SCOPE_SYSTEM);
}
__device__ __forceinline__ void st_sys(char* p, unsigned v) {
    __hip_atomic_store((gu32*)p, v, __ATOMIC_RELAXED, __HIP_MEMORY_SCOPE_SYSTEM);
}
__device__ __forceinline__ void set_tmo(char* ws, unsigned call) {
    __hip_atomic_store((gu32*)(ws + OFF_TMO), call, __ATOMIC_RELAXED, __HIP_MEMORY_SCOPE_AGENT);
}
__device__ __forceinline__ bool expired(char* ws, unsigned call, u64 t0, unsigned& round) {
    bool out = __builtin_amdgcn_s_memrealtime() - t0 > DEADLINE;
    if ((round++ & 7u) == 0u) out = out || rfl((int)ld_sys(ws + OFF_TMO)) == (int)call;
    if (out) set_tmo(ws, call);
    return out;
}
template <int SLEEP> __device__ __forceinline__ bool wait_words(char* ws, unsigned call, int lane, u64 mask, const char* p, unsigned target) {
    const u64 t0 = __builtin_amdgcn_s_memrealtime();
    if (!((mask >> lane) & 1)) p = ws + OFF_TMO;
    for (;;) {
        const unsigned v = ld_sys(p);
        if (ballot(v == call) >> 63) return false;
        if ((ballot((int)(v - target) < 0) & mask) == 0) return true;
        if (__builtin_amdgcn_s_memrealtime() - t0 > DEADLINE) { set_tmo(ws, call); return false; }
        __builtin_amdgcn_s_sleep(SLEEP);
    }
}
#define WAIT_VM0() asm volatile("s_waitcnt vmcnt(0)" ::: "memory")
#define STR2(x) #x
#define STR(x) STR2(x)

#define STAMPV(ws, idx, v) do {} while (0)
#define STAMP(ws, idx) STAMPV(ws, idx, __builtin_amdgcn_s_memrealtime())

__device__ __forceinline__ float silu(int acc, float s) {
    float y = (float)acc * s;
    float c = fminf(fmaxf(y, -60.0f), 60.0f);
    float e = __builtin_amdgcn_exp2f(c * -1.4426950408889634f);
    return y * __builtin_amdgcn_rcpf(1.0f + e);
}
__device__ __forceinline__ unsigned ticket(char* p, int lane) {
    unsigned v = 0, z = 0;
    asm volatile("" : "+v"(z));
    if (lane == 0) v = __hip_atomic_fetch_add((gu32*)(p + z), 1u, __ATOMIC_RELAXED, __HIP_MEMORY_SCOPE_AGENT);
    return v;
}

__global__ void __launch_bounds__(512, 2) fused_gemm_scatter(Params P) {
    __shared__ __attribute__((aligned(16))) char smem[3 * STAGE + 16];
    const int rank = P.rank;
    const int lane = threadIdx.x & 63, wid = rfl(threadIdx.x >> 6);
    char* const ws = P.ws[rank];
    const int xcd = blockIdx.x & 7;
    const unsigned lin = lane * 16;
    STAMP(ws, 0);
    if (blockIdx.x < 8 && (!RELAY || wid != 0 || blockIdx.x >= 4 || (int)blockIdx.x == (rank & 3))) return;
    const unsigned callv = ticket(ws + OFF_CNT + (blockIdx.x * 8 + wid) * 4, lane);
    if (blockIdx.x < 8) {
        const unsigned call = (unsigned)rfl((int)callv) + 1u;
        wait_words<32>(ws, call, lane, 1, ws + OFF_MS + ((CC_R + blockIdx.x) * 8 + (rank ^ 4)) * 128, call * 64u);
        if (lane == 0) st_sys(P.ws[((rank & 4) ^ 4) | blockIdx.x] + OFF_RDY + (rank * NCC * 8 + NEED_RELAY) * 4, call);
        return;
    }

    if (blockIdx.x < NDMA) {
        const int ps = blockIdx.x >> 3, c = (rank + ps) & 7;
        char* const pws = P.ws[c];
        const auto prs = rsrc(pws, WS_NEED);
        const bool same = ((c ^ rank) & 4) == 0;
        const int d = same ? ((rank - c) & 3) - 1 : 3 + (rank & 3);
        if (wid < PUSH_NW) {
            const auto xrs = rsrc(P.x, X_BYTES);
            const int h0 = (xcd * PUSH_NW + wid) * PUSH_NP;
            v4u v[PUSH_NP * 8];
            unsigned dof[PUSH_NP];
#pragma unroll
            for (int i = 0; i < PUSH_NP; ++i) {
                const int sl = xslot((h0 + i) >> 1, d), half = (h0 + i) & 1;
                const unsigned so = (slot_row(c, sl) + half * 16) * 512;
                dof[i] = OFF_XS + rank * XS_SRC + sl * 16384 + half * 8192;
#pragma unroll
                for (int u = 0; u < 8; ++u) v[i * 8 + u] = __builtin_amdgcn_raw_buffer_load_b128(xrs, lin, so + u * 1024, 0);
            }
            const unsigned call = (unsigned)rfl((int)callv) + 1u;
            const unsigned po = (call & 1u) * XS_PAR + lin;
#pragma unroll
            for (int u = 0; u < PUSH_NP * 8; ++u) {
                if (u >= PUSH_WIN) asm volatile("s_waitcnt vmcnt(%0)" :: "n"(PUSH_WIN - 1) : "memory");
                st128<AUX_SYS>(v[u], prs, po + dof[u >> 3] + (u & 7) * 1024);
            }
            WAIT_VM0();
            if (lane == 0) __hip_atomic_fetch_add((gu32*)(pws + OFF_MS + rank * 128 + MS_XA * 1024), 8u * PUSH_NP, __ATOMIC_RELAXED, __HIP_MEMORY_SCOPE_SYSTEM);
        }
        STAMP(ws, 1);
        if ((7 - wid) * 8 + xcd < NL) {
            const unsigned call = (unsigned)rfl((int)callv) + 1u;
            const unsigned par = call & 1u;
            const auto frs = rsrc(ws + OFF_RDY + c * 1024, 1024);
            const char* const exf = ws + OFF_EXF + c * 256 + lane * 4;
            const int nblk = same ? SAME_BLK : CROSS_BLK;
            const int dc = same ? ((c - rank) & 3) - 1 : 3 + (c & 3);
            const int B_WB = B_XB + SLOT_OWN - nblk - NXA;
            const auto ors = rsrc(P.out, OUT_BYTES);
            const float scale = P.sx[0] * P.sw[0];
            const int row0 = same ? 0 : SAME_BLK * 32;
            constexpr int PARTS = 16 / PULL_U;
            const int nca = same ? CC_A : CC_A / 4 * 3, ncb = same ? NCC - CC_A : (NCC - CC_A) / 4 * 3;
            const int nb1 = nca * 4, nb2 = nca * nblk;
            const bool skip = RELAY && same, relay = RELAY && !same && ((c ^ rank) & 3) != 0;
            const int B_XO = B_WB + 4 * (NCC - CC_A) - (skip ? 4 : 0), B_OUT = B_XO + 16;
            const unsigned nunits = (B_OUT + (nca + ncb) * nblk + (relay ? 4 : 0)) * PARTS;
            char* const q = ws + OFF_Q + ps * 128;
            u64 exm = 0;
            u64 r0 = 0, r1 = 0, r2 = 0, r3 = 0;
            const unsigned ol = (lane >> 5) * (NTOT * 4) + (lane & 31) * 16;
            u64 stall = 0, npoll = 0, niter = 0;
            bool dead = false;
            auto decode = [&](unsigned un, unsigned& so, unsigned& dof, int& ms, int& need) {
                const int part = un % PARTS;
                int t = un / PARTS;
                const bool rl = relay && t >= B_OUT + nb1 && t < B_OUT + nb1 + 4;
                if (t < B_OUT || rl) {
                    if (rl) {
                        const int i = t - B_OUT - nb1, src = c ^ 4;
                        ms = (CC_R + (c & 3)) * 8 + src;
                        need = NEED_RELAY;
                        so = OFF_WS + par * WS_PAR + src * WS_SRC + (CC_R + (rank & 3)) * 65536 + i * 16384;
                        dof = OFF_WS + par * WS_PAR + src * WS_SRC + (CC_R + (c & 3)) * 65536 + i * 16384;
                    } else if ((t >= B_WA && t < B_XB) || (t >= B_WB && t < B_XO)) {
                        int i = t < B_XB ? t - B_WA : t - B_WB + 4 * CC_A;
                        if (skip && i >= 4 * (CC_R + (c & 3))) i += 4;
                        const int cc = i >> 2;
                        ms = cc * 8 + c;
                        need = NEED_EX + cc;
                        so = OFF_EX + par * EX_PAR + chunk_col(rank, cc) * 512 + (i & 3) * 16384;
                        dof = OFF_WS + par * WS_PAR + c * WS_SRC + i * 16384;
                    } else {
                        int sl = t < B_WB ? xslot(t < B_WA ? t + NPS : t - B_XB + NXA, dc) : t - B_XO + SLOT_OWN;
                        ms = (sl < SLOT_A ? MS_XA : sl < SLOT_B2 ? MS_XB1 : sl < SLOT_B3 ? MS_XB2 : sl < SLOT_OWN ? MS_XB3 : MS_XO) * 8 + c;
                        const int row = slot_row(rank, sl);
                        need = NEED_EX + 32 + (row >> 7);
                        so = OFF_EX + par * EX_PAR + W_BYTES + row * 512;
                        dof = OFF_XS + par * XS_PAR + c * XS_SRC + sl * 16384;
                    }
                    so += part * (PULL_U * 1024);
                    dof += part * (PULL_U * 1024);
                    return;
                }
                t -= B_OUT;
                if (relay && t >= nb1) t -= 4;
                int ci, blk;
                if (t < nb1) { ci = t >> 2; blk = t & 3; }
                else if (t < nb2) { blk = t / nca; ci = t - blk * nca; }
                else { ci = t / nblk; blk = t - ci * nblk; }
                int cc = ci;
                if (!same) { int sub = ci / 3, gi = ci - sub * 3; cc = sub * 4 + (gi < (rank & 3) ? gi : gi + 1); }
                const int e = entry_of(cc & 3, d, blk);
                ms = -1;
                need = cc * 8 + (e >> 3);
                so = OFF_OS + (cc * 48 + e) * 16384 + part * (PULL_U * 1024);
                dof = (unsigned)(row0 + blk * 32 + part * (2 * PULL_U)) * (NTOT * 4) + chunk_col(c, cc) * 4;
            };
            auto put = [&](v4u (&v)[PULL_U], unsigned dof, int ms) {
                if (ms < 0) {
                    const float sc = dead ? __builtin_nanf("") : scale;
#pragma unroll
                    for (int u = 0; u < PULL_U; ++u) {
                        v4f o;
#pragma unroll
                        for (int j = 0; j < 4; ++j) o[j] = silu((int)v[u][j], sc);
                        st128<AUX_SYS>(__builtin_bit_cast(v4u, o), ors, dof + ol + u * 2 * (NTOT * 4));
                    }
                } else {
                    const auto rs = rsrc(ws, WS_NEED);
#pragma unroll
                    for (int u = 0; u < PULL_U; ++u) st128<AUX_SYS>(v[u], rs, dof + lin + u * 1024);
                }
            };
            auto signal = [&](int ms) {
                if (ms >= 0 && lane == 0)
                    __hip_atomic_fetch_add((gu32*)(ws + OFF_MS + ms * 128), (unsigned)PULL_U, __ATOMIC_RELAXED, __HIP_MEMORY_SCOPE_AGENT);
            };
            auto masks = [&](v4u f) {
                r0 = ballot((int)(f[0] - call) >= 0); r1 = ballot((int)(f[1] - call) >= 0);
                r2 = ballot((int)(f[2] - call) >= 0); r3 = ballot((int)(f[3] - call) >= 0);
            };
            auto have = [&](int k) {
                if (k >= NEED_EX) return rfl((int)((exm >> (k - NEED_EX)) & 1)) != 0;
                const int sh = k >> 2;
                const unsigned m = (unsigned)((r0 >> sh) & 1) | (unsigned)((r1 >> sh) & 1) << 1 | (unsigned)((r2 >> sh) & 1) << 2 | (unsigned)((r3 >> sh) & 1) << 3;
                return rfl((int)((m >> (k & 3)) & 1)) != 0;
            };
            auto await = [&](int k) {
                const u64 t0 = __builtin_amdgcn_s_memrealtime();
                unsigned round = 1;
                for (;;) {
                    asm volatile("" ::: "memory");
                    if (k >= NEED_EX) {
                        exm = ballot((int)(ld_sys(exf) - call) >= 0);
                    } else {
                        masks(__builtin_amdgcn_raw_buffer_load_b128(frs, lin, 0, AUX_SYS));
                    }
                    if (have(k)) break;
                    if (expired(ws, call, t0, round)) { dead = true; break; }
                    __builtin_amdgcn_s_sleep(1);
                }
                if (k < NEED_EX) { stall += __builtin_amdgcn_s_memrealtime() - t0; ++npoll; }
            };
            constexpr int NONE = -1000;
            unsigned tnext = (unsigned)__builtin_amdgcn_s_memrealtime() << 4;
            unsigned un1 = (7 - wid) * 8 + xcd;
            unsigned so, dof, pdof = 0;
            int ms, need, pms = NONE, ppms = NONE;
            v4u pv[PULL_U], v2[PULL_U];
            bool pok = false;
            const char* const pcnt = ws + OFF_MS + MS_PART * 1024;
            decode(un1, so, dof, ms, need);
            un1 += NL;
            for (int u = 0; u < PULL_U; ++u) pv[u] = v2[u] = v4u{0, 0, 0, 0};
            auto add2 = [&]() {
#pragma unroll
                for (int u = 0; u < PULL_U; ++u) { pv[u] += v2[u]; asm volatile("" : "+v"(pv[u])); }
            };
            auto flush = [&]() {
                add2();
                if (pms != NONE) put(pv, pdof, pms);
                signal(ppms);
                if (pms >= 0) { WAIT_VM0(); signal(pms); }
                pms = ppms = NONE;
            };
            for (;;) {
                if (!dead && (!have(need) || (ms < 0 && !pok))) {
                    flush();
                    if (!have(need)) await(need);
                    if (ms < 0 && !pok && !dead) { dead = !wait_words<1>(ws, call, lane, 1, pcnt, call * 192u); pok = true; }
                }
                if (PACE) {
                    unsigned now = (unsigned)__builtin_amdgcn_s_memrealtime() << 4;
                    if ((int)(now - tnext) >= 0) { if ((int)(now - tnext) > CREDIT) tnext = now - CREDIT; }
                    else while ((int)(((unsigned)__builtin_amdgcn_s_memrealtime() << 4) - tnext) < 0) __builtin_amdgcn_s_sleep(1);
                    tnext += PACE;
                }
                asm volatile("" ::: "memory");
                v4u v[PULL_U], f;
                unsigned fe, fp;
#pragma unroll
                for (int u = 0; u < PULL_U; ++u) v[u] = __builtin_amdgcn_raw_buffer_load_b128(prs, lin, so + u * 1024, AUX_SYS);
                add2();
                if (pms != NONE) put(pv, pdof, pms);
                signal(ppms);
                const unsigned nxt = ticket(q, lane);
                ppms = pms; pms = ms; pdof = dof;
                const bool more = un1 < nunits;
                if (more) decode(un1, so, dof, ms, need);
                const bool miss = more && !have(need);
                const bool ref_e = miss && need >= NEED_EX, ref_r = miss && need < NEED_EX;
                if (ref_e) fe = ld_sys(exf);
                if (ref_r) f = __builtin_amdgcn_raw_buffer_load_b128(frs, lin, 0, AUX_SYS);
                const bool ref_p = more && ms < 0 && !pok;
                if (ref_p) fp = ld_sys(pcnt);
                asm volatile("" ::: "memory");
#pragma unroll
                for (int u = 0; u < PULL_U; ++u)
                    v2[u] = __builtin_amdgcn_raw_buffer_load_b128(ors, ol, pms < 0 ? pdof + u * 2 * (NTOT * 4) : OUT_BYTES, AUX_DEV);
                asm volatile("s_waitcnt vmcnt(%0)" :: "n"(PULL_U) : "memory");
#pragma unroll
                for (int u = 0; u < PULL_U; ++u) pv[u] = v[u];
                if (!more) break;
                if (ref_e) exm = ballot((int)(fe - call) >= 0);
                if (ref_r) masks(f);
                if (ref_p) pok = rfl((int)(fp - call * 192u)) >= 0;
                un1 = (unsigned)rfl((int)nxt) - (call - 1u) * nunits + 2 * NL;
            }
            flush();
            STAMPV(ws, 4, stall);
            STAMPV(ws, 5, npoll);
            STAMPV(ws, 7, niter | (u64)(__builtin_amdgcn_s_getreg(20 | (0 << 6) | (3 << 11)) & 15) << 32);
        }
        STAMP(ws, 2);
        return;
    }

    const int job = blockIdx.x - NDMA;
    const unsigned tstart = (unsigned)__builtin_amdgcn_s_memrealtime();
    unsigned call;
    if (job < 160) {
        const bool isw = job < 128;
        const int gid = job & 15, sub = job >> 4, piece = isw ? gid * 8 + sub : job;
        const auto rs = rsrc(isw ? P.w : P.x, isw ? W_BYTES : X_BYTES);
        const unsigned so = isw ? (wid * 64 + (lane >> 3)) * NTOT + piece * 128 + (lane & 7) * 16 : (job - 128) * 65536 + wid * 8192 + lin;
        const unsigned st = isw ? 8 * NTOT : 1024;
        v4u v[8];
        const int to = isw ? (lane & 1 ? 4 + (gid & 3) : gid >> 2) : lane & 7;
        char* const flag = P.ws[to] + OFF_EXF + (rank * 64 + (isw ? sub * 4 + (lane & 1 ? gid >> 2 : gid & 3) : job - 96)) * 4;
        if (EXP_DELAY && isw && sub >= EXP_FIRST) __builtin_amdgcn_s_sleep(EXP_DELAY);
#pragma unroll
        for (int u = 0; u < 8; ++u) v[u] = __builtin_amdgcn_raw_buffer_load_b128(rs, so + u * st, 0, 0);
        call = (unsigned)rfl((int)callv) + 1u;
        const auto wrs = rsrc(ws, WS_NEED);
        const unsigned dof = OFF_EX + (call & 1u) * EX_PAR + piece * 65536 + wid * 8192 + lin;
#pragma unroll
        for (int u = 0; u < 8; ++u) st128<AUX_SYS>(v[u], wrs, dof + u * 1024);
        WAIT_VM0();
        __syncthreads();
        if (threadIdx.x < (isw ? 2 : 8) && to != rank) st_sys(flag, call);
    } else {
        call = (unsigned)rfl((int)callv) + 1u;
    }
    STAMP(ws, 7);
    const unsigned par = call & 1u;
    const int li = job >> 3;
    const float scale = P.sx[0] * P.sw[0];
    const int wm = wid >> 1, wn = wid & 1, fr = lane & 15, fq = lane >> 4;
    const unsigned lds0 = (unsigned)(size_t)(LDS_AS char*)smem;
    volatile LDS_AS int* const dead_w = (volatile LDS_AS int*)((LDS_AS char*)smem + 3 * STAGE);
    unsigned xa[2], wa[4];
    for (int kh = 0; kh < 2; ++kh) xa[kh] = (wm * 64 + fr) * 128 + (((kh * 4 + fq) ^ (fr >> 1)) << 4);
    for (int ni = 0; ni < 4; ++ni)
        wa[ni] = 32768 + (fq * 16 + (fr >> 1)) * 128 + (((wn * 4 + ni) ^ (((fr >> 2) & 3) | ((fq & 1) << 2))) << 4) + 8 * (fr & 1);

    const int tend = (xcd < 4 && li < 16) ? 4 : 3;
    for (int ti = xcd < 4 ? 2 : 0; ti < tend; ++ti) {
        int cc = 0, rt, col;
        if (ti < 2) {
            const int sub = ti * 4 + xcd - 4, i12 = li >> 1, me = rank & 3;
            const int gid = (rank & 4) ? (i12 / 3) * 4 + (i12 % 3 < me ? i12 % 3 : i12 % 3 + 1) : (i12 < 4 * me ? i12 : i12 + 4);
            rt = 6 + (li & 1);
            col = gid * 1024 + sub * 128;
            while ((unsigned)__builtin_amdgcn_s_memrealtime() - tstart < PART_T0 + sub * PART_DT) __builtin_amdgcn_s_sleep(8);
        } else {
            if (ti == 3) { int idx = xcd * 16 + li; cc = idx >> 1; rt = 6 + (idx & 1); }
            else if (xcd < 2) { cc = li / 3; rt = xcd * 3 + li % 3; }
            else { cc = xcd * 4 + li / 6; rt = li % 6; }
            col = chunk_col(rank, cc);
        }
        cc = rfl(cc); rt = rfl(rt); col = rfl(col);
        const int g = cc & 3;
        bool dead = false;

        v4i acc[4][4];
#pragma unroll
        for (int i = 0; i < 4; ++i)
#pragma unroll
            for (int j = 0; j < 4; ++j) acc[i][j] = v4i{0, 0, 0, 0};

#pragma unroll 1
        for (int seg = 0; seg < (ti < 2 ? 1 : 2); ++seg) {
            if (seg == 1) {
                if (wid == 0) {
                    const int m1 = rt < 3 ? MS_XA : rt < 5 ? MS_XB1 : rt < 6 ? MS_XB3 : MS_XO;
                    const unsigned n1 = rt < 3 ? NXA : rt < 5 ? 12 : rt < 6 ? (((lane ^ rank) & 4) ? 6 : 8) : 16;
                    const u64 peers = 0xffull & ~(1ull << rank);
                    const char* mb = ws + OFF_MS + (lane & 7) * 128;
                    bool ok = wait_words<GEMM_SLEEP>(ws, call, lane, peers, mb + cc * 1024, call * 64u);
                    if (ok) ok = wait_words<GEMM_SLEEP>(ws, call, lane, (rt == 4 || rt == 5) ? peers | peers << 8 : peers,
                                               mb + (lane & 8 ? MS_XB2 : m1) * 1024, call * 16u * (lane & 8 ? 6 : n1));
                    __builtin_amdgcn_fence(__ATOMIC_ACQUIRE, ACQ_SCOPE);
                    *dead_w = ok ? 0 : 1;
                    WAIT_VM0();
                }
                __syncthreads();
                dead = rfl(*dead_w) != 0;
                if (wid == 0) STAMP(ws, ti * 3 - 5);
                if (dead) break;
            }
            unsigned xo[4], wo[2];
            int own[4];
#pragma unroll
            for (int i = 0; i < 4; ++i) {
                int row = i * 64 + wid * 8 + (lane >> 3), e = rt * 8 + i * 2 + (wid >> 2), d, slot;
                if (rt >= 6) { slot = SLOT_OWN + e - 48; own[i] = -1; } else { entry_inv(g, e, d, slot); own[i] = seg ? dest_rank(rank, d) : -1; }
                xo[i] = (seg ? slot * 32 : slot_row(rank, slot)) * 512 + (row & 31) * 512 + (((lane & 7) ^ ((row >> 1) & 7)) << 4);
            }
#pragma unroll
            for (int i = 0; i < 2; ++i) {
                int k = i * 64 + wid * 8 + (lane >> 3);
                wo[i] = k * (seg ? 128 : NTOT) + (((lane & 7) ^ (((k >> 1) & 3) | (((k >> 4) & 1) << 2))) << 4);
            }
            const char* xs = seg ? ws + OFF_XS + par * XS_PAR : P.x;
            const char* wsb = seg ? ws + OFF_WS + par * WS_PAR + cc * 65536 : P.w + col;
            const unsigned wstep = seg ? 16384 : 128 * NTOT;
            const int nt = seg ? 28 : 4;
            auto stage = [&](int t, int buf) {
                int p = t >> 2;
                p = seg ? p + (p >= rank) : 0;
                const char* xb = xs + p * XS_SRC + (t & 3) * 128;
                const char* wb = wsb + p * WS_SRC + (t & 3) * wstep;
                const unsigned l = lds0 + buf * STAGE + wid * 1024;
                const unsigned z = OFF_ZERO - (OFF_XS + par * XS_PAR + p * XS_SRC + (t & 3) * 128) + lin;
                const unsigned x0 = own[0] == p ? z : xo[0], x1 = own[1] == p ? z : xo[1], x2 = own[2] == p ? z : xo[2], x3 = own[3] == p ? z : xo[3];
                unsigned keep;
                asm volatile(
                    "s_mov_b32 %0, m0\n\t"
                    "s_mov_b32 m0, %1\n\ts_nop 0\n\tglobal_load_lds_dwordx4 %4, %2\n\t"
                    "s_add_u32 m0, m0, 0x2000\n\ts_nop 0\n\tglobal_load_lds_dwordx4 %5, %2\n\t"
                    "s_add_u32 m0, m0, 0x2000\n\ts_nop 0\n\tglobal_load_lds_dwordx4 %6, %2\n\t"
                    "s_add_u32 m0, m0, 0x2000\n\ts_nop 0\n\tglobal_load_lds_dwordx4 %7, %2\n\t"
                    "s_add_u32 m0, m0, 0x2000\n\ts_nop 0\n\tglobal_load_lds_dwordx4 %8, %3\n\t"
                    "s_add_u32 m0, m0, 0x2000\n\ts_nop 0\n\tglobal_load_lds_dwordx4 %9, %3\n\t"
                    "s_mov_b32 m0, %0"
                    : "=&s"(keep)
                    : "s"(l), "s"(xb), "s"(wb), "v"(x0), "v"(x1), "v"(x2), "v"(x3), "v"(wo[0]), "v"(wo[1])
                    : "memory", "scc");
            };
            v4i xf[2][4], wf[2][4];
            auto frags = [&](int buf, int kh) {
                LDS_AS char* sb = (LDS_AS char*)smem + buf * STAGE;
#pragma unroll
                for (int ni = 0; ni < 4; ++ni) {
                    v2i lo = __builtin_amdgcn_ds_read_tr8_b64_v2i32((LDS_AS v2i*)(sb + wa[ni] + kh * 8192));
                    v2i hi = __builtin_amdgcn_ds_read_tr8_b64_v2i32((LDS_AS v2i*)(sb + wa[ni] + kh * 8192 + 1024));
                    wf[kh][ni] = v4i{lo[0], lo[1], hi[0], hi[1]};
                }
#pragma unroll
                for (int mi = 0; mi < 4; ++mi) xf[kh][mi] = *(LDS_AS v4i*)(sb + xa[kh] + mi * 2048);
            };
            auto mma = [&](int kh) {
#pragma unroll
                for (int mi = 0; mi < 4; ++mi)
#pragma unroll
                    for (int ni = 0; ni < 4; ++ni)
                        acc[mi][ni] = __builtin_amdgcn_mfma_i32_16x16x64_i8(wf[kh][ni], xf[kh][mi], acc[mi][ni], 0, 0, 0);
            };
            WAIT_VM0();
            stage(0, 0);
            stage(1, 1);
            stage(2, 2);
            asm volatile("s_waitcnt vmcnt(12)" ::: "memory");
            __builtin_amdgcn_s_barrier();
            asm volatile("" ::: "memory");
            frags(0, 0);
            int buf = 0;
            for (int t = 0; t < nt; ++t) {
                frags(buf, 1);
                __builtin_amdgcn_sched_barrier(0);
                mma(0);
                __builtin_amdgcn_sched_barrier(0);
                if (t < nt - 2) asm volatile("s_waitcnt vmcnt(6) lgkmcnt(0)" ::: "memory");
                else asm volatile("s_waitcnt vmcnt(0) lgkmcnt(0)" ::: "memory");
                __builtin_amdgcn_s_barrier();
                asm volatile("" ::: "memory");
                if (t < nt - 3) stage(t + 3, buf);
                buf = buf == 2 ? 0 : buf + 1;
                if (t < nt - 1) frags(buf, 0);
                __builtin_amdgcn_sched_barrier(0);
                mma(1);
                __builtin_amdgcn_sched_barrier(0);
            }
        }
        if (wid == 0 && ti >= 2) STAMP(ws, ti * 3 - 4);
        {
            const bool loc = rt >= 6;
            const float sc = dead ? __builtin_nanf("") : scale;
            const auto rs = loc ? rsrc(P.out, OUT_BYTES) : rsrc(ws, WS_NEED);
            const unsigned pitch = loc ? NTOT * 4 : 512;
            const unsigned base = loc ? (unsigned)(rt - 6) * 256 * (NTOT * 4) + col * 4 : OFF_OS + (cc * 6 + rt) * 131072;
            const unsigned lo = (wm * 64 + fr) * pitch + wn * 256 + fq * 16;
            if (ti == 3) {
#pragma unroll
                for (int mi = 0; mi < 4; ++mi)
#pragma unroll
                    for (int ni = 0; ni < 4; ++ni)
#pragma unroll
                        for (int j = 0; j < 4; ++j) acc[mi][ni][j] = __builtin_bit_cast(int, silu(acc[mi][ni][j], sc));
            }
#pragma unroll
            for (int mi = 0; mi < 4; ++mi)
#pragma unroll
                for (int ni = 0; ni < 4; ++ni)
                    st128<AUX_SYS>(__builtin_bit_cast(v4u, acc[mi][ni]), rs, lo + base + mi * 16 * pitch + ni * 64);
            WAIT_VM0();
            __syncthreads();
            if (ti < 2 && threadIdx.x == 0)
                __hip_atomic_fetch_add((gu32*)(ws + OFF_MS + MS_PART * 1024), 1u, __ATOMIC_RELAXED, __HIP_MEMORY_SCOPE_AGENT);
            if (ti == 2 && threadIdx.x < (rt < 5 ? 6 : 3)) {
                int d, slot;
                entry_inv(g, rt * 8 + (rt < 5 ? 0 : 2) + threadIdx.x, d, slot);
                st_sys(P.ws[dest_rank(rank, d) & 7] + OFF_RDY + ((rank * NCC + cc) * 8 + rt) * 4, call);
                if (dead) st_sys(P.out + col * 4, 0x7fc00000u);
            }
        }
        if (wid == 0 && ti >= 2) STAMP(ws, ti * 3 - 3);
        if (wid == 2 && ti < 2) STAMP(ws, 1);
    }
}

}

extern "C" void kernel_launch(int rank, int nranks, ncclComm_t comm,
                              hipStream_t stream,
                              void* const* d_in, const int* in_sizes, int n_in,
                              void* d_out, int out_size,
                              const int* in_dtypes, int out_dtype,
                              void* const* d_out_all,
                              void* const* d_ws_all, size_t ws_bytes) {
    if (nranks != NR || (unsigned)rank >= (unsigned)NR || ws_bytes < WS_NEED || n_in != 4 ||
        in_sizes[0] != (int)X_BYTES || in_sizes[1] != (int)W_BYTES || out_size != 512 * NTOT) {
        fprintf(stderr, "unexpected problem shape\n");
        return;
    }
    Params p;
    p.x = (const char*)d_in[0]; p.w = (const char*)d_in[1];
    p.sx = (const float*)d_in[2]; p.sw = (const float*)d_in[3];
    p.out = (char*)d_out;
    for (int i = 0; i < NR; ++i) { p.ws[i] = (char*)d_ws_all[phys(i)]; if (phys(i) == rank) p.rank = i; }
    hipLaunchKernelGGL(fused_gemm_scatter, dim3(NWG), dim3(512), 0, stream, p);
    HIP_CHECK(hipGetLastError());
}
```

```cpp
#include <hip/hip_runtime.h>
#include <rccl/rccl.h>
#include <cstdio>

#define HIP_CHECK(expr) do { hipError_t _e = (expr); if (_e != hipSuccess) { \
    fprintf(stderr, "HIP error %s at %s:%d\n", hipGetErrorString(_e), __FILE__, __LINE__); \
    return; } } while (0)

#define PULL_U 2
#define PULL_NL 56
#define PUSH_NW 1
#define PUSH_NP 4
#define PUSH_WIN 6
#define GEMM_SLEEP 64
#define RELAY 1
#define GATE 0
#define HINT 8
#define GRID 1
#define TAU 864
#define LATE 3
#define PACE 2100
#define PERM 0x75416320u
#define EXP_FIRST 1
#define EXP_DELAY 100
#define PS_WS 0
#define PART_T0 600u
#define PART_DT 300u
#define ACQ_SCOPE ""

namespace {

typedef int v2i __attribute__((ext_vector_type(2)));
typedef int v4i __attribute__((ext_vector_type(4)));
typedef unsigned v4u __attribute__((ext_vector_type(4)));
typedef float v4f __attribute__((ext_vector_type(4)));
typedef unsigned long long u64;
#define LDS_AS __attribute__((address_space(3)))
#define GLB_AS __attribute__((address_space(1)))
typedef GLB_AS unsigned gu32;

constexpr int NR = 8;
constexpr int NTOT = 16384;
constexpr int SAME_BLK = 7, CROSS_BLK = 9;
constexpr int SLOT_A = 28;
constexpr int SLOT_B2 = 42, SLOT_B3 = 49, SLOT_OWN = 57;
constexpr int NSLOT = SLOT_OWN + 16;
constexpr int NCC = 32;
constexpr int CC_A = 8;
constexpr int NDMA = 64;
constexpr int NWG = 256;
constexpr int NL = PULL_NL;
constexpr int MS_XA = 32, MS_XB1 = 33, MS_XB2 = 34, MS_XB3 = 35, MS_XO = 36, MS_PART = 37, MS_HINT = 38;
constexpr int NPS = 8 * PUSH_NW * PUSH_NP / 2;
constexpr int NXA = 24;
constexpr int B_WA = NXA - NPS, B_XB = B_WA + 4 * CC_A;
constexpr int CC_R = 28, NEED_RELAY = 7, NEED_EX = 256;
static_assert(NPS <= NXA, "pushed slots");

constexpr unsigned OFF_CNT = 0;
constexpr unsigned OFF_TMO = 8192;
constexpr unsigned OFF_Q = 12288;
constexpr unsigned OFF_RDY = 16384;
constexpr unsigned OFF_EXF = 32768;
constexpr unsigned OFF_DBG = 65536;
constexpr unsigned OFF_MS = 262144;
constexpr unsigned XS_SRC = NSLOT * 16384;
constexpr unsigned XS_PAR = NR * XS_SRC;
constexpr unsigned WS_SRC = NCC * 65536;
constexpr unsigned WS_PAR = NR * WS_SRC;
constexpr unsigned EX_PAR = 160 * 65536;
constexpr unsigned OFF_XS = 1u << 20;
constexpr unsigned OFF_WS = OFF_XS + 2 * XS_PAR;
constexpr unsigned OFF_EX = OFF_WS + 2 * WS_PAR;
constexpr unsigned OFF_OS = OFF_EX + 2 * EX_PAR;
constexpr unsigned OFF_PS = OFF_OS + NCC * 6 * 131072;
constexpr unsigned OFF_ZERO = OFF_PS + 128 * 262144;
constexpr unsigned WS_NEED = OFF_ZERO + 4096;
constexpr unsigned OUT_BYTES = 512u * NTOT * 4u;
constexpr unsigned X_BYTES = 4096u * 512u, W_BYTES = 512u * NTOT;

constexpr u64 DEADLINE = 50000000ull;
constexpr int STAGE = 49152;
constexpr int AUX_SYS = 17;
constexpr int AUX_DEV = 16;

struct Params {
    const char* x; const char* w; const float* sx; const float* sw;
    char* out; char* ws[NR];
    char* wsr[NR];
    int rank;
};

__device__ __forceinline__ int rfl(int v) { return __builtin_amdgcn_readfirstlane(v); }
__device__ __forceinline__ u64 ballot(bool b) { return __builtin_amdgcn_ballot_w64(b); }
__device__ __forceinline__ auto rsrc(const void* p, unsigned bytes) {
    return __builtin_amdgcn_make_buffer_rsrc((void*)p, 0, bytes, 0x00020000);
}
template <int AUX, class R> __device__ __forceinline__ void st128(v4u v, R rs, unsigned off) {
    __builtin_amdgcn_raw_buffer_store_b128(v, rs, off, 0, AUX);
}
__host__ __device__ __forceinline__ int phys(int l) { return (PERM >> (4 * l)) & 7; }
__device__ __forceinline__ int dest_rank(int c, int d) { return d < 3 ? (c & 4) | ((c + 1 + d) & 3) : ((c & 4) ^ 4) | (d - 3); }
__device__ __forceinline__ int slot_row(int c, int s) {
    if (s >= SLOT_OWN) return phys(c) * 512 + (s - SLOT_OWN) * 32;
    int j, d;
    if (s < SLOT_B3) { j = s / 7; d = s - j * 7; } else { j = 7 + ((s - SLOT_B3) >> 2); d = 3 + ((s - SLOT_B3) & 3); }
    return phys(dest_rank(c, d)) * 512 + (d < 3 ? 0 : SAME_BLK * 32) + j * 32;
}
__device__ __forceinline__ int xslot(int n, int skip) {
    if (n < 42) { int j = n / 6, i = n - j * 6; return j * 7 + (i < skip ? i : i + 1); }
    n -= 42;
    if (skip < 3) return SLOT_B3 + n;
    int j = n / 3, i = n - j * 3;
    return SLOT_B3 + j * 4 + (i < skip - 3 ? i : i + 1);
}
__device__ __forceinline__ int entry_of(int g, int d, int j) {
    int i = d < 3 + g ? d : d - 1;
    return j < 7 ? j * 6 + i : 42 + (j - 7) * 3 + (i - 3);
}
__device__ __forceinline__ void entry_inv(int g, int e, int& d, int& slot) {
    int j, i;
    if (e < 42) { j = e / 6; i = e - j * 6; } else { j = 7 + (e - 42) / 3; i = 3 + (e - 42) % 3; }
    d = i < 3 + g ? i : i + 1;
    slot = j < 7 ? j * 7 + d : SLOT_B3 + (j - 7) * 4 + (d - 3);
}
__device__ __forceinline__ int chunk_col(int c, int cc) {
    int g = cc & 3, gid = (c & 4) ? g * 4 + (c & 3) : (c & 3) * 4 + g;
    return gid * 1024 + (cc >> 2) * 128;
}
__device__ __forceinline__ unsigned ld_sys(const char* p) {
    return __hip_atomic_load((gu32*)p, __ATOMIC_RELAXED, __HIP_MEMORY_SCOPE_SYSTEM);
}
__device__ __forceinline__ void st_sys(char* p, unsigned v) {
    __hip_atomic_store((gu32*)p, v, __ATOMIC_RELAXED, __HIP_MEMORY_SCOPE_SYSTEM);
}
__device__ __forceinline__ void set_tmo(char* ws, unsigned call) {
    __hip_atomic_store((gu32*)(ws + OFF_TMO), call, __ATOMIC_RELAXED, __HIP_MEMORY_SCOPE_AGENT);
}
__device__ __forceinline__ bool expired(char* ws, unsigned call, u64 t0, unsigned& round) {
    bool out = __builtin_amdgcn_s_memrealtime() - t0 > DEADLINE;
    if ((round++ & 7u) == 0u) out = out || rfl((int)ld_sys(ws + OFF_TMO)) == (int)call;
    if (out) set_tmo(ws, call);
    return out;
}
template <int SLEEP> __device__ __forceinline__ bool wait_words(char* ws, unsigned call, int lane, u64 mask, const char* p, unsigned target) {
    const u64 t0 = __builtin_amdgcn_s_memrealtime();
    if (!((mask >> lane) & 1)) p = ws + OFF_TMO;
    for (;;) {
        const unsigned v = ld_sys(p);
        if (ballot(v == call) >> 63) return false;
        if ((ballot((int)(v - target) < 0) & mask) == 0) return true;
        if (__builtin_amdgcn_s_memrealtime() - t0 > DEADLINE) { set_tmo(ws, call); return false; }
        __builtin_amdgcn_s_sleep(SLEEP);
    }
}
#define WAIT_VM0() asm volatile("s_waitcnt vmcnt(0)" ::: "memory")
#define STR2(x) #x
#define STR(x) STR2(x)

#define STAMPV(ws, idx, v) do {} while (0)
#define STAMP(ws, idx) STAMPV(ws, idx, __builtin_amdgcn_s_memrealtime())

__device__ __forceinline__ float silu(int acc, float s) {
    float y = (float)acc * s;
    float c = fminf(fmaxf(y, -60.0f), 60.0f);
    float e = __builtin_amdgcn_exp2f(c * -1.4426950408889634f);
    return y * __builtin_amdgcn_rcpf(1.0f + e);
}
__device__ __forceinline__ unsigned ticket(char* p, int lane) {
    unsigned v = 0, z = 0;
    asm volatile("" : "+v"(z));
    if (lane == 0) v = __hip_atomic_fetch_add((gu32*)(p + z), 1u, __ATOMIC_RELAXED, __HIP_MEMORY_SCOPE_AGENT);
    return v;
}

__global__ void __launch_bounds__(512, 2) fused_gemm_scatter(Params P) {
    __shared__ __attribute__((aligned(16))) char smem[3 * STAGE + 16];
    const int rank = P.rank;
    const int lane = threadIdx.x & 63, wid = rfl(threadIdx.x >> 6);
    char* const ws = P.wsr[0];
    char* const pws = P.wsr[(blockIdx.x >> 3) & 7];
    const char* const px = P.x;
    const char* const pw = P.w;
    asm volatile("" :: "s"(rank), "s"(ws), "s"(pws), "s"(px), "s"(pw));
    const int xcd = blockIdx.x & 7;
    const unsigned lin = lane * 16;
    STAMP(ws, 0);
    if (blockIdx.x < 8 && (!RELAY || wid != 0 || blockIdx.x >= 4 || (int)blockIdx.x == (rank & 3))) return;
    const unsigned callv = ticket(ws + OFF_CNT + (blockIdx.x * 8 + wid) * 4, lane);
    if (blockIdx.x < 8) {
        const unsigned call = (unsigned)rfl((int)callv) + 1u;
        wait_words<32>(ws, call, lane, 1, ws + OFF_MS + ((CC_R + blockIdx.x) * 8 + (rank ^ 4)) * 128, call * 64u);
        if (lane == 0) st_sys(P.ws[((rank & 4) ^ 4) | blockIdx.x] + OFF_RDY + (rank * NCC * 8 + NEED_RELAY) * 4, call);
        return;
    }

    if (blockIdx.x < NDMA) {
        const int ps = blockIdx.x >> 3, c = (rank + ps) & 7;
        const auto prs = rsrc(pws, WS_NEED);
        const bool same = ((c ^ rank) & 4) == 0;
        const int d = same ? ((rank - c) & 3) - 1 : 3 + (rank & 3);
        if (wid < PUSH_NW) {
            const auto xrs = rsrc(px, X_BYTES);
            const int h0 = (xcd * PUSH_NW + wid) * PUSH_NP;
            v4u v[PUSH_NP * 8];
            unsigned dof[PUSH_NP];
#pragma unroll
            for (int i = 0; i < PUSH_NP; ++i) {
                const int sl = xslot((h0 + i) >> 1, d), half = (h0 + i) & 1;
                const unsigned so = (slot_row(c, sl) + half * 16) * 512;
                dof[i] = OFF_XS + rank * XS_SRC + sl * 16384 + half * 8192;
#pragma unroll
                for (int u = 0; u < 8; ++u) v[i * 8 + u] = __builtin_amdgcn_raw_buffer_load_b128(xrs, lin, so + u * 1024, 0);
            }
            const unsigned call = (unsigned)rfl((int)callv) + 1u;
            const unsigned po = (call & 1u) * XS_PAR + lin;
#pragma unroll
            for (int u = 0; u < PUSH_NP * 8; ++u) {
                if (u >= PUSH_WIN) asm volatile("s_waitcnt vmcnt(%0)" :: "n"(PUSH_WIN - 1) : "memory");
                if (GATE && u == PUSH_NP * 8 - HINT && lane == 0)
                    __hip_atomic_fetch_add((gu32*)(pws + OFF_MS + (MS_HINT * 8 + rank) * 128), 1u, __ATOMIC_RELAXED, __HIP_MEMORY_SCOPE_SYSTEM);
                st128<AUX_SYS>(v[u], prs, po + dof[u >> 3] + (u & 7) * 1024);
            }
            WAIT_VM0();
            if (lane == 0) __hip_atomic_fetch_add((gu32*)(pws + OFF_MS + rank * 128 + MS_XA * 1024), 8u * PUSH_NP, __ATOMIC_RELAXED, __HIP_MEMORY_SCOPE_SYSTEM);
        }
        STAMP(ws, 1);
        if ((7 - wid) * 8 + xcd < NL) {
            const unsigned call = (unsigned)rfl((int)callv) + 1u;
            const unsigned par = call & 1u;
            const auto frs = rsrc(ws + OFF_RDY + c * 1024, 1024);
            const char* const exf = ws + OFF_EXF + c * 256 + lane * 4;
            const int nblk = same ? SAME_BLK : CROSS_BLK;
            const int dc = same ? ((c - rank) & 3) - 1 : 3 + (c & 3);
            const int B_WB = B_XB + SLOT_OWN - nblk - NXA;
            const auto ors = rsrc(P.out, OUT_BYTES);
            const float scale = P.sx[0] * P.sw[0];
            const int row0 = same ? 0 : SAME_BLK * 32;
            constexpr int PARTS = 16 / PULL_U;
            const int nca = same ? CC_A : CC_A / 4 * 3, ncb = same ? NCC - CC_A : (NCC - CC_A) / 4 * 3;
            const int nb1 = nca * 4, nb2 = nca * nblk;
            const bool skip = RELAY && same, relay = RELAY && !same && ((c ^ rank) & 3) != 0;
            const int B_XO = B_WB + 4 * (NCC - CC_A) - (skip ? 4 : 0), B_OUT = B_XO + 16;
            const unsigned nunits = (B_OUT + (nca + ncb) * nblk + (relay ? 4 : 0)) * PARTS;
            char* const q = ws + OFF_Q + ps * 128;
            u64 exm = 0;
            u64 r0 = 0, r1 = 0, r2 = 0, r3 = 0;
            const unsigned ol = (lane >> 5) * (NTOT * 4) + (lane & 31) * 16;
            u64 stall = 0, npoll = 0, niter = 0;
            bool dead = false;
            auto decode = [&](unsigned un, unsigned& so, unsigned& dof, int& ms, int& need) {
                const int part = un % PARTS;
                int t = un / PARTS;
                const bool rl = relay && t >= B_OUT + nb1 && t < B_OUT + nb1 + 4;
                if (t < B_OUT || rl) {
                    if (rl) {
                        const int i = t - B_OUT - nb1, src = c ^ 4;
                        ms = (CC_R + (c & 3)) * 8 + src;
                        need = NEED_RELAY;
                        so = OFF_WS + par * WS_PAR + src * WS_SRC + (CC_R + (rank & 3)) * 65536 + i * 16384;
                        dof = OFF_WS + par * WS_PAR + src * WS_SRC + (CC_R + (c & 3)) * 65536 + i * 16384;
                    } else if ((t >= B_WA && t < B_XB) || (t >= B_WB && t < B_XO)) {
                        int i = t < B_XB ? t - B_WA : t - B_WB + 4 * CC_A;
                        if (skip && i >= 4 * (CC_R + (c & 3))) i += 4;
                        const int cc = i >> 2;
                        ms = cc * 8 + c;
                        need = NEED_EX + cc;
                        so = OFF_EX + par * EX_PAR + chunk_col(rank, cc) * 512 + (i & 3) * 16384;
                        dof = OFF_WS + par * WS_PAR + c * WS_SRC + i * 16384;
                    } else {
                        int sl = t < B_WB ? xslot(t < B_WA ? t + NPS : t - B_XB + NXA, dc) : t - B_XO + SLOT_OWN;
                        ms = (sl < SLOT_A ? MS_XA : sl < SLOT_B2 ? MS_XB1 : sl < SLOT_B3 ? MS_XB2 : sl < SLOT_OWN ? MS_XB3 : MS_XO) * 8 + c;
                        const int row = slot_row(rank, sl);
                        need = NEED_EX + 32 + (row >> 7);
                        so = OFF_EX + par * EX_PAR + W_BYTES + row * 512;
                        dof = OFF_XS + par * XS_PAR + c * XS_SRC + sl * 16384;
                    }
                    so += part * (PULL_U * 1024);
                    dof += part * (PULL_U * 1024);
                    return;
                }
                t -= B_OUT;
                if (relay && t >= nb1) t -= 4;
                int ci, blk;
                if (t < nb1) { ci = t >> 2; blk = t & 3; }
                else if (t < nb2) { blk = t / nca; ci = t - blk * nca; }
                else { ci = t / nblk; blk = t - ci * nblk; }
                int cc = ci;
                if (!same) { int sub = ci / 3, gi = ci - sub * 3; cc = sub * 4 + (gi < (rank & 3) ? gi : gi + 1); }
                const int e = entry_of(cc & 3, d, blk);
                ms = -1;
                need = cc * 8 + (e >> 3);
                so = OFF_OS + (cc * 48 + e) * 16384 + part * (PULL_U * 1024);
                dof = (unsigned)(row0 + blk * 32 + part * (2 * PULL_U)) * (NTOT * 4) + chunk_col(c, cc) * 4;
            };
            auto put = [&](v4u (&v)[PULL_U], unsigned dof, int ms) {
                if (ms < 0) {
                    const float sc = dead ? __builtin_nanf("") : scale;
#pragma unroll
                    for (int u = 0; u < PULL_U; ++u) {
                        v4f o;
#pragma unroll
                        for (int j = 0; j < 4; ++j) o[j] = silu((int)v[u][j], sc);
                        st128<AUX_SYS>(__builtin_bit_cast(v4u, o), ors, dof + ol + u * 2 * (NTOT * 4));
                    }
                } else {
                    const auto rs = rsrc(ws, WS_NEED);
#pragma unroll
                    for (int u = 0; u < PULL_U; ++u) st128<AUX_SYS>(v[u], rs, dof + lin + u * 1024);
                }
            };
            auto signal = [&](int ms) {
                if (ms >= 0 && lane == 0)
                    __hip_atomic_fetch_add((gu32*)(ws + OFF_MS + ms * 128), (unsigned)PULL_U, __ATOMIC_RELAXED, __HIP_MEMORY_SCOPE_AGENT);
            };
            auto masks = [&](v4u f) {
                r0 = ballot((int)(f[0] - call) >= 0); r1 = ballot((int)(f[1] - call) >= 0);
                r2 = ballot((int)(f[2] - call) >= 0); r3 = ballot((int)(f[3] - call) >= 0);
            };
            auto have = [&](int k) {
                if (k >= NEED_EX) return rfl((int)((exm >> (k - NEED_EX)) & 1)) != 0;
                const int sh = k >> 2;
                const unsigned m = (unsigned)((r0 >> sh) & 1) | (unsigned)((r1 >> sh) & 1) << 1 | (unsigned)((r2 >> sh) & 1) << 2 | (unsigned)((r3 >> sh) & 1) << 3;
                return rfl((int)((m >> (k & 3)) & 1)) != 0;
            };
            auto await = [&](int k) {
                const u64 t0 = __builtin_amdgcn_s_memrealtime();
                unsigned round = 1;
                for (;;) {
                    asm volatile("" ::: "memory");
                    if (k >= NEED_EX) {
                        exm = ballot((int)(ld_sys(exf) - call) >= 0);
                    } else {
                        masks(__builtin_amdgcn_raw_buffer_load_b128(frs, lin, 0, AUX_SYS));
                    }
                    if (have(k)) break;
                    if (expired(ws, call, t0, round)) { dead = true; break; }
                    __builtin_amdgcn_s_sleep(1);
                }
                if (k < NEED_EX) { stall += __builtin_amdgcn_s_memrealtime() - t0; ++npoll; }
            };
            constexpr int NONE = -1000;
            unsigned un1 = (7 - wid) * 8 + xcd;
            unsigned tnext = ((unsigned)__builtin_amdgcn_s_memrealtime() << 8) / (NL * TAU) * (NL * TAU) + un1 * TAU;
            unsigned so, dof, pdof = 0;
            int ms, need, pms = NONE, ppms = NONE;
            v4u pv[PULL_U], v2[PULL_U];
            bool pok = false;
            const char* const pcnt = ws + OFF_MS + MS_PART * 1024;
            decode(un1, so, dof, ms, need);
            un1 += NL;
            if (GATE) dead = !wait_words<1>(ws, call, lane, 1, ws + OFF_MS + (MS_HINT * 8 + c) * 128, (call - 1u) * (8u * PUSH_NW) + GATE);
            for (int u = 0; u < PULL_U; ++u) pv[u] = v2[u] = v4u{0, 0, 0, 0};
            auto add2 = [&]() {
#pragma unroll
                for (int u = 0; u < PULL_U; ++u) { pv[u] += v2[u]; asm volatile("" : "+v"(pv[u])); }
            };
            auto flush = [&]() {
                add2();
                if (pms != NONE) put(pv, pdof, pms);
                signal(ppms);
                if (pms >= 0) { WAIT_VM0(); signal(pms); }
                pms = ppms = NONE;
            };
            for (;;) {
                if (!dead && (!have(need) || (ms < 0 && !pok))) {
                    flush();
                    if (!have(need)) await(need);
                    if (ms < 0 && !pok && !dead) { dead = !wait_words<1>(ws, call, lane, 1, pcnt, call * 192u); pok = true; }
                }
                {
                    constexpr unsigned PER = NL * TAU;
                    const int late = (int)(((unsigned)__builtin_amdgcn_s_memrealtime() << 8) - tnext);
                    if (late >= (int)(PER * LATE / 16)) tnext += ((unsigned)late + PER - PER * LATE / 16) / PER * PER;
                    while ((int)(((unsigned)__builtin_amdgcn_s_memrealtime() << 8) - tnext) < 0) __builtin_amdgcn_s_sleep(1);
                    tnext += PER;
                }
                asm volatile("" ::: "memory");
                v4u v[PULL_U], f;
                unsigned fe, fp;
#pragma unroll
                for (int u = 0; u < PULL_U; ++u) v[u] = __builtin_amdgcn_raw_buffer_load_b128(prs, lin, so + u * 1024, AUX_SYS);
                add2();
                if (pms != NONE) put(pv, pdof, pms);
                signal(ppms);
                const unsigned nxt = ticket(q, lane);
                ppms = pms; pms = ms; pdof = dof;
                const bool more = un1 < nunits;
                if (more) decode(un1, so, dof, ms, need);
                const bool miss = more && !have(need);
                const bool ref_e = miss && need >= NEED_EX, ref_r = miss && need < NEED_EX;
                if (ref_e) fe = ld_sys(exf);
                if (ref_r) f = __builtin_amdgcn_raw_buffer_load_b128(frs, lin, 0, AUX_SYS);
                const bool ref_p = more && ms < 0 && !pok;
                if (ref_p) fp = ld_sys(pcnt);
                asm volatile("" ::: "memory");
#pragma unroll
                for (int u = 0; u < PULL_U; ++u)
                    v2[u] = __builtin_amdgcn_raw_buffer_load_b128(ors, ol, pms < 0 ? pdof + u * 2 * (NTOT * 4) : OUT_BYTES, AUX_DEV);
                asm volatile("s_waitcnt vmcnt(%0)" :: "n"(PULL_U) : "memory");
#pragma unroll
                for (int u = 0; u < PULL_U; ++u) pv[u] = v[u];
                if (!more) break;
                if (ref_e) exm = ballot((int)(fe - call) >= 0);
                if (ref_r) masks(f);
                if (ref_p) pok = rfl((int)(fp - call * 192u)) >= 0;
                un1 = (unsigned)rfl((int)nxt) - (call - 1u) * nunits + 2 * NL;
            }
            flush();
            STAMPV(ws, 4, stall);
            STAMPV(ws, 5, npoll);
            STAMPV(ws, 7, niter | (u64)(__builtin_amdgcn_s_getreg(20 | (0 << 6) | (3 << 11)) & 15) << 32);
        }
        STAMP(ws, 2);
        return;
    }

    const int job = blockIdx.x - NDMA;
    const unsigned tstart = (unsigned)__builtin_amdgcn_s_memrealtime();
    unsigned call;
    if (job < 160) {
        const bool isw = job < 128;
        const int gid = job & 15, sub = job >> 4, piece = isw ? gid * 8 + sub : job;
        const auto rs = rsrc(isw ? pw : px, isw ? W_BYTES : X_BYTES);
        const unsigned so = isw ? (wid * 64 + (lane >> 3)) * NTOT + piece * 128 + (lane & 7) * 16 : (job - 128) * 65536 + wid * 8192 + lin;
        const unsigned st = isw ? 8 * NTOT : 1024;
        v4u v[8];
        const int to = isw ? (lane & 1 ? 4 + (gid & 3) : gid >> 2) : lane & 7;
        char* const flag = P.ws[to] + OFF_EXF + (rank * 64 + (isw ? sub * 4 + (lane & 1 ? gid >> 2 : gid & 3) : job - 96)) * 4;
        if (EXP_DELAY && isw && sub >= EXP_FIRST) __builtin_amdgcn_s_sleep(EXP_DELAY);
#pragma unroll
        for (int u = 0; u < 8; ++u) v[u] = __builtin_amdgcn_raw_buffer_load_b128(rs, so + u * st, 0, 0);
        call = (unsigned)rfl((int)callv) + 1u;
        const auto wrs = rsrc(ws, WS_NEED);
        const unsigned dof = OFF_EX + (call & 1u) * EX_PAR + piece * 65536 + wid * 8192 + lin;
#pragma unroll
        for (int u = 0; u < 8; ++u) st128<AUX_SYS>(v[u], wrs, dof + u * 1024);
        WAIT_VM0();
        __syncthreads();
        if (threadIdx.x < (isw ? 2 : 8) && to != rank) st_sys(flag, call);
    } else {
        call = (unsigned)rfl((int)callv) + 1u;
    }
    STAMP(ws, 7);
    const unsigned par = call & 1u;
    const int li = job >> 3;
    const float scale = P.sx[0] * P.sw[0];
    const int wm = wid >> 1, wn = wid & 1, fr = lane & 15, fq = lane >> 4;
    const unsigned lds0 = (unsigned)(size_t)(LDS_AS char*)smem;
    volatile LDS_AS int* const dead_w = (volatile LDS_AS int*)((LDS_AS char*)smem + 3 * STAGE);
    unsigned xa[2], wa[4];
    for (int kh = 0; kh < 2; ++kh) xa[kh] = (wm * 64 + fr) * 128 + (((kh * 4 + fq) ^ (fr >> 1)) << 4);
    for (int ni = 0; ni < 4; ++ni)
        wa[ni] = 32768 + (fq * 16 + (fr >> 1)) * 128 + (((wn * 4 + ni) ^ (((fr >> 2) & 3) | ((fq & 1) << 2))) << 4) + 8 * (fr & 1);

    const int tend = (xcd < 4 && li < 16) ? 4 : 3;
    for (int ti = xcd < 4 ? 2 : 0; ti < tend; ++ti) {
        int cc = 0, rt, col;
        if (ti < 2) {
            const int sub = ti * 4 + xcd - 4, i12 = li >> 1, me = rank & 3;
            const int gid = (rank & 4) ? (i12 / 3) * 4 + (i12 % 3 < me ? i12 % 3 : i12 % 3 + 1) : (i12 < 4 * me ? i12 : i12 + 4);
            rt = 6 + (li & 1);
            col = gid * 1024 + sub * 128;
            while ((unsigned)__builtin_amdgcn_s_memrealtime() - tstart < PART_T0 + sub * PART_DT) __builtin_amdgcn_s_sleep(8);
        } else {
            if (ti == 3) { int idx = xcd * 16 + li; cc = idx >> 1; rt = 6 + (idx & 1); }
            else if (xcd < 2) { cc = li / 3; rt = xcd * 3 + li % 3; }
            else { cc = xcd * 4 + li / 6; rt = li % 6; }
            col = chunk_col(rank, cc);
        }
        cc = rfl(cc); rt = rfl(rt); col = rfl(col);
        const int g = cc & 3;
        bool dead = false;

        v4i acc[4][4];
#pragma unroll
        for (int i = 0; i < 4; ++i)
#pragma unroll
            for (int j = 0; j < 4; ++j) acc[i][j] = v4i{0, 0, 0, 0};

#pragma unroll 1
        for (int seg = 0; seg < (ti < 2 ? 1 : 2); ++seg) {
            if (seg == 1) {
                if (wid == 0) {
                    const int m1 = rt < 3 ? MS_XA : rt < 5 ? MS_XB1 : rt < 6 ? MS_XB3 : MS_XO;
                    const unsigned n1 = rt < 3 ? NXA : rt < 5 ? 12 : rt < 6 ? (((lane ^ rank) & 4) ? 6 : 8) : 16;
                    const u64 peers = 0xffull & ~(1ull << rank);
                    const char* mb = ws + OFF_MS + (lane & 7) * 128;
                    bool ok = wait_words<GEMM_SLEEP>(ws, call, lane, peers, mb + cc * 1024, call * 64u);
                    if (ok) ok = wait_words<GEMM_SLEEP>(ws, call, lane, (rt == 4 || rt == 5) ? peers | peers << 8 : peers,
                                               mb + (lane & 8 ? MS_XB2 : m1) * 1024, call * 16u * (lane & 8 ? 6 : n1));
                    __builtin_amdgcn_fence(__ATOMIC_ACQUIRE, ACQ_SCOPE);
                    *dead_w = ok ? 0 : 1;
                    WAIT_VM0();
                }
                __syncthreads();
                dead = rfl(*dead_w) != 0;
                if (wid == 0) STAMP(ws, ti * 3 - 5);
                if (dead) break;
            }
            unsigned xo[4], wo[2];
            int own[4];
#pragma unroll
            for (int i = 0; i < 4; ++i) {
                int row = i * 64 + wid * 8 + (lane >> 3), e = rt * 8 + i * 2 + (wid >> 2), d, slot;
                if (rt >= 6) { slot = SLOT_OWN + e - 48; own[i] = -1; } else { entry_inv(g, e, d, slot); own[i] = seg ? dest_rank(rank, d) : -1; }
                xo[i] = (seg ? slot * 32 : slot_row(rank, slot)) * 512 + (row & 31) * 512 + (((lane & 7) ^ ((row >> 1) & 7)) << 4);
            }
#pragma unroll
            for (int i = 0; i < 2; ++i) {
                int k = i * 64 + wid * 8 + (lane >> 3);
                wo[i] = k * (seg ? 128 : NTOT) + (((lane & 7) ^ (((k >> 1) & 3) | (((k >> 4) & 1) << 2))) << 4);
            }
            const char* xs = seg ? ws + OFF_XS + par * XS_PAR : px;
            const char* wsb = seg ? ws + OFF_WS + par * WS_PAR + cc * 65536 : pw + col;
            const unsigned wstep = seg ? 16384 : 128 * NTOT;
            const int nt = seg ? 28 : 4;
            auto stage = [&](int t, int buf) {
                int p = t >> 2;
                p = seg ? p + (p >= rank) : 0;
                const char* xb = xs + p * XS_SRC + (t & 3) * 128;
                const char* wb = wsb + p * WS_SRC + (t & 3) * wstep;
                const unsigned l = lds0 + buf * STAGE + wid * 1024;
                const unsigned z = OFF_ZERO - (OFF_XS + par * XS_PAR + p * XS_SRC + (t & 3) * 128) + lin;
                const unsigned x0 = own[0] == p ? z : xo[0], x1 = own[1] == p ? z : xo[1], x2 = own[2] == p ? z : xo[2], x3 = own[3] == p ? z : xo[3];
                unsigned keep;
                asm volatile(
                    "s_mov_b32 %0, m0\n\t"
                    "s_mov_b32 m0, %1\n\ts_nop 0\n\tglobal_load_lds_dwordx4 %4, %2\n\t"
                    "s_add_u32 m0, m0, 0x2000\n\ts_nop 0\n\tglobal_load_lds_dwordx4 %5, %2\n\t"
                    "s_add_u32 m0, m0, 0x2000\n\ts_nop 0\n\tglobal_load_lds_dwordx4 %6, %2\n\t"
                    "s_add_u32 m0, m0, 0x2000\n\ts_nop 0\n\tglobal_load_lds_dwordx4 %7, %2\n\t"
                    "s_add_u32 m0, m0, 0x2000\n\ts_nop 0\n\tglobal_load_lds_dwordx4 %8, %3\n\t"
                    "s_add_u32 m0, m0, 0x2000\n\ts_nop 0\n\tglobal_load_lds_dwordx4 %9, %3\n\t"
                    "s_mov_b32 m0, %0"
                    : "=&s"(keep)
                    : "s"(l), "s"(xb), "s"(wb), "v"(x0), "v"(x1), "v"(x2), "v"(x3), "v"(wo[0]), "v"(wo[1])
                    : "memory", "scc");
            };
            v4i xf[2][4], wf[2][4];
            auto frags = [&](int buf, int kh) {
                LDS_AS char* sb = (LDS_AS char*)smem + buf * STAGE;
#pragma unroll
                for (int ni = 0; ni < 4; ++ni) {
                    v2i lo = __builtin_amdgcn_ds_read_tr8_b64_v2i32((LDS_AS v2i*)(sb + wa[ni] + kh * 8192));
                    v2i hi = __builtin_amdgcn_ds_read_tr8_b64_v2i32((LDS_AS v2i*)(sb + wa[ni] + kh * 8192 + 1024));
                    wf[kh][ni] = v4i{lo[0], lo[1], hi[0], hi[1]};
                }
#pragma unroll
                for (int mi = 0; mi < 4; ++mi) xf[kh][mi] = *(LDS_AS v4i*)(sb + xa[kh] + mi * 2048);
            };
            auto mma = [&](int kh) {
#pragma unroll
                for (int mi = 0; mi < 4; ++mi)
#pragma unroll
                    for (int ni = 0; ni < 4; ++ni)
                        acc[mi][ni] = __builtin_amdgcn_mfma_i32_16x16x64_i8(wf[kh][ni], xf[kh][mi], acc[mi][ni], 0, 0, 0);
            };
            WAIT_VM0();
            stage(0, 0);
            stage(1, 1);
            stage(2, 2);
            asm volatile("s_waitcnt vmcnt(12)" ::: "memory");
            __builtin_amdgcn_s_barrier();
            asm volatile("" ::: "memory");
            frags(0, 0);
            int buf = 0;
            for (int t = 0; t < nt; ++t) {
                frags(buf, 1);
                __builtin_amdgcn_sched_barrier(0);
                mma(0);
                __builtin_amdgcn_sched_barrier(0);
                if (t < nt - 2) asm volatile("s_waitcnt vmcnt(6) lgkmcnt(0)" ::: "memory");
                else asm volatile("s_waitcnt vmcnt(0) lgkmcnt(0)" ::: "memory");
                __builtin_amdgcn_s_barrier();
                asm volatile("" ::: "memory");
                if (t < nt - 3) stage(t + 3, buf);
                buf = buf == 2 ? 0 : buf + 1;
                if (t < nt - 1) frags(buf, 0);
                __builtin_amdgcn_sched_barrier(0);
                mma(1);
                __builtin_amdgcn_sched_barrier(0);
            }
        }
        if (wid == 0 && ti >= 2) STAMP(ws, ti * 3 - 4);
        {
            const bool loc = PS_WS ? ti == 3 : rt >= 6;
            const float sc = dead ? __builtin_nanf("") : scale;
            const auto rs = loc ? rsrc(P.out, OUT_BYTES) : rsrc(ws, WS_NEED);
            const unsigned pitch = loc ? NTOT * 4 : 512;
            const unsigned base = loc ? (unsigned)(rt - 6) * 256 * (NTOT * 4) + col * 4 : ti < 2 ? OFF_PS + col * 2048 + (rt - 6) * 131072 : OFF_OS + (cc * 6 + rt) * 131072;
            const unsigned lo = (wm * 64 + fr) * pitch + wn * 256 + fq * 16;
            if (ti == 3) {
#pragma unroll
                for (int mi = 0; mi < 4; ++mi)
#pragma unroll
                    for (int ni = 0; ni < 4; ++ni)
#pragma unroll
                        for (int j = 0; j < 4; ++j) acc[mi][ni][j] = __builtin_bit_cast(int, silu(acc[mi][ni][j], sc));
            }
#pragma unroll
            for (int mi = 0; mi < 4; ++mi)
#pragma unroll
                for (int ni = 0; ni < 4; ++ni)
                    st128<AUX_SYS>(__builtin_bit_cast(v4u, acc[mi][ni]), rs, lo + base + mi * 16 * pitch + ni * 64);
            WAIT_VM0();
            __syncthreads();
            if (ti < 2 && threadIdx.x == 0)
                __hip_atomic_fetch_add((gu32*)(ws + OFF_MS + MS_PART * 1024), 1u, __ATOMIC_RELAXED, __HIP_MEMORY_SCOPE_AGENT);
            if (ti == 2 && threadIdx.x < (rt < 5 ? 6 : 3)) {
                int d, slot;
                entry_inv(g, rt * 8 + (rt < 5 ? 0 : 2) + threadIdx.x, d, slot);
                st_sys(P.ws[dest_rank(rank, d) & 7] + OFF_RDY + ((rank * NCC + cc) * 8 + rt) * 4, call);
                if (dead) st_sys(P.out + col * 4, 0x7fc00000u);
            }
        }
        if (wid == 0 && ti >= 2) STAMP(ws, ti * 3 - 3);
        if (wid == 2 && ti < 2) STAMP(ws, 1);
    }
}

}

extern "C" void kernel_launch(int rank, int nranks, ncclComm_t comm,
                              hipStream_t stream,
                              void* const* d_in, const int* in_sizes, int n_in,
                              void* d_out, int out_size,
                              const int* in_dtypes, int out_dtype,
                              void* const* d_out_all,
                              void* const* d_ws_all, size_t ws_bytes) {
    if (nranks != NR || (unsigned)rank >= (unsigned)NR || ws_bytes < WS_NEED || n_in != 4 ||
        in_sizes[0] != (int)X_BYTES || in_sizes[1] != (int)W_BYTES || out_size != 512 * NTOT) {
        fprintf(stderr, "unexpected problem shape\n");
        return;
    }
    Params p;
    p.x = (const char*)d_in[0]; p.w = (const char*)d_in[1];
    p.sx = (const float*)d_in[2]; p.sw = (const float*)d_in[3];
    p.out = (char*)d_out;
    for (int i = 0; i < NR; ++i) { p.ws[i] = (char*)d_ws_all[phys(i)]; if (phys(i) == rank) p.rank = i; }
    for (int i = 0; i < NR; ++i) p.wsr[i] = p.ws[(p.rank + i) & 7];
    hipLaunchKernelGGL(fused_gemm_scatter, dim3(NWG), dim3(512), 0, stream, p);
    HIP_CHECK(hipGetLastError());
}
```

```cpp
#include <hip/hip_runtime.h>
#include <rccl/rccl.h>
#include <cstdio>

#define HIP_CHECK(expr) do { hipError_t _e = (expr); if (_e != hipSuccess) { \
    fprintf(stderr, "HIP error %s at %s:%d\n", hipGetErrorString(_e), __FILE__, __LINE__); \
    return; } } while (0)

#define PULL_NL 56
#define TAU 864
#define LATE 3
#define PUSH_NP 4
#define PUSH_WIN 6
#define PART_T0 600u
#define PART_DT 300u

namespace {

typedef int v2i __attribute__((ext_vector_type(2)));
typedef int v4i __attribute__((ext_vector_type(4)));
typedef unsigned v4u __attribute__((ext_vector_type(4)));
typedef float v4f __attribute__((ext_vector_type(4)));
typedef unsigned long long u64;
#define LDS_AS __attribute__((address_space(3)))
#define GLB_AS __attribute__((address_space(1)))
typedef GLB_AS unsigned gu32;

constexpr int NR = 8;
constexpr int NTOT = 16384;
constexpr int SAME_BLK = 7, CROSS_BLK = 9;
constexpr int SLOT_A = 28;
constexpr int SLOT_B2 = 42, SLOT_B3 = 49, SLOT_OWN = 57;
constexpr int NSLOT = SLOT_OWN + 16;
constexpr int NCC = 32;
constexpr int CC_A = 8;
constexpr int NDMA = 64;
constexpr int NWG = 256;
constexpr int NL = PULL_NL, PULL_U = 2;
constexpr unsigned PERM = 0x75416320u;
constexpr int MS_XA = 32, MS_XB1 = 33, MS_XB2 = 34, MS_XB3 = 35, MS_XO = 36, MS_PART = 37;
constexpr int NPS = 4 * PUSH_NP;
constexpr int NXA = 24;
constexpr int B_WA = NXA - NPS, B_XB = B_WA + 4 * CC_A;
constexpr int CC_R = 28, NEED_RELAY = 7, NEED_EX = 256;
static_assert(NPS <= NXA, "pushed slots");

constexpr unsigned OFF_CNT = 0;
constexpr unsigned OFF_TMO = 8192;
constexpr unsigned OFF_Q = 12288;
constexpr unsigned OFF_RDY = 16384;
constexpr unsigned OFF_EXF = 32768;
constexpr unsigned OFF_MS = 262144;
constexpr unsigned XS_SRC = NSLOT * 16384;
constexpr unsigned XS_PAR = NR * XS_SRC;
constexpr unsigned WS_SRC = NCC * 65536;
constexpr unsigned WS_PAR = NR * WS_SRC;
constexpr unsigned EX_PAR = 160 * 65536;
constexpr unsigned OFF_XS = 1u << 20;
constexpr unsigned OFF_WS = OFF_XS + 2 * XS_PAR;
constexpr unsigned OFF_EX = OFF_WS + 2 * WS_PAR;
constexpr unsigned OFF_OS = OFF_EX + 2 * EX_PAR;
constexpr unsigned WS_NEED = OFF_OS + NCC * 6 * 131072;
constexpr unsigned OUT_BYTES = 512u * NTOT * 4u;
constexpr unsigned X_BYTES = 4096u * 512u, W_BYTES = 512u * NTOT;

constexpr u64 DEADLINE = 100000000ull;
constexpr int STAGE = 49152;
constexpr int AUX_SYS = 17;

struct Params {
    const char* x; const char* w; const float* sx; const float* sw;
    char* out; char* ws[NR];
    char* wsr[NR];
    int rank;
};

__device__ __forceinline__ int rfl(int v) { return __builtin_amdgcn_readfirstlane(v); }
__device__ __forceinline__ u64 ballot(bool b) { return __builtin_amdgcn_ballot_w64(b); }
__device__ __forceinline__ auto rsrc(const void* p, unsigned bytes) {
    return __builtin_amdgcn_make_buffer_rsrc((void*)p, 0, bytes, 0x00020000);
}
template <int AUX, class R> __device__ __forceinline__ void st128(v4u v, R rs, unsigned off) {
    __builtin_amdgcn_raw_buffer_store_b128(v, rs, off, 0, AUX);
}
__host__ __device__ __forceinline__ int phys(int l) { return (PERM >> (4 * l)) & 7; }
__device__ __forceinline__ int dest_rank(int c, int d) { return d < 3 ? (c & 4) | ((c + 1 + d) & 3) : ((c & 4) ^ 4) | (d - 3); }
__device__ __forceinline__ int slot_row(int c, int s) {
    if (s >= SLOT_OWN) return phys(c) * 512 + (s - SLOT_OWN) * 32;
    int j, d;
    if (s < SLOT_B3) { j = s / 7; d = s - j * 7; } else { j = 7 + ((s - SLOT_B3) >> 2); d = 3 + ((s - SLOT_B3) & 3); }
    return phys(dest_rank(c, d)) * 512 + (d < 3 ? 0 : SAME_BLK * 32) + j * 32;
}
__device__ __forceinline__ int xslot(int n, int skip) {
    if (n < 42) { int j = n / 6, i = n - j * 6; return j * 7 + (i < skip ? i : i + 1); }
    n -= 42;
    if (skip < 3) return SLOT_B3 + n;
    int j = n / 3, i = n - j * 3;
    return SLOT_B3 + j * 4 + (i < skip - 3 ? i : i + 1);
}
__device__ __forceinline__ int entry_of(int g, int d, int j) {
    int i = d < 3 + g ? d : d - 1;
    return j < 7 ? j * 6 + i : 42 + (j - 7) * 3 + (i - 3);
}
__device__ __forceinline__ void entry_inv(int g, int e, int& d, int& slot) {
    int j, i;
    if (e < 42) { j = e / 6; i = e - j * 6; } else { j = 7 + (e - 42) / 3; i = 3 + (e - 42) % 3; }
    d = i < 3 + g ? i : i + 1;
    slot = j < 7 ? j * 7 + d : SLOT_B3 + (j - 7) * 4 + (d - 3);
}
__device__ __forceinline__ int chunk_col(int c, int cc) {
    int g = cc & 3, gid = (c & 4) ? g * 4 + (c & 3) : (c & 3) * 4 + g;
    return gid * 1024 + (cc >> 2) * 128;
}
__device__ __forceinline__ unsigned ld_sys(const char* p) {
    return __hip_atomic_load((gu32*)p, __ATOMIC_RELAXED, __HIP_MEMORY_SCOPE_SYSTEM);
}
__device__ __forceinline__ void st_sys(char* p, unsigned v) {
    __hip_atomic_store((gu32*)p, v, __ATOMIC_RELAXED, __HIP_MEMORY_SCOPE_SYSTEM);
}
__device__ __forceinline__ void set_tmo(char* ws, unsigned call) {
    __hip_atomic_store((gu32*)(ws + OFF_TMO), call, __ATOMIC_RELAXED, __HIP_MEMORY_SCOPE_AGENT);
}
__device__ __forceinline__ bool expired(char* ws, unsigned call, u64 t0, unsigned& round) {
    bool out = __builtin_amdgcn_s_memrealtime() - t0 > DEADLINE;
    if ((round++ & 7u) == 0u) out = out || rfl((int)ld_sys(ws + OFF_TMO)) == (int)call;
    if (out) set_tmo(ws, call);
    return out;
}
template <int SLEEP> __device__ __forceinline__ bool wait_words(char* ws, unsigned call, int lane, u64 mask, const char* p, unsigned target) {
    const u64 t0 = __builtin_amdgcn_s_memrealtime();
    if (!((mask >> lane) & 1)) p = ws + OFF_TMO;
    for (;;) {
        const unsigned v = ld_sys(p);
        if (ballot(v == call) >> 63) return false;
        if ((ballot((int)(v - target) < 0) & mask) == 0) return true;
        if (__builtin_amdgcn_s_memrealtime() - t0 > DEADLINE) { set_tmo(ws, call); return false; }
        __builtin_amdgcn_s_sleep(SLEEP);
    }
}
#define WAIT_VM0() asm volatile("s_waitcnt vmcnt(0)" ::: "memory")

__device__ __forceinline__ float silu(int acc, float s) {
    float y = (float)acc * s;
    float c = fminf(fmaxf(y, -60.0f), 60.0f);
    float e = __builtin_amdgcn_exp2f(c * -1.4426950408889634f);
    return y * __builtin_amdgcn_rcpf(1.0f + e);
}
__device__ __forceinline__ unsigned ticket(char* p, int lane) {
    unsigned v = 0, z = 0;
    asm volatile("" : "+v"(z));
    if (lane == 0) v = __hip_atomic_fetch_add((gu32*)(p + z), 1u, __ATOMIC_RELAXED, __HIP_MEMORY_SCOPE_AGENT);
    return v;
}

__global__ void __launch_bounds__(512, 2) fused_gemm_scatter(Params P) {
    __shared__ __attribute__((aligned(16))) char smem[3 * STAGE + 16];
    const int rank = P.rank;
    const int lane = threadIdx.x & 63, wid = rfl(threadIdx.x >> 6);
    char* const ws = P.wsr[0];
    char* const pws = P.wsr[(blockIdx.x >> 3) & 7];
    const char* const px = P.x;
    const char* const pw = P.w;
    asm volatile("" :: "s"(rank), "s"(ws), "s"(pws), "s"(px), "s"(pw));
    const int xcd = blockIdx.x & 7;
    const unsigned tstart = (unsigned)__builtin_amdgcn_s_memrealtime();
    const unsigned lin = lane * 16;
    if (blockIdx.x < 8 && (wid != 0 || blockIdx.x >= 4 || (int)blockIdx.x == (rank & 3))) return;
    const unsigned callv = ticket(ws + OFF_CNT + (blockIdx.x * 8 + wid) * 4, lane);
    if (blockIdx.x < 8) {
        const unsigned call = (unsigned)rfl((int)callv) + 1u;
        wait_words<32>(ws, call, lane, 1, ws + OFF_MS + ((CC_R + blockIdx.x) * 8 + (rank ^ 4)) * 128, call * 64u);
        if (lane == 0) st_sys(P.ws[((rank & 4) ^ 4) | blockIdx.x] + OFF_RDY + (rank * NCC * 8 + NEED_RELAY) * 4, call);
        return;
    }

    if (blockIdx.x < NDMA) {
        const int ps = blockIdx.x >> 3, c = (rank + ps) & 7;
        const auto prs = rsrc(pws, WS_NEED);
        const bool same = ((c ^ rank) & 4) == 0;
        const int d = same ? ((rank - c) & 3) - 1 : 3 + (rank & 3);
        if (wid == 0) {
            const auto xrs = rsrc(px, X_BYTES);
            const int h0 = xcd * PUSH_NP;
            v4u v[PUSH_NP * 8];
            unsigned dof[PUSH_NP];
#pragma unroll
            for (int i = 0; i < PUSH_NP; ++i) {
                const int sl = xslot((h0 + i) >> 1, d), half = (h0 + i) & 1;
                const unsigned so = (slot_row(c, sl) + half * 16) * 512;
                dof[i] = OFF_XS + rank * XS_SRC + sl * 16384 + half * 8192;
#pragma unroll
                for (int u = 0; u < 8; ++u) v[i * 8 + u] = __builtin_amdgcn_raw_buffer_load_b128(xrs, lin, so + u * 1024, 0);
            }
            const unsigned call = (unsigned)rfl((int)callv) + 1u;
            const unsigned po = (call & 1u) * XS_PAR + lin;
#pragma unroll
            for (int u = 0; u < PUSH_NP * 8; ++u) {
                if (u >= PUSH_WIN) asm volatile("s_waitcnt vmcnt(%0)" :: "n"(PUSH_WIN - 1) : "memory");
                st128<AUX_SYS>(v[u], prs, po + dof[u >> 3] + (u & 7) * 1024);
            }
            WAIT_VM0();
            if (lane == 0) __hip_atomic_fetch_add((gu32*)(pws + OFF_MS + rank * 128 + MS_XA * 1024), 8u * PUSH_NP, __ATOMIC_RELAXED, __HIP_MEMORY_SCOPE_SYSTEM);
        }
        if ((7 - wid) * 8 + xcd < NL) {
            const unsigned call = (unsigned)rfl((int)callv) + 1u;
            const unsigned par = call & 1u;
            const auto frs = rsrc(ws + OFF_RDY + c * 1024, 1024);
            const char* const exf = ws + OFF_EXF + c * 256 + lane * 4;
            const int nblk = same ? SAME_BLK : CROSS_BLK;
            const int dc = same ? ((c - rank) & 3) - 1 : 3 + (c & 3);
            const int B_WB = B_XB + SLOT_OWN - nblk - NXA;
            const auto ors = rsrc(P.out, OUT_BYTES);
            const float scale = P.sx[0] * P.sw[0];
            const int row0 = same ? 0 : SAME_BLK * 32;
            constexpr int PARTS = 16 / PULL_U;
            const int nca = same ? CC_A : CC_A / 4 * 3, ncb = same ? NCC - CC_A : (NCC - CC_A) / 4 * 3;
            const int nb1 = nca * 4, nb2 = nca * nblk;
            const bool relay = !same && ((c ^ rank) & 3) != 0;
            const int B_XO = B_WB + 4 * (NCC - CC_A) - (same ? 4 : 0), B_OUT = B_XO + 16;
            const unsigned nunits = (B_OUT + (nca + ncb) * nblk + (relay ? 4 : 0)) * PARTS;
            char* const q = ws + OFF_Q + ps * 128;
            u64 exm = 0;
            u64 r0 = 0, r1 = 0, r2 = 0, r3 = 0;
            const unsigned ol = (lane >> 5) * (NTOT * 4) + (lane & 31) * 16;
            bool dead = false;
            auto decode = [&](unsigned un, unsigned& so, unsigned& dof, int& ms, int& need) {
                const int part = un % PARTS;
                int t = un / PARTS;
                const bool rl = relay && t >= B_OUT + nb1 && t < B_OUT + nb1 + 4;
                if (t < B_OUT || rl) {
                    if (rl) {
                        const int i = t - B_OUT - nb1, src = c ^ 4;
                        ms = (CC_R + (c & 3)) * 8 + src;
                        need = NEED_RELAY;
                        so = OFF_WS + par * WS_PAR + src * WS_SRC + (CC_R + (rank & 3)) * 65536 + i * 16384;
                        dof = OFF_WS + par * WS_PAR + src * WS_SRC + (CC_R + (c & 3)) * 65536 + i * 16384;
                    } else if ((t >= B_WA && t < B_XB) || (t >= B_WB && t < B_XO)) {
                        int i = t < B_XB ? t - B_WA : t - B_WB + 4 * CC_A;
                        if (same && i >= 4 * (CC_R + (c & 3))) i += 4;
                        const int cc = i >> 2;
                        ms = cc * 8 + c;
                        need = NEED_EX + cc;
                        so = OFF_EX + par * EX_PAR + chunk_col(rank, cc) * 512 + (i & 3) * 16384;
                        dof = OFF_WS + par * WS_PAR + c * WS_SRC + i * 16384;
                    } else {
                        int sl = t < B_WB ? xslot(t < B_WA ? t + NPS : t - B_XB + NXA, dc) : t - B_XO + SLOT_OWN;
                        ms = (sl < SLOT_A ? MS_XA : sl < SLOT_B2 ? MS_XB1 : sl < SLOT_B3 ? MS_XB2 : sl < SLOT_OWN ? MS_XB3 : MS_XO) * 8 + c;
                        const int row = slot_row(rank, sl);
                        need = NEED_EX + 32 + (row >> 7);
                        so = OFF_EX + par * EX_PAR + W_BYTES + row * 512;
                        dof = OFF_XS + par * XS_PAR + c * XS_SRC + sl * 16384;
                    }
                    so += part * (PULL_U * 1024);
                    dof += part * (PULL_U * 1024);
                    return;
                }
                t -= B_OUT;
                if (relay && t >= nb1) t -= 4;
                int ci, blk;
                if (t < nb1) { ci = t >> 2; blk = t & 3; }
                else if (t < nb2) { blk = t / nca; ci = t - blk * nca; }
                else { ci = t / nblk; blk = t - ci * nblk; }
                int cc = ci;
                if (!same) { int sub = ci / 3, gi = ci - sub * 3; cc = sub * 4 + (gi < (rank & 3) ? gi : gi + 1); }
                const int e = entry_of(cc & 3, d, blk);
                ms = -1;
                need = cc * 8 + (e >> 3);
                so = OFF_OS + (cc * 48 + e) * 16384 + part * (PULL_U * 1024);
                dof = (unsigned)(row0 + blk * 32 + part * (2 * PULL_U)) * (NTOT * 4) + chunk_col(c, cc) * 4;
            };
            auto put = [&](v4u (&v)[PULL_U], unsigned dof, int ms) {
                if (ms < 0) {
                    const float sc = dead ? __builtin_nanf("") : scale;
#pragma unroll
                    for (int u = 0; u < PULL_U; ++u) {
                        v4f o;
#pragma unroll
                        for (int j = 0; j < 4; ++j) o[j] = silu((int)v[u][j], sc);
                        st128<AUX_SYS>(__builtin_bit_cast(v4u, o), ors, dof + ol + u * 2 * (NTOT * 4));
                    }
                } else {
                    const auto rs = rsrc(ws, WS_NEED);
#pragma unroll
                    for (int u = 0; u < PULL_U; ++u) st128<AUX_SYS>(v[u], rs, dof + lin + u * 1024);
                }
            };
            auto signal = [&](int ms) {
                if (ms >= 0 && lane == 0)
                    __hip_atomic_fetch_add((gu32*)(ws + OFF_MS + ms * 128), (unsigned)PULL_U, __ATOMIC_RELAXED, __HIP_MEMORY_SCOPE_AGENT);
            };
            auto masks = [&](v4u f) {
                r0 = ballot((int)(f[0] - call) >= 0); r1 = ballot((int)(f[1] - call) >= 0);
                r2 = ballot((int)(f[2] - call) >= 0); r3 = ballot((int)(f[3] - call) >= 0);
            };
            auto have = [&](int k) {
                if (k >= NEED_EX) return rfl((int)((exm >> (k - NEED_EX)) & 1)) != 0;
                const int sh = k >> 2;
                const unsigned m = (unsigned)((r0 >> sh) & 1) | (unsigned)((r1 >> sh) & 1) << 1 | (unsigned)((r2 >> sh) & 1) << 2 | (unsigned)((r3 >> sh) & 1) << 3;
                return rfl((int)((m >> (k & 3)) & 1)) != 0;
            };
            auto await = [&](int k) {
                const u64 t0 = __builtin_amdgcn_s_memrealtime();
                unsigned round = 1;
                for (;;) {
                    asm volatile("" ::: "memory");
                    if (k >= NEED_EX) {
                        exm = ballot((int)(ld_sys(exf) - call) >= 0);
                    } else {
                        masks(__builtin_amdgcn_raw_buffer_load_b128(frs, lin, 0, AUX_SYS));
                    }
                    if (have(k)) break;
                    if (expired(ws, call, t0, round)) { dead = true; break; }
                    __builtin_amdgcn_s_sleep(1);
                }
            };
            constexpr int NONE = -1000;
            unsigned un1 = (7 - wid) * 8 + xcd;
            constexpr unsigned PER = NL * TAU, TOL = PER * LATE / 16;
            const unsigned phase = un1 * TAU;
            unsigned tnext = (tstart << 8) - 4 * PER;
            unsigned so, dof, pdof = 0;
            int ms, need, pms = NONE, ppms = NONE;
            v4u pv[PULL_U], v2[PULL_U];
            bool pok = false;
            const char* const pcnt = ws + OFF_MS + MS_PART * 1024;
            decode(un1, so, dof, ms, need);
            un1 += NL;
            for (int u = 0; u < PULL_U; ++u) pv[u] = v2[u] = v4u{0, 0, 0, 0};
            auto add2 = [&]() {
#pragma unroll
                for (int u = 0; u < PULL_U; ++u) { pv[u] += v2[u]; asm volatile("" : "+v"(pv[u])); }
            };
            auto flush = [&]() {
                add2();
                if (pms != NONE) put(pv, pdof, pms);
                signal(ppms);
                if (pms >= 0) { WAIT_VM0(); signal(pms); }
                pms = ppms = NONE;
            };
            for (;;) {
                if (!dead && (!have(need) || (ms < 0 && !pok))) {
                    flush();
                    if (!have(need)) await(need);
                    if (ms < 0 && !pok && !dead) { dead = !wait_words<1>(ws, call, lane, 1, pcnt, call * 192u); pok = true; }
                }
                {
                    const unsigned now = (unsigned)__builtin_amdgcn_s_memrealtime() << 8;
                    if (now - tnext + PER >= PER + TOL) tnext = (now - TOL - phase + PER) / PER * PER + phase;
                    for (int i = 0; i < 256 && (int)(((unsigned)__builtin_amdgcn_s_memrealtime() << 8) - tnext) < 0; ++i) __builtin_amdgcn_s_sleep(1);
                    tnext += PER;
                }
                asm volatile("" ::: "memory");
                v4u v[PULL_U], f;
                unsigned fe, fp;
#pragma unroll
                for (int u = 0; u < PULL_U; ++u) v[u] = __builtin_amdgcn_raw_buffer_load_b128(prs, lin, so + u * 1024, AUX_SYS);
                add2();
                if (pms != NONE) put(pv, pdof, pms);
                signal(ppms);
                const unsigned nxt = ticket(q, lane);
                ppms = pms; pms = ms; pdof = dof;
                const bool more = un1 < nunits;
                if (more) decode(un1, so, dof, ms, need);
                const bool miss = more && !have(need);
                const bool ref_e = miss && need >= NEED_EX, ref_r = miss && need < NEED_EX;
                if (ref_e) fe = ld_sys(exf);
                if (ref_r) f = __builtin_amdgcn_raw_buffer_load_b128(frs, lin, 0, AUX_SYS);
                const bool ref_p = more && ms < 0 && !pok;
                if (ref_p) fp = ld_sys(pcnt);
                asm volatile("" ::: "memory");
#pragma unroll
                for (int u = 0; u < PULL_U; ++u)
                    v2[u] = __builtin_amdgcn_raw_buffer_load_b128(ors, ol, pms < 0 ? pdof + u * 2 * (NTOT * 4) : OUT_BYTES, AUX_SYS);
                asm volatile("s_waitcnt vmcnt(%0)" :: "n"(PULL_U) : "memory");
#pragma unroll
                for (int u = 0; u < PULL_U; ++u) pv[u] = v[u];
                if (!more) break;
                if (ref_e) exm = ballot((int)(fe - call) >= 0);
                if (ref_r) masks(f);
                if (ref_p) pok = rfl((int)(fp - call * 192u)) >= 0;
                un1 = (unsigned)rfl((int)nxt) - (call - 1u) * nunits + 2 * NL;
            }
            flush();
        }
        return;
    }

    const int job = blockIdx.x - NDMA;
    unsigned call;
    if (job < 160) {
        const bool isw = job < 128;
        const int gid = job & 15, sub = job >> 4, piece = isw ? gid * 8 + sub : job;
        const auto rs = rsrc(isw ? pw : px, isw ? W_BYTES : X_BYTES);
        const unsigned so = isw ? (wid * 64 + (lane >> 3)) * NTOT + piece * 128 + (lane & 7) * 16 : (job - 128) * 65536 + wid * 8192 + lin;
        const unsigned st = isw ? 8 * NTOT : 1024;
        v4u v[8];
        const int to = isw ? (lane & 1 ? 4 + (gid & 3) : gid >> 2) : lane & 7;
        char* const flag = P.ws[to] + OFF_EXF + (rank * 64 + (isw ? sub * 4 + (lane & 1 ? gid >> 2 : gid & 3) : job - 96)) * 4;
        if (isw && sub >= 1) __builtin_amdgcn_s_sleep(100);
#pragma unroll
        for (int u = 0; u < 8; ++u) v[u] = __builtin_amdgcn_raw_buffer_load_b128(rs, so + u * st, 0, 0);
        call = (unsigned)rfl((int)callv) + 1u;
        const auto wrs = rsrc(ws, WS_NEED);
        const unsigned dof = OFF_EX + (call & 1u) * EX_PAR + piece * 65536 + wid * 8192 + lin;
#pragma unroll
        for (int u = 0; u < 8; ++u) st128<AUX_SYS>(v[u], wrs, dof + u * 1024);
        WAIT_VM0();
        __syncthreads();
        if (threadIdx.x < (isw ? 2 : 8) && to != rank) st_sys(flag, call);
    } else {
        call = (unsigned)rfl((int)callv) + 1u;
    }
    const unsigned par = call & 1u;
    const int li = job >> 3;
    const float scale = P.sx[0] * P.sw[0];
    const int wm = wid >> 1, wn = wid & 1, fr = lane & 15, fq = lane >> 4;
    const unsigned lds0 = (unsigned)(size_t)(LDS_AS char*)smem;
    volatile LDS_AS int* const dead_w = (volatile LDS_AS int*)((LDS_AS char*)smem + 3 * STAGE);
    unsigned xa[2], wa[4];
    for (int kh = 0; kh < 2; ++kh) xa[kh] = (wm * 64 + fr) * 128 + (((kh * 4 + fq) ^ (fr >> 1)) << 4);
    for (int ni = 0; ni < 4; ++ni)
        wa[ni] = 32768 + (fq * 16 + (fr >> 1)) * 128 + (((wn * 4 + ni) ^ (((fr >> 2) & 3) | ((fq & 1) << 2))) << 4) + 8 * (fr & 1);

    const int tend = (xcd < 4 && li < 16) ? 4 : 3;
    for (int ti = xcd < 4 ? 2 : 0; ti < tend; ++ti) {
        int cc = 0, rt, col;
        if (ti < 2) {
            const int sub = ti * 4 + xcd - 4, i12 = li >> 1, me = rank & 3;
            const int gid = (rank & 4) ? (i12 / 3) * 4 + (i12 % 3 < me ? i12 % 3 : i12 % 3 + 1) : (i12 < 4 * me ? i12 : i12 + 4);
            rt = 6 + (li & 1);
            col = gid * 1024 + sub * 128;
            while ((unsigned)__builtin_amdgcn_s_memrealtime() - tstart < PART_T0 + sub * PART_DT) __builtin_amdgcn_s_sleep(8);
        } else {
            if (ti == 3) { int idx = xcd * 16 + li; cc = idx >> 1; rt = 6 + (idx & 1); }
            else if (xcd < 2) { cc = li / 3; rt = xcd * 3 + li % 3; }
            else { cc = xcd * 4 + li / 6; rt = li % 6; }
            col = chunk_col(rank, cc);
        }
        cc = rfl(cc); rt = rfl(rt); col = rfl(col);
        const int g = cc & 3;
        bool dead = false;

        v4i acc[4][4];
#pragma unroll
        for (int i = 0; i < 4; ++i)
#pragma unroll
            for (int j = 0; j < 4; ++j) acc[i][j] = v4i{0, 0, 0, 0};

#pragma unroll 1
        for (int seg = 0; seg < (ti < 2 ? 1 : 2); ++seg) {
            if (seg == 1) {
                if (wid == 0) {
                    const int m1 = rt < 3 ? MS_XA : rt < 5 ? MS_XB1 : rt < 6 ? MS_XB3 : MS_XO;
                    const unsigned n1 = rt < 3 ? NXA : rt < 5 ? 12 : rt < 6 ? (((lane ^ rank) & 4) ? 6 : 8) : 16;
                    const u64 peers = 0xffull & ~(1ull << rank);
                    const char* mb = ws + OFF_MS + (lane & 7) * 128;
                    bool ok = wait_words<64>(ws, call, lane, peers, mb + cc * 1024, call * 64u);
                    if (ok) ok = wait_words<64>(ws, call, lane, (rt == 4 || rt == 5) ? peers | peers << 8 : peers,
                                               mb + (lane & 8 ? MS_XB2 : m1) * 1024, call * 16u * (lane & 8 ? 6 : n1));
                    __builtin_amdgcn_fence(__ATOMIC_ACQUIRE, "");
                    *dead_w = ok ? 0 : 1;
                    WAIT_VM0();
                }
                __syncthreads();
                dead = rfl(*dead_w) != 0;
                if (dead) break;
            }
            unsigned xo[4], wo[2];
            int own[4];
#pragma unroll
            for (int i = 0; i < 4; ++i) {
                int row = i * 64 + wid * 8 + (lane >> 3), e = rt * 8 + i * 2 + (wid >> 2), d, slot;
                if (rt >= 6) { slot = SLOT_OWN + e - 48; own[i] = -1; } else { entry_inv(g, e, d, slot); own[i] = seg ? dest_rank(rank, d) : -1; }
                xo[i] = (seg ? slot * 32 : slot_row(rank, slot)) * 512 + (row & 31) * 512 + (((lane & 7) ^ ((row >> 1) & 7)) << 4);
            }
#pragma unroll
            for (int i = 0; i < 2; ++i) {
                int k = i * 64 + wid * 8 + (lane >> 3);
                wo[i] = k * (seg ? 128 : NTOT) + (((lane & 7) ^ (((k >> 1) & 3) | (((k >> 4) & 1) << 2))) << 4);
            }
            const auto xrs = rsrc(seg ? ws : px, seg ? WS_NEED : X_BYTES);
            const auto wrs = rsrc(seg ? ws : pw, seg ? WS_NEED : W_BYTES);
            const unsigned xs = seg ? OFF_XS + par * XS_PAR : 0;
            const unsigned wsb = seg ? OFF_WS + par * WS_PAR + cc * 65536 : col;
            const unsigned wstep = seg ? 16384 : 128 * NTOT;
            const int nt = seg ? 28 : 4;
            auto stage = [&](int t, int buf) {
                int p = t >> 2;
                p = seg ? p + (p >= rank) : 0;
                const unsigned xb = xs + p * XS_SRC + (t & 3) * 128;
                const unsigned wb = wsb + p * WS_SRC + (t & 3) * wstep;
                const unsigned l = lds0 + buf * STAGE + wid * 1024;
                const int x0 = rfl(own[0] == p ? WS_NEED : xb), x1 = rfl(own[1] == p ? WS_NEED : xb);
                const int x2 = rfl(own[2] == p ? WS_NEED : xb), x3 = rfl(own[3] == p ? WS_NEED : xb);
                unsigned keep;
                asm volatile(
                    "s_nop 4\n\ts_mov_b32 %0, m0\n\t"
                    "s_mov_b32 m0, %1\n\ts_nop 0\n\tbuffer_load_dwordx4 %9, %2, %4 offen lds\n\t"
                    "s_add_u32 m0, m0, 0x2000\n\ts_nop 0\n\tbuffer_load_dwordx4 %10, %2, %5 offen lds\n\t"
                    "s_add_u32 m0, m0, 0x2000\n\ts_nop 0\n\tbuffer_load_dwordx4 %11, %2, %6 offen lds\n\t"
                    "s_add_u32 m0, m0, 0x2000\n\ts_nop 0\n\tbuffer_load_dwordx4 %12, %2, %7 offen lds\n\t"
                    "s_add_u32 m0, m0, 0x2000\n\ts_nop 0\n\tbuffer_load_dwordx4 %13, %3, %8 offen lds\n\t"
                    "s_add_u32 m0, m0, 0x2000\n\ts_nop 0\n\tbuffer_load_dwordx4 %14, %3, %8 offen lds\n\t"
                    "s_mov_b32 m0, %0"
                    : "=&s"(keep)
                    : "s"(l), "s"(xrs), "s"(wrs), "s"(x0), "s"(x1), "s"(x2), "s"(x3), "s"(wb),
                      "v"(xo[0]), "v"(xo[1]), "v"(xo[2]), "v"(xo[3]), "v"(wo[0]), "v"(wo[1])
                    : "memory", "scc");
            };
            v4i xf[2][4], wf[2][4];
            auto frags = [&](int buf, int kh) {
                LDS_AS char* sb = (LDS_AS char*)smem + buf * STAGE;
#pragma unroll
                for (int ni = 0; ni < 4; ++ni) {
                    v2i lo = __builtin_amdgcn_ds_read_tr8_b64_v2i32((LDS_AS v2i*)(sb + wa[ni] + kh * 8192));
                    v2i hi = __builtin_amdgcn_ds_read_tr8_b64_v2i32((LDS_AS v2i*)(sb + wa[ni] + kh * 8192 + 1024));
                    wf[kh][ni] = v4i{lo[0], lo[1], hi[0], hi[1]};
                }
#pragma unroll
                for (int mi = 0; mi < 4; ++mi) xf[kh][mi] = *(LDS_AS v4i*)(sb + xa[kh] + mi * 2048);
            };
            auto mma = [&](int kh) {
#pragma unroll
                for (int mi = 0; mi < 4; ++mi)
#pragma unroll
                    for (int ni = 0; ni < 4; ++ni)
                        acc[mi][ni] = __builtin_amdgcn_mfma_i32_16x16x64_i8(wf[kh][ni], xf[kh][mi], acc[mi][ni], 0, 0, 0);
            };
            WAIT_VM0();
            stage(0, 0);
            stage(1, 1);
            stage(2, 2);
            asm volatile("s_waitcnt vmcnt(12)" ::: "memory");
            __builtin_amdgcn_s_barrier();
            asm volatile("" ::: "memory");
            frags(0, 0);
            int buf = 0;
            for (int t = 0; t < nt; ++t) {
                frags(buf, 1);
                __builtin_amdgcn_sched_barrier(0);
                mma(0);
                __builtin_amdgcn_sched_barrier(0);
                if (t < nt - 2) asm volatile("s_waitcnt vmcnt(6) lgkmcnt(0)" ::: "memory");
                else asm volatile("s_waitcnt vmcnt(0) lgkmcnt(0)" ::: "memory");
                __builtin_amdgcn_s_barrier();
                asm volatile("" ::: "memory");
                if (t < nt - 3) stage(t + 3, buf);
                buf = buf == 2 ? 0 : buf + 1;
                if (t < nt - 1) frags(buf, 0);
                __builtin_amdgcn_sched_barrier(0);
                mma(1);
                __builtin_amdgcn_sched_barrier(0);
            }
        }
        {
            const bool loc = rt >= 6;
            const float sc = dead ? __builtin_nanf("") : scale;
            const auto rs = loc ? rsrc(P.out, OUT_BYTES) : rsrc(ws, WS_NEED);
            const unsigned pitch = loc ? NTOT * 4 : 512;
            const unsigned base = loc ? (unsigned)(rt - 6) * 256 * (NTOT * 4) + col * 4 : OFF_OS + (cc * 6 + rt) * 131072;
            const unsigned lo = (wm * 64 + fr) * pitch + wn * 256 + fq * 16;
            if (ti == 3) {
#pragma unroll
                for (int mi = 0; mi < 4; ++mi)
#pragma unroll
                    for (int ni = 0; ni < 4; ++ni)
#pragma unroll
                        for (int j = 0; j < 4; ++j) acc[mi][ni][j] = __builtin_bit_cast(int, silu(acc[mi][ni][j], sc));
            }
#pragma unroll
            for (int mi = 0; mi < 4; ++mi)
#pragma unroll
                for (int ni = 0; ni < 4; ++ni)
                    st128<AUX_SYS>(__builtin_bit_cast(v4u, acc[mi][ni]), rs, lo + base + mi * 16 * pitch + ni * 64);
            WAIT_VM0();
            __syncthreads();
            if (ti < 2 && threadIdx.x == 0)
                __hip_atomic_fetch_add((gu32*)(ws + OFF_MS + MS_PART * 1024), 1u, __ATOMIC_RELAXED, __HIP_MEMORY_SCOPE_AGENT);
            if (ti == 2 && threadIdx.x < (rt < 5 ? 6 : 3)) {
                int d, slot;
                entry_inv(g, rt * 8 + (rt < 5 ? 0 : 2) + threadIdx.x, d, slot);
                st_sys(P.ws[dest_rank(rank, d) & 7] + OFF_RDY + ((rank * NCC + cc) * 8 + rt) * 4, call);
                if (dead) st_sys(P.out + col * 4, 0x7fc00000u);
            }
        }
    }
}

}

extern "C" void kernel_launch(int rank, int nranks, ncclComm_t comm,
                              hipStream_t stream,
                              void* const* d_in, const int* in_sizes, int n_in,
                              void* d_out, int out_size,
                              const int* in_dtypes, int out_dtype,
                              void* const* d_out_all,
                              void* const* d_ws_all, size_t ws_bytes) {
    if (nranks != NR || (unsigned)rank >= (unsigned)NR || ws_bytes < WS_NEED || n_in != 4 ||
        in_sizes[0] != (int)X_BYTES || in_sizes[1] != (int)W_BYTES || out_size != 512 * NTOT) {
        fprintf(stderr, "unexpected problem shape\n");
        return;
    }
    Params p;
    p.x = (const char*)d_in[0]; p.w = (const char*)d_in[1];
    p.sx = (const float*)d_in[2]; p.sw = (const float*)d_in[3];
    p.out = (char*)d_out;
    for (int i = 0; i < NR; ++i) { p.ws[i] = (char*)d_ws_all[phys(i)]; if (phys(i) == rank) p.rank = i; }
    for (int i = 0; i < NR; ++i) p.wsr[i] = p.ws[(p.rank + i) & 7];
    hipLaunchKernelGGL(fused_gemm_scatter, dim3(NWG), dim3(512), 0, stream, p);
    HIP_CHECK(hipGetLastError());
}
```

```cpp
#include <hip/hip_runtime.h>
#include <rccl/rccl.h>
#include <cstdio>

#define HIP_CHECK(expr) do { hipError_t _e = (expr); if (_e != hipSuccess) { \
    fprintf(stderr, "HIP error %s at %s:%d\n", hipGetErrorString(_e), __FILE__, __LINE__); \
    return; } } while (0)

#define PULL_NL 56
#define TAU 864
#define LATE 3
#define PUSH_NP 4
#define PUSH_WIN 6
#define PART_T0 600u
#define PART_DT 300u

namespace {

typedef int v2i __attribute__((ext_vector_type(2)));
typedef int v4i __attribute__((ext_vector_type(4)));
typedef unsigned v4u __attribute__((ext_vector_type(4)));
typedef float v4f __attribute__((ext_vector_type(4)));
typedef unsigned long long u64;
#define LDS_AS __attribute__((address_space(3)))
#define GLB_AS __attribute__((address_space(1)))
typedef GLB_AS unsigned gu32;

constexpr int NR = 8;
constexpr int NTOT = 16384;
constexpr int SAME_BLK = 7, CROSS_BLK = 9;
constexpr int SLOT_A = 28;
constexpr int SLOT_B2 = 42, SLOT_B3 = 49, SLOT_OWN = 57;
constexpr int NSLOT = SLOT_OWN + 16;
constexpr int NCC = 32;
constexpr int CC_A = 8;
constexpr int NDMA = 64;
constexpr int NWG = 256;
constexpr int NL = PULL_NL, PULL_U = 2;
constexpr unsigned PERM = 0x75416320u;
constexpr int MS_XA = 32, MS_XB1 = 33, MS_XB2 = 34, MS_XB3 = 35, MS_XO = 36, MS_PART = 37;
constexpr int NPS = 4 * PUSH_NP;
constexpr int NXA = 24;
constexpr int B_WA = NXA - NPS, B_XB = B_WA + 4 * CC_A;
constexpr int CC_R = 28, NEED_RELAY = 7, NEED_EX = 256;
static_assert(NPS <= NXA, "pushed slots");

constexpr unsigned OFF_CNT = 0;
constexpr unsigned OFF_TMO = 8192;
constexpr unsigned OFF_Q = 12288;
constexpr unsigned OFF_RDY = 16384;
constexpr unsigned OFF_EXF = 32768;
constexpr unsigned OFF_MS = 262144;
constexpr unsigned XS_SRC = NSLOT * 16384;
constexpr unsigned XS_PAR = NR * XS_SRC;
constexpr unsigned WS_SRC = NCC * 65536;
constexpr unsigned WS_PAR = NR * WS_SRC;
constexpr unsigned EX_PAR = 160 * 65536;
constexpr unsigned OFF_XS = 1u << 20;
constexpr unsigned OFF_WS = OFF_XS + 2 * XS_PAR;
constexpr unsigned OFF_EX = OFF_WS + 2 * WS_PAR;
constexpr unsigned OFF_OS = OFF_EX + 2 * EX_PAR;
constexpr unsigned WS_NEED = OFF_OS + NCC * 6 * 131072;
constexpr unsigned OUT_BYTES = 512u * NTOT * 4u;
constexpr unsigned X_BYTES = 4096u * 512u, W_BYTES = 512u * NTOT;

constexpr u64 DEADLINE = 100000000ull;
constexpr int STAGE = 49152;
constexpr int AUX_SYS = 17;

struct Params {
    const char* x; const char* w; const float* sx; const float* sw;
    char* out; char* ws[NR];
    char* wsr[NR];
    int rank;
};

__device__ __forceinline__ int rfl(int v) { return __builtin_amdgcn_readfirstlane(v); }
__device__ __forceinline__ u64 ballot(bool b) { return __builtin_amdgcn_ballot_w64(b); }
__device__ __forceinline__ auto rsrc(const void* p, unsigned bytes) {
    return __builtin_amdgcn_make_buffer_rsrc((void*)p, 0, bytes, 0x00020000);
}
template <int AUX, class R> __device__ __forceinline__ void st128(v4u v, R rs, unsigned off) {
    __builtin_amdgcn_raw_buffer_store_b128(v, rs, off, 0, AUX);
}
__host__ __device__ __forceinline__ int phys(int l) { return (PERM >> (4 * l)) & 7; }
__device__ __forceinline__ int dest_rank(int c, int d) { return d < 3 ? (c & 4) | ((c + 1 + d) & 3) : ((c & 4) ^ 4) | (d - 3); }
__device__ __forceinline__ int slot_row(int c, int s) {
    if (s >= SLOT_OWN) return phys(c) * 512 + (s - SLOT_OWN) * 32;
    int j, d;
    if (s < SLOT_B3) { j = s / 7; d = s - j * 7; } else { j = 7 + ((s - SLOT_B3) >> 2); d = 3 + ((s - SLOT_B3) & 3); }
    return phys(dest_rank(c, d)) * 512 + (d < 3 ? 0 : SAME_BLK * 32) + j * 32;
}
__device__ __forceinline__ int xslot(int n, int skip) {
    if (n < 42) { int j = n / 6, i = n - j * 6; return j * 7 + (i < skip ? i : i + 1); }
    n -= 42;
    if (skip < 3) return SLOT_B3 + n;
    int j = n / 3, i = n - j * 3;
    return SLOT_B3 + j * 4 + (i < skip - 3 ? i : i + 1);
}
__device__ __forceinline__ int entry_of(int g, int d, int j) {
    int i = d < 3 + g ? d : d - 1;
    return j < 7 ? j * 6 + i : 42 + (j - 7) * 3 + (i - 3);
}
__device__ __forceinline__ void entry_inv(int g, int e, int& d, int& slot) {
    int j, i;
    if (e < 42) { j = e / 6; i = e - j * 6; } else { j = 7 + (e - 42) / 3; i = 3 + (e - 42) % 3; }
    d = i < 3 + g ? i : i + 1;
    slot = j < 7 ? j * 7 + d : SLOT_B3 + (j - 7) * 4 + (d - 3);
}
__device__ __forceinline__ int chunk_col(int c, int cc) {
    int g = cc & 3, gid = (c & 4) ? g * 4 + (c & 3) : (c & 3) * 4 + g;
    return gid * 1024 + (cc >> 2) * 128;
}
__device__ __forceinline__ unsigned ld_sys(const char* p) {
    return __hip_atomic_load((gu32*)p, __ATOMIC_RELAXED, __HIP_MEMORY_SCOPE_SYSTEM);
}
__device__ __forceinline__ void st_sys(char* p, unsigned v) {
    __hip_atomic_store((gu32*)p, v, __ATOMIC_RELAXED, __HIP_MEMORY_SCOPE_SYSTEM);
}
__device__ __forceinline__ void set_tmo(char* ws, unsigned call) {
    __hip_atomic_store((gu32*)(ws + OFF_TMO), call, __ATOMIC_RELAXED, __HIP_MEMORY_SCOPE_AGENT);
}
__device__ __forceinline__ bool expired(char* ws, unsigned call, u64 t0, unsigned& round) {
    bool out = __builtin_amdgcn_s_memrealtime() - t0 > DEADLINE;
    if ((round++ & 7u) == 0u) out = out || rfl((int)ld_sys(ws + OFF_TMO)) == (int)call;
    if (out) set_tmo(ws, call);
    return out;
}
template <int SLEEP> __device__ __forceinline__ bool wait_words(char* ws, unsigned call, int lane, u64 mask, const char* p, unsigned target) {
    const u64 t0 = __builtin_amdgcn_s_memrealtime();
    if (!((mask >> lane) & 1)) p = ws + OFF_TMO;
    for (;;) {
        const unsigned v = ld_sys(p);
        if (ballot(v == call) >> 63) return false;
        if ((ballot((int)(v - target) < 0) & mask) == 0) return true;
        if (__builtin_amdgcn_s_memrealtime() - t0 > DEADLINE) { set_tmo(ws, call); return false; }
        __builtin_amdgcn_s_sleep(SLEEP);
    }
}
#define WAIT_VM0() asm volatile("s_waitcnt vmcnt(0)" ::: "memory")

__device__ __forceinline__ float silu(int acc, float s) {
    float y = (float)acc * s;
    float c = fminf(fmaxf(y, -60.0f), 60.0f);
    float e = __builtin_amdgcn_exp2f(c * -1.4426950408889634f);
    return y * __builtin_amdgcn_rcpf(1.0f + e);
}
__device__ __forceinline__ unsigned ticket(char* p, int lane) {
    unsigned v = 0, z = 0;
    asm volatile("" : "+v"(z));
    if (lane == 0) v = __hip_atomic_fetch_add((gu32*)(p + z), 1u, __ATOMIC_RELAXED, __HIP_MEMORY_SCOPE_AGENT);
    return v;
}

__global__ void __launch_bounds__(512, 2) fused_gemm_scatter(Params P) {
    __shared__ __attribute__((aligned(16))) char smem[3 * STAGE + 16];
    const int rank = P.rank;
    const int lane = threadIdx.x & 63, wid = rfl(threadIdx.x >> 6);
    char* const ws = P.wsr[0];
    char* const pws = P.wsr[(blockIdx.x >> 3) & 7];
    const char* const px = P.x;
    const char* const pw = P.w;
    asm volatile("" :: "s"(rank), "s"(ws), "s"(pws), "s"(px), "s"(pw));
    const int xcd = blockIdx.x & 7;
    const unsigned tstart = (unsigned)__builtin_amdgcn_s_memrealtime();
    const unsigned lin = lane * 16;
    if (blockIdx.x < 8 && (wid != 0 || blockIdx.x >= 4 || (int)blockIdx.x == (rank & 3))) return;
    const unsigned callv = ticket(ws + OFF_CNT + (blockIdx.x * 8 + wid) * 4, lane);
    const int ps = blockIdx.x >> 3, c = (rank + ps) & 7;
    const auto prs = rsrc(pws, WS_NEED);
    const bool same = ((c ^ rank) & 4) == 0;
    const int d = same ? ((rank - c) & 3) - 1 : 3 + (rank & 3);
    if (blockIdx.x >= 8 && blockIdx.x < NDMA && wid == 0) {
        const auto xrs = rsrc(px, X_BYTES);
        const int h0 = xcd * PUSH_NP;
        v4u v[PUSH_NP * 8];
        unsigned dof[PUSH_NP];
#pragma unroll
        for (int i = 0; i < PUSH_NP; ++i) {
            const int sl = xslot((h0 + i) >> 1, d), half = (h0 + i) & 1;
            const unsigned so = (slot_row(c, sl) + half * 16) * 512;
            dof[i] = OFF_XS + rank * XS_SRC + sl * 16384 + half * 8192;
#pragma unroll
            for (int u = 0; u < 8; ++u) v[i * 8 + u] = __builtin_amdgcn_raw_buffer_load_b128(xrs, lin, so + u * 1024, 0);
        }
        const unsigned call = (unsigned)rfl((int)callv) + 1u;
        const unsigned po = (call & 1u) * XS_PAR + lin;
#pragma unroll
        for (int u = 0; u < PUSH_NP * 8; ++u) {
            if (u >= PUSH_WIN) asm volatile("s_waitcnt vmcnt(%0)" :: "n"(PUSH_WIN - 1) : "memory");
            st128<AUX_SYS>(v[u], prs, po + dof[u >> 3] + (u & 7) * 1024);
        }
        WAIT_VM0();
        if (lane == 0) __hip_atomic_fetch_add((gu32*)(pws + OFF_MS + rank * 128 + MS_XA * 1024), 8u * PUSH_NP, __ATOMIC_RELAXED, __HIP_MEMORY_SCOPE_SYSTEM);
    }
    const int job = blockIdx.x - NDMA;
    unsigned call;
    if (__builtin_expect(job >= 0 && job < 160, 1)) {
        const bool isw = job < 128;
        const int gid = job & 15, sub = job >> 4, piece = isw ? gid * 8 + sub : job;
        const auto rs = rsrc(isw ? pw : px, isw ? W_BYTES : X_BYTES);
        const unsigned so = isw ? (wid * 64 + (lane >> 3)) * NTOT + piece * 128 + (lane & 7) * 16 : (job - 128) * 65536 + wid * 8192 + lin;
        const unsigned st = isw ? 8 * NTOT : 1024;
        v4u v[8];
        const int to = isw ? (lane & 1 ? 4 + (gid & 3) : gid >> 2) : lane & 7;
        char* const flag = P.ws[to] + OFF_EXF + (rank * 64 + (isw ? sub * 4 + (lane & 1 ? gid >> 2 : gid & 3) : job - 96)) * 4;
        if (isw && sub >= 1) __builtin_amdgcn_s_sleep(100);
#pragma unroll
        for (int u = 0; u < 8; ++u) v[u] = __builtin_amdgcn_raw_buffer_load_b128(rs, so + u * st, 0, 0);
        call = (unsigned)rfl((int)callv) + 1u;
        const auto wrs = rsrc(ws, WS_NEED);
        const unsigned dof = OFF_EX + (call & 1u) * EX_PAR + piece * 65536 + wid * 8192 + lin;
#pragma unroll
        for (int u = 0; u < 8; ++u) st128<AUX_SYS>(v[u], wrs, dof + u * 1024);
        WAIT_VM0();
        __syncthreads();
        if (threadIdx.x < (isw ? 2 : 8) && to != rank) st_sys(flag, call);
    }
    if (blockIdx.x < 8) {
        const unsigned call = (unsigned)rfl((int)callv) + 1u;
        wait_words<32>(ws, call, lane, 1, ws + OFF_MS + ((CC_R + blockIdx.x) * 8 + (rank ^ 4)) * 128, call * 64u);
        if (lane == 0) st_sys(P.ws[((rank & 4) ^ 4) | blockIdx.x] + OFF_RDY + (rank * NCC * 8 + NEED_RELAY) * 4, call);
        return;
    }

    if (blockIdx.x < NDMA) {
        if ((7 - wid) * 8 + xcd < NL) {
            const unsigned call = (unsigned)rfl((int)callv) + 1u;
            const unsigned par = call & 1u;
            const auto frs = rsrc(ws + OFF_RDY + c * 1024, 1024);
            const char* const exf = ws + OFF_EXF + c * 256 + lane * 4;
            const int nblk = same ? SAME_BLK : CROSS_BLK;
            const int dc = same ? ((c - rank) & 3) - 1 : 3 + (c & 3);
            const int B_WB = B_XB + SLOT_OWN - nblk - NXA;
            const auto ors = rsrc(P.out, OUT_BYTES);
            const float scale = P.sx[0] * P.sw[0];
            const int row0 = same ? 0 : SAME_BLK * 32;
            constexpr int PARTS = 16 / PULL_U;
            const int nca = same ? CC_A : CC_A / 4 * 3, ncb = same ? NCC - CC_A : (NCC - CC_A) / 4 * 3;
            const int nb1 = nca * 4, nb2 = nca * nblk;
            const bool relay = !same && ((c ^ rank) & 3) != 0;
            const int B_XO = B_WB + 4 * (NCC - CC_A) - (same ? 4 : 0), B_OUT = B_XO + 16;
            const unsigned nunits = (B_OUT + (nca + ncb) * nblk + (relay ? 4 : 0)) * PARTS;
            char* const q = ws + OFF_Q + ps * 128;
            u64 exm = 0;
            u64 r0 = 0, r1 = 0, r2 = 0, r3 = 0;
            const unsigned ol = (lane >> 5) * (NTOT * 4) + (lane & 31) * 16;
            bool dead = false;
            auto decode = [&](unsigned un, unsigned& so, unsigned& dof, int& ms, int& need) {
                const int part = un % PARTS;
                int t = un / PARTS;
                const bool rl = relay && t >= B_OUT + nb1 && t < B_OUT + nb1 + 4;
                if (t < B_OUT || rl) {
                    if (rl) {
                        const int i = t - B_OUT - nb1, src = c ^ 4;
                        ms = (CC_R + (c & 3)) * 8 + src;
                        need = NEED_RELAY;
                        so = OFF_WS + par * WS_PAR + src * WS_SRC + (CC_R + (rank & 3)) * 65536 + i * 16384;
                        dof = OFF_WS + par * WS_PAR + src * WS_SRC + (CC_R + (c & 3)) * 65536 + i * 16384;
                    } else if ((t >= B_WA && t < B_XB) || (t >= B_WB && t < B_XO)) {
                        int i = t < B_XB ? t - B_WA : t - B_WB + 4 * CC_A;
                        if (same && i >= 4 * (CC_R + (c & 3))) i += 4;
                        const int cc = i >> 2;
                        ms = cc * 8 + c;
                        need = NEED_EX + cc;
                        so = OFF_EX + par * EX_PAR + chunk_col(rank, cc) * 512 + (i & 3) * 16384;
                        dof = OFF_WS + par * WS_PAR + c * WS_SRC + i * 16384;
                    } else {
                        int sl = t < B_WB ? xslot(t < B_WA ? t + NPS : t - B_XB + NXA, dc) : t - B_XO + SLOT_OWN;
                        ms = (sl < SLOT_A ? MS_XA : sl < SLOT_B2 ? MS_XB1 : sl < SLOT_B3 ? MS_XB2 : sl < SLOT_OWN ? MS_XB3 : MS_XO) * 8 + c;
                        const int row = slot_row(rank, sl);
                        need = NEED_EX + 32 + (row >> 7);
                        so = OFF_EX + par * EX_PAR + W_BYTES + row * 512;
                        dof = OFF_XS + par * XS_PAR + c * XS_SRC + sl * 16384;
                    }
                    so += part * (PULL_U * 1024);
                    dof += part * (PULL_U * 1024);
                    return;
                }
                t -= B_OUT;
                if (relay && t >= nb1) t -= 4;
                int ci, blk;
                if (t < nb1) { ci = t >> 2; blk = t & 3; }
                else if (t < nb2) { blk = t / nca; ci = t - blk * nca; }
                else { ci = t / nblk; blk = t - ci * nblk; }
                int cc = ci;
                if (!same) { int sub = ci / 3, gi = ci - sub * 3; cc = sub * 4 + (gi < (rank & 3) ? gi : gi + 1); }
                const int e = entry_of(cc & 3, d, blk);
                ms = -1;
                need = cc * 8 + (e >> 3);
                so = OFF_OS + (cc * 48 + e) * 16384 + part * (PULL_U * 1024);
                dof = (unsigned)(row0 + blk * 32 + part * (2 * PULL_U)) * (NTOT * 4) + chunk_col(c, cc) * 4;
            };
            auto put = [&](v4u (&v)[PULL_U], unsigned dof, int ms) {
                if (ms < 0) {
                    const float sc = dead ? __builtin_nanf("") : scale;
#pragma unroll
                    for (int u = 0; u < PULL_U; ++u) {
                        v4f o;
#pragma unroll
                        for (int j = 0; j < 4; ++j) o[j] = silu((int)v[u][j], sc);
                        st128<AUX_SYS>(__builtin_bit_cast(v4u, o), ors, dof + ol + u * 2 * (NTOT * 4));
                    }
                } else {
                    const auto rs = rsrc(ws, WS_NEED);
#pragma unroll
                    for (int u = 0; u < PULL_U; ++u) st128<AUX_SYS>(v[u], rs, dof + lin + u * 1024);
                }
            };
            auto signal = [&](int ms) {
                if (ms >= 0 && lane == 0)
                    __hip_atomic_fetch_add((gu32*)(ws + OFF_MS + ms * 128), (unsigned)PULL_U, __ATOMIC_RELAXED, __HIP_MEMORY_SCOPE_AGENT);
            };
            auto masks = [&](v4u f) {
                r0 = ballot((int)(f[0] - call) >= 0); r1 = ballot((int)(f[1] - call) >= 0);
                r2 = ballot((int)(f[2] - call) >= 0); r3 = ballot((int)(f[3] - call) >= 0);
            };
            auto have = [&](int k) {
                if (k >= NEED_EX) return rfl((int)((exm >> (k - NEED_EX)) & 1)) != 0;
                const int sh = k >> 2;
                const unsigned m = (unsigned)((r0 >> sh) & 1) | (unsigned)((r1 >> sh) & 1) << 1 | (unsigned)((r2 >> sh) & 1) << 2 | (unsigned)((r3 >> sh) & 1) << 3;
                return rfl((int)((m >> (k & 3)) & 1)) != 0;
            };
            auto await = [&](int k) {
                const u64 t0 = __builtin_amdgcn_s_memrealtime();
                unsigned round = 1;
                for (;;) {
                    asm volatile("" ::: "memory");
                    if (k >= NEED_EX) {
                        exm = ballot((int)(ld_sys(exf) - call) >= 0);
                    } else {
                        masks(__builtin_amdgcn_raw_buffer_load_b128(frs, lin, 0, AUX_SYS));
                    }
                    if (have(k)) break;
                    if (expired(ws, call, t0, round)) { dead = true; break; }
                    __builtin_amdgcn_s_sleep(1);
                }
            };
            constexpr int NONE = -1000;
            unsigned un1 = (7 - wid) * 8 + xcd;
            constexpr unsigned PER = NL * TAU, TOL = PER * LATE / 16;
            const unsigned phase = un1 * TAU;
            unsigned tnext = (tstart << 8) - 4 * PER;
            unsigned so, dof, pdof = 0;
            int ms, need, pms = NONE, ppms = NONE;
            v4u pv[PULL_U], v2[PULL_U];
            bool pok = false;
            const char* const pcnt = ws + OFF_MS + MS_PART * 1024;
            decode(un1, so, dof, ms, need);
            un1 += NL;
            for (int u = 0; u < PULL_U; ++u) pv[u] = v2[u] = v4u{0, 0, 0, 0};
            auto add2 = [&]() {
#pragma unroll
                for (int u = 0; u < PULL_U; ++u) { pv[u] += v2[u]; asm volatile("" : "+v"(pv[u])); }
            };
            auto flush = [&]() {
                add2();
                if (pms != NONE) put(pv, pdof, pms);
                signal(ppms);
                if (pms >= 0) { WAIT_VM0(); signal(pms); }
                pms = ppms = NONE;
            };
            for (;;) {
                if (!dead && (!have(need) || (ms < 0 && !pok))) {
                    flush();
                    if (!have(need)) await(need);
                    if (ms < 0 && !pok && !dead) { dead = !wait_words<1>(ws, call, lane, 1, pcnt, call * 192u); pok = true; }
                }
                {
                    const unsigned now = (unsigned)__builtin_amdgcn_s_memrealtime() << 8;
                    if (now - tnext + PER >= PER + TOL) tnext = (now - TOL - phase + PER) / PER * PER + phase;
                    for (int i = 0; i < 256 && (int)(((unsigned)__builtin_amdgcn_s_memrealtime() << 8) - tnext) < 0; ++i) __builtin_amdgcn_s_sleep(1);
                    tnext += PER;
                }
                asm volatile("" ::: "memory");
                v4u v[PULL_U], f;
                unsigned fe, fp;
#pragma unroll
                for (int u = 0; u < PULL_U; ++u) v[u] = __builtin_amdgcn_raw_buffer_load_b128(prs, lin, so + u * 1024, AUX_SYS);
                add2();
                if (pms != NONE) put(pv, pdof, pms);
                signal(ppms);
                const unsigned nxt = ticket(q, lane);
                ppms = pms; pms = ms; pdof = dof;
                const bool more = un1 < nunits;
                if (more) decode(un1, so, dof, ms, need);
                const bool miss = more && !have(need);
                const bool ref_e = miss && need >= NEED_EX, ref_r = miss && need < NEED_EX;
                if (ref_e) fe = ld_sys(exf);
                if (ref_r) f = __builtin_amdgcn_raw_buffer_load_b128(frs, lin, 0, AUX_SYS);
                const bool ref_p = more && ms < 0 && !pok;
                if (ref_p) fp = ld_sys(pcnt);
                asm volatile("" ::: "memory");
#pragma unroll
                for (int u = 0; u < PULL_U; ++u)
                    v2[u] = __builtin_amdgcn_raw_buffer_load_b128(ors, ol, pms < 0 ? pdof + u * 2 * (NTOT * 4) : OUT_BYTES, AUX_SYS);
                asm volatile("s_waitcnt vmcnt(%0)" :: "n"(PULL_U) : "memory");
#pragma unroll
                for (int u = 0; u < PULL_U; ++u) pv[u] = v[u];
                if (!more) break;
                if (ref_e) exm = ballot((int)(fe - call) >= 0);
                if (ref_r) masks(f);
                if (ref_p) pok = rfl((int)(fp - call * 192u)) >= 0;
                un1 = (unsigned)rfl((int)nxt) - (call - 1u) * nunits + 2 * NL;
            }
            flush();
        }
        return;
    }

    call = (unsigned)rfl((int)callv) + 1u;
    const unsigned par = call & 1u;
    const int li = job >> 3;
    const float scale = P.sx[0] * P.sw[0];
    const int wm = wid >> 1, wn = wid & 1, fr = lane & 15, fq = lane >> 4;
    const unsigned lds0 = (unsigned)(size_t)(LDS_AS char*)smem;
    volatile LDS_AS int* const dead_w = (volatile LDS_AS int*)((LDS_AS char*)smem + 3 * STAGE);
    unsigned xa[2], wa[4];
    for (int kh = 0; kh < 2; ++kh) xa[kh] = (wm * 64 + fr) * 128 + (((kh * 4 + fq) ^ (fr >> 1)) << 4);
    for (int ni = 0; ni < 4; ++ni)
        wa[ni] = 32768 + (fq * 16 + (fr >> 1)) * 128 + (((wn * 4 + ni) ^ (((fr >> 2) & 3) | ((fq & 1) << 2))) << 4) + 8 * (fr & 1);

    const int tend = (xcd < 4 && li < 16) ? 4 : 3;
    for (int ti = xcd < 4 ? 2 : 0; ti < tend; ++ti) {
        int cc = 0, rt, col;
        if (ti < 2) {
            const int sub = ti * 4 + xcd - 4, i12 = li >> 1, me = rank & 3;
            const int gid = (rank & 4) ? (i12 / 3) * 4 + (i12 % 3 < me ? i12 % 3 : i12 % 3 + 1) : (i12 < 4 * me ? i12 : i12 + 4);
            rt = 6 + (li & 1);
            col = gid * 1024 + sub * 128;
            while ((unsigned)__builtin_amdgcn_s_memrealtime() - tstart < PART_T0 + sub * PART_DT) __builtin_amdgcn_s_sleep(8);
        } else {
            if (ti == 3) { int idx = xcd * 16 + li; cc = idx >> 1; rt = 6 + (idx & 1); }
            else if (xcd < 2) { cc = li / 3; rt = xcd * 3 + li % 3; }
            else { cc = xcd * 4 + li / 6; rt = li % 6; }
            col = chunk_col(rank, cc);
        }
        cc = rfl(cc); rt = rfl(rt); col = rfl(col);
        const int g = cc & 3;
        bool dead = false;

        v4i acc[4][4];
#pragma unroll
        for (int i = 0; i < 4; ++i)
#pragma unroll
            for (int j = 0; j < 4; ++j) acc[i][j] = v4i{0, 0, 0, 0};

#pragma unroll 1
        for (int seg = 0; seg < (ti < 2 ? 1 : 2); ++seg) {
            if (seg == 1) {
                if (wid == 0) {
                    const int m1 = rt < 3 ? MS_XA : rt < 5 ? MS_XB1 : rt < 6 ? MS_XB3 : MS_XO;
                    const unsigned n1 = rt < 3 ? NXA : rt < 5 ? 12 : rt < 6 ? (((lane ^ rank) & 4) ? 6 : 8) : 16;
                    const u64 peers = 0xffull & ~(1ull << rank);
                    const char* mb = ws + OFF_MS + (lane & 7) * 128;
                    bool ok = wait_words<64>(ws, call, lane, peers, mb + cc * 1024, call * 64u);
                    if (ok) ok = wait_words<64>(ws, call, lane, (rt == 4 || rt == 5) ? peers | peers << 8 : peers,
                                               mb + (lane & 8 ? MS_XB2 : m1) * 1024, call * 16u * (lane & 8 ? 6 : n1));
                    __builtin_amdgcn_fence(__ATOMIC_ACQUIRE, "");
                    *dead_w = ok ? 0 : 1;
                    WAIT_VM0();
                }
                __syncthreads();
                dead = rfl(*dead_w) != 0;
                if (dead) break;
            }
            unsigned xo[4], wo[2];
            int own[4];
#pragma unroll
            for (int i = 0; i < 4; ++i) {
                int row = i * 64 + wid * 8 + (lane >> 3), e = rt * 8 + i * 2 + (wid >> 2), d, slot;
                if (rt >= 6) { slot = SLOT_OWN + e - 48; own[i] = -1; } else { entry_inv(g, e, d, slot); own[i] = seg ? dest_rank(rank, d) : -1; }
                xo[i] = (seg ? slot * 32 : slot_row(rank, slot)) * 512 + (row & 31) * 512 + (((lane & 7) ^ ((row >> 1) & 7)) << 4);
            }
#pragma unroll
            for (int i = 0; i < 2; ++i) {
                int k = i * 64 + wid * 8 + (lane >> 3);
                wo[i] = k * (seg ? 128 : NTOT) + (((lane & 7) ^ (((k >> 1) & 3) | (((k >> 4) & 1) << 2))) << 4);
            }
            const auto xrs = rsrc(seg ? ws : px, seg ? WS_NEED : X_BYTES);
            const auto wrs = rsrc(seg ? ws : pw, seg ? WS_NEED : W_BYTES);
            const unsigned xs = seg ? OFF_XS + par * XS_PAR : 0;
            const unsigned wsb = seg ? OFF_WS + par * WS_PAR + cc * 65536 : col;
            const unsigned wstep = seg ? 16384 : 128 * NTOT;
            const int nt = seg ? 28 : 4;
            auto stage = [&](int t, int buf) {
                int p = t >> 2;
                p = seg ? p + (p >= rank) : 0;
                const unsigned xb = xs + p * XS_SRC + (t & 3) * 128;
                const unsigned wb = wsb + p * WS_SRC + (t & 3) * wstep;
                const unsigned l = lds0 + buf * STAGE + wid * 1024;
                const int x0 = rfl(own[0] == p ? WS_NEED : xb), x1 = rfl(own[1] == p ? WS_NEED : xb);
                const int x2 = rfl(own[2] == p ? WS_NEED : xb), x3 = rfl(own[3] == p ? WS_NEED : xb);
                unsigned keep;
                asm volatile(
                    "s_nop 4\n\ts_mov_b32 %0, m0\n\t"
                    "s_mov_b32 m0, %1\n\ts_nop 0\n\tbuffer_load_dwordx4 %9, %2, %4 offen lds\n\t"
                    "s_add_u32 m0, m0, 0x2000\n\ts_nop 0\n\tbuffer_load_dwordx4 %10, %2, %5 offen lds\n\t"
                    "s_add_u32 m0, m0, 0x2000\n\ts_nop 0\n\tbuffer_load_dwordx4 %11, %2, %6 offen lds\n\t"
                    "s_add_u32 m0, m0, 0x2000\n\ts_nop 0\n\tbuffer_load_dwordx4 %12, %2, %7 offen lds\n\t"
                    "s_add_u32 m0, m0, 0x2000\n\ts_nop 0\n\tbuffer_load_dwordx4 %13, %3, %8 offen lds\n\t"
                    "s_add_u32 m0, m0, 0x2000\n\ts_nop 0\n\tbuffer_load_dwordx4 %14, %3, %8 offen lds\n\t"
                    "s_mov_b32 m0, %0"
                    : "=&s"(keep)
                    : "s"(l), "s"(xrs), "s"(wrs), "s"(x0), "s"(x1), "s"(x2), "s"(x3), "s"(wb),
                      "v"(xo[0]), "v"(xo[1]), "v"(xo[2]), "v"(xo[3]), "v"(wo[0]), "v"(wo[1])
                    : "memory", "scc");
            };
            v4i xf[2][4], wf[2][4];
            auto frags = [&](int buf, int kh) {
                LDS_AS char* sb = (LDS_AS char*)smem + buf * STAGE;
#pragma unroll
                for (int ni = 0; ni < 4; ++ni) {
                    v2i lo = __builtin_amdgcn_ds_read_tr8_b64_v2i32((LDS_AS v2i*)(sb + wa[ni] + kh * 8192));
                    v2i hi = __builtin_amdgcn_ds_read_tr8_b64_v2i32((LDS_AS v2i*)(sb + wa[ni] + kh * 8192 + 1024));
                    wf[kh][ni] = v4i{lo[0], lo[1], hi[0], hi[1]};
                }
#pragma unroll
                for (int mi = 0; mi < 4; ++mi) xf[kh][mi] = *(LDS_AS v4i*)(sb + xa[kh] + mi * 2048);
            };
            auto mma = [&](int kh) {
#pragma unroll
                for (int mi = 0; mi < 4; ++mi)
#pragma unroll
                    for (int ni = 0; ni < 4; ++ni)
                        acc[mi][ni] = __builtin_amdgcn_mfma_i32_16x16x64_i8(wf[kh][ni], xf[kh][mi], acc[mi][ni], 0, 0, 0);
            };
            WAIT_VM0();
            stage(0, 0);
            stage(1, 1);
            stage(2, 2);
            asm volatile("s_waitcnt vmcnt(12)" ::: "memory");
            __builtin_amdgcn_s_barrier();
            asm volatile("" ::: "memory");
            frags(0, 0);
            int buf = 0;
            for (int t = 0; t < nt; ++t) {
                frags(buf, 1);
                __builtin_amdgcn_sched_barrier(0);
                mma(0);
                __builtin_amdgcn_sched_barrier(0);
                if (t < nt - 2) asm volatile("s_waitcnt vmcnt(6) lgkmcnt(0)" ::: "memory");
                else asm volatile("s_waitcnt vmcnt(0) lgkmcnt(0)" ::: "memory");
                __builtin_amdgcn_s_barrier();
                asm volatile("" ::: "memory");
                if (t < nt - 3) stage(t + 3, buf);
                buf = buf == 2 ? 0 : buf + 1;
                if (t < nt - 1) frags(buf, 0);
                __builtin_amdgcn_sched_barrier(0);
                mma(1);
                __builtin_amdgcn_sched_barrier(0);
            }
        }
        {
            const bool loc = rt >= 6;
            const float sc = dead ? __builtin_nanf("") : scale;
            const auto rs = loc ? rsrc(P.out, OUT_BYTES) : rsrc(ws, WS_NEED);
            const unsigned pitch = loc ? NTOT * 4 : 512;
            const unsigned base = loc ? (unsigned)(rt - 6) * 256 * (NTOT * 4) + col * 4 : OFF_OS + (cc * 6 + rt) * 131072;
            const unsigned lo = (wm * 64 + fr) * pitch + wn * 256 + fq * 16;
            if (ti == 3) {
#pragma unroll
                for (int mi = 0; mi < 4; ++mi)
#pragma unroll
                    for (int ni = 0; ni < 4; ++ni)
#pragma unroll
                        for (int j = 0; j < 4; ++j) acc[mi][ni][j] = __builtin_bit_cast(int, silu(acc[mi][ni][j], sc));
            }
#pragma unroll
            for (int mi = 0; mi < 4; ++mi)
#pragma unroll
                for (int ni = 0; ni < 4; ++ni)
                    st128<AUX_SYS>(__builtin_bit_cast(v4u, acc[mi][ni]), rs, lo + base + mi * 16 * pitch + ni * 64);
            WAIT_VM0();
            __syncthreads();
            if (ti < 2 && threadIdx.x == 0)
                __hip_atomic_fetch_add((gu32*)(ws + OFF_MS + MS_PART * 1024), 1u, __ATOMIC_RELAXED, __HIP_MEMORY_SCOPE_AGENT);
            if (ti == 2 && threadIdx.x < (rt < 5 ? 6 : 3)) {
                int d, slot;
                entry_inv(g, rt * 8 + (rt < 5 ? 0 : 2) + threadIdx.x, d, slot);
                st_sys(P.ws[dest_rank(rank, d) & 7] + OFF_RDY + ((rank * NCC + cc) * 8 + rt) * 4, call);
                if (dead) st_sys(P.out + col * 4, 0x7fc00000u);
            }
        }
    }
}

}

extern "C" void kernel_launch(int rank, int nranks, ncclComm_t comm,
                              hipStream_t stream,
                              void* const* d_in, const int* in_sizes, int n_in,
                              void* d_out, int out_size,
                              const int* in_dtypes, int out_dtype,
                              void* const* d_out_all,
                              void* const* d_ws_all, size_t ws_bytes) {
    if (nranks != NR || (unsigned)rank >= (unsigned)NR || ws_bytes < WS_NEED || n_in != 4 ||
        in_sizes[0] != (int)X_BYTES || in_sizes[1] != (int)W_BYTES || out_size != 512 * NTOT) {
        fprintf(stderr, "unexpected problem shape\n");
        return;
    }
    Params p;
    p.x = (const char*)d_in[0]; p.w = (const char*)d_in[1];
    p.sx = (const float*)d_in[2]; p.sw = (const float*)d_in[3];
    p.out = (char*)d_out;
    for (int i = 0; i < NR; ++i) { p.ws[i] = (char*)d_ws_all[phys(i)]; if (phys(i) == rank) p.rank = i; }
    for (int i = 0; i < NR; ++i) p.wsr[i] = p.ws[(p.rank + i) & 7];
    hipLaunchKernelGGL(fused_gemm_scatter, dim3(NWG), dim3(512), 0, stream, p);
    HIP_CHECK(hipGetLastError());
}
```
